```python
import numpy as np

kernel_name = "gpu_gemm_rs_m4096_k4096_n16384_bf16_silu"


ABSMAX_RTOL = 0.008

COMPUTE_DTYPE = "bfloat16"
EPILOGUE = "silu"
IN_QUANT = None
OUT_QUANT = None
IN_DTYPES = {"x": "bfloat16", "w_mat": "bfloat16"}
WORLD_SIZE = 8

def _to_bf16(x):
    u = np.ascontiguousarray(x, dtype=np.float32).view(np.uint32)
    return ((u + 0x7FFF + ((u >> 16) & 1)) & 0xFFFF0000).view(np.float32)

def setup_inputs():

    x = _to_bf16(np.random.randn(4096, 4096).astype(np.float32))
    w_mat = _to_bf16((np.random.randn(4096, 16384) / np.sqrt(4096)).astype(np.float32))
    return {"x": x, "w_mat": w_mat}

def _epilogue(y):
    return (y / (1.0 + np.exp(-np.clip(y, -60.0, 60.0)))).astype(np.float32)

def reference(x, w_mat):
    y = (x @ w_mat).astype(np.float32)
    return _epilogue(y)
```

```cpp
#include <hip/hip_runtime.h>
#include <rccl/rccl.h>
#include <cstdio>
#include <cstdint>

#define HIP_CHECK(expr) do { hipError_t _e = (expr); if (_e != hipSuccess) { \
    fprintf(stderr, "HIP error %s at %s:%d\n", hipGetErrorString(_e), __FILE__, __LINE__); \
    return; } } while (0)

#define LAS __attribute__((address_space(3)))
#define GAS __attribute__((address_space(1)))

namespace {

typedef __bf16 bf16x8 __attribute__((ext_vector_type(8)));
typedef __bf16 bf16x2 __attribute__((ext_vector_type(2)));
typedef short s16x4 __attribute__((ext_vector_type(4)));
typedef short s16x8 __attribute__((ext_vector_type(8)));
typedef float f32x4 __attribute__((ext_vector_type(4)));
typedef float f32x2 __attribute__((ext_vector_type(2)));
typedef unsigned u32x4 __attribute__((ext_vector_type(4)));
typedef unsigned u32x2 __attribute__((ext_vector_type(2)));

constexpr int NR = 8;
constexpr unsigned XROW = 1024u;
constexpr unsigned WROW = 32768u;
constexpr unsigned X_BYTES = 4096u * XROW;
constexpr unsigned W_BYTES = 512u * WROW;
constexpr unsigned OUT_BYTES = 512u * WROW;
#define NPUSH 4
constexpr int NPW = NPUSH;
constexpr unsigned NPWAVE = NPW * 8u, NBATCH = 1408u;
constexpr int NTILE = 26;
constexpr int NPIECE = 24;
constexpr int GRID = 256;

#define LAG 3
#define YD 0
#define YD1 1
#define PUSH_D 3

constexpr unsigned OFF_CTR = 0u;
constexpr unsigned OFF_TMO = 32768u;
constexpr unsigned OFF_TKT = 40960u;
constexpr unsigned OFF_FIN = 65536u;
constexpr unsigned OFF_FOUT = 131072u;
constexpr unsigned OFF_DBG = 262144u;
constexpr unsigned OFF_XS = 1u << 20;
constexpr unsigned XS_SRC = 4u << 20, XS_PAR = 8u * XS_SRC;
constexpr unsigned OFF_WS = OFF_XS + 2u * XS_PAR;
constexpr unsigned WS_SRC = 2u << 20, WS_PAR = 8u * WS_SRC;
constexpr unsigned OFF_YP = OFF_WS + 2u * WS_PAR;
constexpr unsigned YP_WAVE = 10240u, YP_TILE = 8u * YP_WAVE, YP_SRC = NTILE * YP_TILE;
constexpr size_t WS_NEED = (size_t)OFF_YP + (size_t)NR * YP_SRC;

constexpr unsigned STAGE = 53248u;
constexpr unsigned LDS_CTL = 3u * STAGE;
constexpr unsigned LDS_BYTES = LDS_CTL + 256u;

constexpr unsigned long long DEADLINE_TICKS = 100000000ull;

struct Params {
    const char* x;
    const char* w;
    char* out;
    char* ws[NR];
    int rank;
    unsigned qmap;
};

__device__ __forceinline__ unsigned rfl(unsigned v) { return (unsigned)__builtin_amdgcn_readfirstlane((int)v); }

__device__ __forceinline__ u32x4 mk_rsrc(const void* p, unsigned bytes) {
    const unsigned long long a = (unsigned long long)p;
    u32x4 r; r.x = (unsigned)a; r.y = (unsigned)(a >> 32) & 0xffffu; r.z = bytes; r.w = 0x00020000u; return r;
}
__device__ __forceinline__ __amdgpu_buffer_rsrc_t mk_brsrc(const void* p, unsigned bytes) {
    return __builtin_amdgcn_make_buffer_rsrc((void*)p, 0, (int)bytes, 0x00020000);
}

__device__ __forceinline__ void dma16(u32x4 rs, unsigned voff, unsigned soff, unsigned ldsaddr) {
    asm volatile("s_mov_b32 m0, %0\n\ts_nop 0\n\tbuffer_load_dwordx4 %1, %2, %3 offen lds"
                 :: "s"(ldsaddr), "v"(voff), "s"(rs), "s"(soff) : "memory");
}
#define WAIT_V(n) asm volatile("s_waitcnt vmcnt(%0)" :: "i"(n) : "memory")
#define BAR() do { asm volatile("" ::: "memory"); __builtin_amdgcn_s_barrier(); asm volatile("" ::: "memory"); } while (0)

__device__ __forceinline__ unsigned ld_sys(const char* p) {
    return __hip_atomic_load((const GAS unsigned*)p, __ATOMIC_RELAXED, __HIP_MEMORY_SCOPE_SYSTEM);
}
__device__ __forceinline__ void st_sys(char* p, unsigned v) {
    __hip_atomic_store((GAS unsigned*)p, v, __ATOMIC_RELAXED, __HIP_MEMORY_SCOPE_SYSTEM);
}
__device__ __forceinline__ unsigned ld_agent(const char* p) {
    return __hip_atomic_load((const GAS unsigned*)p, __ATOMIC_RELAXED, __HIP_MEMORY_SCOPE_AGENT);
}
__device__ __forceinline__ void st_agent(char* p, unsigned v) {
    __hip_atomic_store((GAS unsigned*)p, v, __ATOMIC_RELAXED, __HIP_MEMORY_SCOPE_AGENT);
}

constexpr int PBASE = 3 * (8 - LAG);
__device__ __forceinline__ unsigned piece_of(unsigned s, unsigned h) {
    return s < 8u - LAG ? 3u * s + 1u + h : (h == 0u ? PBASE + 2u * (s - (8u - LAG)) + 1u : PBASE + 2u * LAG + (s - (8u - LAG)));
}
__device__ __forceinline__ unsigned pk_bf16(float lo, float hi) {
    f32x2 v = {lo, hi};
    bf16x2 b = __builtin_convertvector(v, bf16x2);
    return __builtin_bit_cast(unsigned, b);
}
__device__ __forceinline__ float silu(float y) {
    const float c = __builtin_amdgcn_fmed3f(y, -60.0f, 60.0f);
    const float e = __builtin_amdgcn_exp2f(c * -1.44269504088896341f);
    return y * __builtin_amdgcn_rcpf(1.0f + e);
}

#define STAMP(i) do { } while (0)

struct Batch { unsigned piece, need, isw, so, dofs; };
__device__ __forceinline__ Batch batch_of(unsigned T, unsigned rank, unsigned q) {
    unsigned s, r, type, piece;
    constexpr unsigned B0 = 176u * (8u - LAG), B1 = B0 + 120u * LAG;
    if (T < B0)      { s = T / 176u; r = T - 176u * s; type = r < 64u ? 0u : (r < 120u ? 1u : 2u); piece = 3u * s + type; }
    else if (T < B1) { const unsigned u = (T - B0) / 120u; r = T - B0 - 120u * u; s = 8u - LAG + u; type = r < 64u ? 0u : 1u; piece = PBASE + 2u * u + type; }
    else             { const unsigned u = (T - B1) / 56u; r = T - B1 - 56u * u + 120u; s = 8u - LAG + u; type = 2u; piece = PBASE + 2u * LAG + u; }
    Batch b; b.piece = piece; b.isw = type == 0u; b.need = type == 0u ? 64u : 56u;
    if (type == 0u) {
        b.so = (s * 64u + (r & 1u) * 32u) * WROW + q * 4096u + (r >> 1) * 128u;
        b.dofs = s * (256u << 10) + (r >> 1) * 8192u + (r & 1u) * 2048u;
    } else {
        const unsigned e = r - (type == 1u ? 64u : 120u), blk = ((rank + 1u + (e >> 3)) & 7u) * 64u + (type - 1u) * 32u + (e & 7u) * 4u;
        b.so = blk * (8u * XROW) + s * 128u;
        b.dofs = s * (512u << 10) + blk * 1024u;
    }
    b.piece = rfl(b.piece); b.need = rfl(b.need); b.isw = rfl(b.isw); b.so = rfl(b.so); b.dofs = rfl(b.dofs);
    asm("" : "+s"(b.piece), "+s"(b.need), "+s"(b.isw), "+s"(b.so), "+s"(b.dofs));
    return b;
}

__device__ __forceinline__ void pusher(const Params& P, unsigned q, unsigned j) {
    const unsigned lane = threadIdx.x & 63u, w = rfl(threadIdx.x >> 6), W = j * 8u + w;
    const unsigned rank = (unsigned)P.rank;
    char* const me = P.ws[rank];
    char* const peer = P.ws[q];
    const unsigned callv = ld_agent(me + OFF_CTR + blockIdx.x * 128u) + 1u;

    const unsigned vx = (lane >> 3) * XROW + (((lane & 7u) ^ (lane >> 4)) * 16u);
    const unsigned vw = (lane >> 2) * WROW + (((lane & 3u) ^ ((lane >> 5) << 1)) * 16u);
    const unsigned vd = lane * 16u;
    auto load = [&](const Batch& b, bool valid, int i) {
        const __amdgpu_buffer_rsrc_t rs = mk_brsrc(b.isw ? P.w : P.x, valid ? (b.isw ? W_BYTES : X_BYTES) : 0u);
        const unsigned v = b.isw ? vw : ((i & 1) ? (vx ^ 64u) : vx);
        const unsigned o = b.isw ? (i & 1) * 64u + (i >> 1) * (16u * WROW) : i * (8u * XROW);
        return __builtin_amdgcn_raw_buffer_load_b128(rs, v, b.so + o, 0);
    };
    Batch b0 = batch_of(W, rank, q), b1 = batch_of(NPWAVE + W, rank, q);
    bool v1 = true;
    u32x4 cur[4];
#pragma unroll
    for (int i = 0; i < 4; ++i) cur[i] = load(b0, true, i);
    const unsigned call = rfl(callv), par = call & 1u;
    __syncthreads();
    __builtin_amdgcn_s_waitcnt(0x0F70);
    if (threadIdx.x == 0u) st_agent(me + OFF_CTR + blockIdx.x * 128u, call);
    char* const dx = peer + OFF_XS + par * XS_PAR + rank * XS_SRC;
    char* const dw = peer + OFF_WS + par * WS_PAR + rank * WS_SRC;
    char* const flags = peer + OFF_FIN + rank * 128u;
    const __amdgpu_buffer_rsrc_t rc = mk_brsrc(me + OFF_TKT + q * 256u, 256u);
    const unsigned vc = lane == 0u ? 0u : 4096u;
    const unsigned tbase = (call - 1u) * NBATCH - 2u * NPWAVE;
    constexpr int PEND = 2 * PUSH_D;
    unsigned p1 = 31u, need1 = 0u, p2 = 31u, need2 = 0u;
    auto check = [&](unsigned res, unsigned piece, unsigned need) { if (rfl(res) + 1u == need && lane == 0u) st_sys(flags + piece * 4u, call); };
    STAMP(0);
    for (;;) {
        const unsigned dn = (unsigned)__builtin_amdgcn_raw_ptr_buffer_atomic_add_i32(1, rc, vc, 128u + p2 * 4u, 0);
        const unsigned tk = (unsigned)__builtin_amdgcn_raw_ptr_buffer_atomic_add_i32(1, rc, vc, 0, 0);
        asm volatile("" ::: "memory");
        const __amdgpu_buffer_rsrc_t rd = mk_brsrc(b0.isw ? dw : dx, b0.isw ? WS_SRC : XS_SRC);
#pragma unroll
        for (int i = 0; i < 4; ++i) {
            __builtin_amdgcn_raw_buffer_store_b128(cur[i], rd, vd, b0.dofs + (b0.isw ? (i & 1) * 4096u + (i >> 1) * 1024u : i * 1024u), 17);
            asm volatile("" ::: "memory");
            cur[i] = load(b1, v1, i);
            if (i == 3) WAIT_V(PEND < 4 ? PEND : 4); else if (i < PUSH_D) WAIT_V(PEND + 2); else WAIT_V(PEND);
            if (i == (PUSH_D < 3 ? PUSH_D : 3)) check(dn, p2, need2);
        }
        p2 = p1; need2 = need1; p1 = b0.piece; need1 = b0.need * call;
        if (!v1) break;
        const unsigned T = rfl(tk) - tbase;
        b0 = b1; v1 = T < NBATCH; b1 = batch_of(v1 ? T : 0u, rank, q);
    }
    const unsigned dn2 = (unsigned)__builtin_amdgcn_raw_ptr_buffer_atomic_add_i32(1, rc, vc, 128u + p2 * 4u, 0);
    WAIT_V(0);
    const unsigned dn1 = (unsigned)__builtin_amdgcn_raw_ptr_buffer_atomic_add_i32(1, rc, vc, 128u + p1 * 4u, 0);
    check(dn2, p2, need2);
    check(dn1, p1, need1);
    STAMP(1);
}

__device__ __forceinline__ void gemm_tile(const Params& P, unsigned q, unsigned tile, LAS char* lds) {
    const unsigned lane = threadIdx.x & 63u, w = rfl(threadIdx.x >> 6), wm = w >> 1, wn = w & 1u;
    const unsigned rank = (unsigned)P.rank, g = (q - rank - 1u) & 7u;
    const unsigned h = tile & 1u, c = tile >> 1;
    const bool own = g == 7u;
    char* const me = P.ws[rank];
    const unsigned call = rfl(ld_agent(me + OFF_CTR + blockIdx.x * 128u)) + 1u, par = call & 1u;
    const unsigned lds0 = (unsigned)(uintptr_t)lds;
    LAS unsigned* ctl = (LAS unsigned*)(lds + LDS_CTL);
    STAMP(0);

    const unsigned b0 = w < 4u ? 3u * w : 12u + 2u * (w - 4u);
    const unsigned cbmax = c == 12u ? 3u : 4u;
    u32x4 rsA = mk_rsrc(P.x, X_BYTES), rsB = mk_rsrc(P.w, W_BYTES);
    unsigned vA0 = (lane >> 3) * XROW + (((lane & 7u) ^ (lane >> 4)) * 16u), vA1 = vA0 ^ 64u;
    unsigned vB = (lane >> 2) * WROW + (((lane & 3u) ^ ((lane >> 5) << 1)) * 16u);
    unsigned sA[4], sB[3];
#pragma unroll
    for (int i = 0; i < 4; ++i) sA[i] = (4u * w + i) * (8u * XROW);
#pragma unroll
    for (int i = 0; i < 3; ++i) { const unsigned b = b0 + i, cb = min(b >> 2, cbmax); sB[i] = cb * 64u + (b & 3u) * (16u * WROW); }

    const unsigned nraw = own ? 8u : 16u, nsrc = own ? 7u : 6u;
    unsigned it = 0u, isrc = 0u, isl = 0u;
    auto next = [&](unsigned& oa, unsigned& ob) {
        if (it < nraw) {
            const bool extra = !own && it < 8u;
            const unsigned s = it & 7u;
            oa = ((extra ? rank : q) * 512u + h * 256u) * XROW + s * 128u;
            ob = s * 64u * WROW + (extra ? q : rank) * 4096u + c * 320u;
        } else {
            const unsigned p = (rank + 1u + isrc + (isrc >= g ? 1u : 0u)) & 7u;
            oa = p * XS_SRC + isl * (512u << 10) + (q * 2u + h) * 32768u;
            ob = p * WS_SRC + isl * (256u << 10) + c * 20480u;
            if (++isrc == nsrc) { isrc = 0u; ++isl; }
        }
        ++it;
    };
    const unsigned la0 = lds0 + w * 4096u, lb0 = lds0 + 32768u + b0 * 1024u;
    auto piece = [&](int i, u32x4 ra, u32x4 rb, unsigned oa, unsigned ob, unsigned so) {
        if (i < 4) dma16(ra, (i & 1) ? vA1 : vA0, oa + sA[i], la0 + so + i * 1024u);
        else if (i < 6) dma16(rb, vB, ob + sB[i - 4], lb0 + so + (i - 4) * 1024u);
        else if (w < 4u) dma16(rb, vB, ob + sB[2], lb0 + so + 2048u);
    };

    const unsigned g16 = lane >> 4;
    const unsigned aoff = wm * 8192u + (lane & 15u) * 128u + ((g16 ^ ((lane & 15u) >> 1)) * 16u);
    const unsigned lp = (g16 >> 1) * 1024u + (8u * (g16 & 1u) + ((lane & 15u) >> 2)) * 64u + (lane & 3u) * 8u;
    const unsigned be = 32768u + ((5u * wn) >> 1) * 4096u + lp + (((wn ^ g16) & 1u) * 32u);
    const unsigned bo = 32768u + ((5u * wn + 1u) >> 1) * 4096u + lp + (((wn ^ g16 ^ 1u) & 1u) * 32u);

    f32x4 acc[4][5], sav[4][5];
#pragma unroll
    for (int m = 0; m < 4; ++m)
#pragma unroll
        for (int n = 0; n < 5; ++n) { acc[m][n] = (f32x4){0.f, 0.f, 0.f, 0.f}; sav[m][n] = (f32x4){0.f, 0.f, 0.f, 0.f}; }
    s16x8 fa[2][4]; s16x4 fb[2][5][2];
    auto zero1 = [&]() {
#pragma unroll
        for (int m = 0; m < 4; ++m) fa[1][m] = (s16x8){0, 0, 0, 0, 0, 0, 0, 0};
#pragma unroll
        for (int n = 0; n < 5; ++n) { fb[1][n][0] = (s16x4){0, 0, 0, 0}; fb[1][n][1] = (s16x4){0, 0, 0, 0}; }
    };
    auto rd = [&](int kh, int r, LAS char* st) {
        if (r >= 2 && r < 6) fa[kh][r - 2] = *(const LAS s16x8*)(st + (kh ? (aoff ^ 64u) : aoff) + (r - 2) * 2048);
        else { const int n = r < 2 ? 0 : (r - 4) >> 1, hi = r & 1;
               fb[kh][n][hi] = __builtin_amdgcn_ds_read_tr16_b64_v4i16((LAS s16x4*)(st + ((n & 1) ? bo : be) + (n >> 1) * 4096 + kh * 2048 + hi * 256)); }
    };
    auto mm = [&](int kh, int i) {
        const int n = i >> 2, m = i & 3;
        acc[m][n] = __builtin_amdgcn_mfma_f32_16x16x32_bf16(
            __builtin_bit_cast(bf16x8, __builtin_shufflevector(fb[kh][n][0], fb[kh][n][1], 0, 1, 2, 3, 4, 5, 6, 7)),
            __builtin_bit_cast(bf16x8, fa[kh][m]), acc[m][n], 0, 0, 0);
    };
    zero1();

    unsigned t = 0u, ready = nraw, cso = 0u, rnd = 0u;
    bool bad = false;
    for (;;) {
        if (it == t) {
            unsigned oa, ob;
            next(oa, ob);
#pragma unroll
            for (int i = 0; i < 7; ++i) piece(i, rsA, rsB, oa, ob, cso);
            next(oa, ob);
            const unsigned so = cso == 2u * STAGE ? 0u : cso + STAGE;
#pragma unroll
            for (int i = 0; i < 7; ++i) piece(i, rsA, rsB, oa, ob, so);
        }
        const unsigned stop = (!own && t < 8u) ? 8u : ready;
        for (; t < stop; ++t) {
            unsigned oa = 0u, ob = 0u;
            u32x4 ra = rsA, rb = rsB;
            if (t + 2u < ready) next(oa, ob); else { ra.z = 0u; rb.z = 0u; }
            const unsigned so = cso == 0u ? 2u * STAGE : cso - STAGE;
            LAS char* st = lds + cso;
            asm volatile("s_waitcnt vmcnt(6) lgkmcnt(0)" ::: "memory");
            BAR();
            __builtin_amdgcn_sched_barrier(0);
#pragma unroll
            for (int i = 0; i < 20; ++i) {
                mm(1, i);
                if (i < 7) { rd(0, 2 * i, st); rd(0, 2 * i + 1, st); }
                if (i >= 8 && i < 16 && (i & 1) == 0) piece((i - 8) >> 1, ra, rb, oa, ob, so);
                __builtin_amdgcn_sched_barrier(0);
            }
#pragma unroll
            for (int i = 0; i < 20; ++i) {
                mm(0, i);
                if (i < 7) { rd(1, 2 * i, st); rd(1, 2 * i + 1, st); }
                if (i >= 8 && i < 14 && (i & 1) == 0) piece(4 + ((i - 8) >> 1), ra, rb, oa, ob, so);
                __builtin_amdgcn_sched_barrier(0);
            }
            cso = cso == 2u * STAGE ? 0u : cso + STAGE;
        }
        if (t == 64u || t < ready) {
#pragma unroll
            for (int i = 0; i < 20; ++i) mm(1, i);
            if (t == 64u) break;
#pragma unroll
            for (int m = 0; m < 4; ++m)
#pragma unroll
                for (int n = 0; n < 5; ++n) { sav[m][n] = acc[m][n]; acc[m][n] = (f32x4){0.f, 0.f, 0.f, 0.f}; }
            zero1();
            continue;
        }
        STAMP(1 + 2 * rnd);
        if (t == nraw) {
            if (threadIdx.x == 0u) st_agent(me + OFF_CTR + blockIdx.x * 128u, call);
            rsA = mk_rsrc(me + OFF_XS + par * XS_PAR, XS_PAR); rsB = mk_rsrc(me + OFF_WS + par * WS_PAR, WS_PAR);
            vA0 = lane * 16u; vA1 = lane * 16u; vB = lane * 16u;
#pragma unroll
            for (int i = 0; i < 4; ++i) sA[i] = (4u * w + i) * 1024u;
#pragma unroll
            for (int i = 0; i < 3; ++i) { const unsigned b = b0 + i, cb = min(b >> 2, cbmax); sB[i] = cb * 4096u + (b & 3u) * 1024u; }
        }
        if (w == 0u) {
            const unsigned i = lane & 7u, s = rnd + (lane >> 3);
            const unsigned p = (rank + 1u + i + (i >= g ? 1u : 0u)) & 7u;
            const char* fp = me + OFF_FIN + p * 128u + piece_of(min(s, 7u), h) * 4u;
            const unsigned long long t0 = __builtin_amdgcn_s_memrealtime();
            unsigned res;
            for (;;) {
                const unsigned v = ld_sys(fp);
                const unsigned long long miss = __ballot(s > 7u || (i < nsrc && (int)(v - call) < 0)) | (1ull << 63);
                res = (unsigned)__builtin_ctzll(miss) >> 3;
                if (res != 0u) break;
                if (rfl(ld_agent(me + OFF_TMO)) == call || __builtin_amdgcn_s_memrealtime() - t0 > DEADLINE_TICKS) {
                    if (lane == 0u) st_agent(me + OFF_TMO, call);
                    res = 0x101u; break;
                }
                __builtin_amdgcn_s_sleep(4);
            }
            if (lane == 0u) ctl[0] = res;
            asm volatile("s_waitcnt vmcnt(0) lgkmcnt(0)" ::: "memory");
        }
        BAR();
        const unsigned res = rfl(*(volatile LAS unsigned*)ctl);
        bad |= res > 0xffu;
        STAMP(2 + 2 * rnd);
        rnd += res & 0xffu; ready += (res & 0xffu) * nsrc;
    }
    STAMP(20);

    const unsigned nv = (c == 12u && wn == 1u) ? 3u : 5u;
    const float qnan = __builtin_nanf("");
    if (!own) {
        char* const peer = P.ws[q];
        const __amdgpu_buffer_rsrc_t ry = mk_brsrc(peer + OFF_YP + rank * YP_SRC + tile * YP_TILE + w * YP_WAVE, YP_WAVE);
#pragma unroll
        for (int n = 0; n < 5; ++n) if ((unsigned)n < nv) {
#pragma unroll
            for (int mp = 0; mp < 2; ++mp) {
                const f32x4 v0 = acc[2 * mp][n], v1 = acc[2 * mp + 1][n];
                u32x4 d; d.x = pk_bf16(v0[0], v0[1]); d.y = pk_bf16(v0[2], v0[3]); d.z = pk_bf16(v1[0], v1[1]); d.w = pk_bf16(v1[2], v1[3]);
                if (bad) d = (u32x4){0x7fc07fc0u, 0x7fc07fc0u, 0x7fc07fc0u, 0x7fc07fc0u};
                __builtin_amdgcn_raw_buffer_store_b128(d, ry, lane * 16u, (n * 2 + mp) * 1024, 17);
                if (h == 0u) WAIT_V(YD); else WAIT_V(YD1);
            }
        }
        WAIT_V(0);
        if (lane == 0u) st_sys(peer + OFF_FOUT + rank * 4096u + tile * 128u + w * 16u, call);
        STAMP(21);
        const char* fp = me + OFF_FOUT + q * 4096u + tile * 128u + w * 16u;
        const unsigned long long t0 = __builtin_amdgcn_s_memrealtime();
        while ((int)(rfl(ld_sys(fp)) - call) < 0) {
            if (rfl(ld_agent(me + OFF_TMO)) == call || __builtin_amdgcn_s_memrealtime() - t0 > DEADLINE_TICKS) {
                if (lane == 0u) st_agent(me + OFF_TMO, call);
                bad = true; break;
            }
            __builtin_amdgcn_s_sleep(2);
        }
        asm volatile("" ::: "memory");
        STAMP(22);
        const __amdgpu_buffer_rsrc_t rp = mk_brsrc(me + OFF_YP + q * YP_SRC + tile * YP_TILE + w * YP_WAVE, bad ? 0u : nv * 2048u);
        u32x4 pv[10];
#pragma unroll
        for (int u = 0; u < 10; ++u) pv[u] = __builtin_amdgcn_raw_buffer_load_b128(rp, lane * 16u + u * 1024u, 0, 17);
#pragma unroll
        for (int n = 0; n < 5; ++n)
#pragma unroll
            for (int mp = 0; mp < 2; ++mp) {
                const u32x4 d = pv[n * 2 + mp];
                f32x4 v0 = sav[2 * mp][n], v1 = sav[2 * mp + 1][n];
                v0[0] += __builtin_bit_cast(float, d.x << 16); v0[1] += __builtin_bit_cast(float, d.x & 0xffff0000u);
                v0[2] += __builtin_bit_cast(float, d.y << 16); v0[3] += __builtin_bit_cast(float, d.y & 0xffff0000u);
                v1[0] += __builtin_bit_cast(float, d.z << 16); v1[1] += __builtin_bit_cast(float, d.z & 0xffff0000u);
                v1[2] += __builtin_bit_cast(float, d.w << 16); v1[3] += __builtin_bit_cast(float, d.w & 0xffff0000u);
                acc[2 * mp][n] = v0; acc[2 * mp + 1][n] = v1;
            }
    }
    bad |= rfl(ld_agent(me + OFF_TMO)) == call;
    const __amdgpu_buffer_rsrc_t ro = mk_brsrc(P.out, OUT_BYTES);
    const unsigned vo = (lane & 15u) * WROW + g16 * 8u;
    const unsigned so = (h * 256u + wm * 64u) * WROW + q * 4096u + c * 320u + wn * 160u;
#pragma unroll
    for (int m = 0; m < 4; ++m)
#pragma unroll
        for (int n = 0; n < 5; ++n) if ((unsigned)n < nv) {
            const f32x4 v = acc[m][n];
            u32x2 d; d.x = pk_bf16(silu(v[0]), silu(v[1])); d.y = pk_bf16(silu(v[2]), silu(v[3]));
            if (bad) d = (u32x2){0x7fc07fc0u, 0x7fc07fc0u};
            __builtin_amdgcn_raw_buffer_store_b64(d, ro, vo + n * 32u, so + m * (16u * WROW), 0);
        }
    (void)qnan;
    STAMP(23);
}

__global__ void __launch_bounds__(512, 2) fused_gemm_rs(Params P) {
    __shared__ __attribute__((aligned(16))) char smem[LDS_BYTES];
    LAS char* lds = (LAS char*)smem;
    const unsigned q = (P.qmap >> ((blockIdx.x & 7u) * 4u)) & 7u, slot = blockIdx.x >> 3;
    if (slot < (unsigned)NPW) { if (q != (unsigned)P.rank) pusher(P, q, slot); }
    else if (slot < (unsigned)(NPW + NTILE)) gemm_tile(P, q, slot - NPW, lds);
}

unsigned make_qmap(int rank) {
    unsigned rot = 0u;
    for (int g = 0; g < NR; ++g) rot |= (unsigned)((rank + 1 + g) & 7) << (4 * g);
    static const int kBus[NR] = {0x72, 0x0a, 0x5a, 0x23, 0xf1, 0x8b, 0xd9, 0xa4};
    static const unsigned char kNear[NR][NR] = {
        {6, 0, 2, 3, 1, 5, 4, 7}, {7, 1, 4, 5, 0, 3, 6, 2}, {3, 2, 0, 6, 4, 5, 7, 1}, {2, 3, 6, 0, 5, 1, 7, 4},
        {5, 4, 1, 7, 2, 0, 6, 3}, {4, 5, 7, 1, 3, 2, 0, 6}, {0, 6, 3, 2, 7, 4, 1, 5}, {1, 7, 5, 4, 6, 3, 2, 0}};
    int dev = -1, ndev = 0, pos_of[NR], dev_at[NR];
    if (hipGetDevice(&dev) != hipSuccess || hipGetDeviceCount(&ndev) != hipSuccess || dev != rank || ndev != NR) { (void)hipGetLastError(); return rot; }
    for (int i = 0; i < NR; ++i) dev_at[i] = -1;
    for (int d = 0; d < NR; ++d) {
        int bus = -1;
        if (hipDeviceGetAttribute(&bus, hipDeviceAttributePciBusId, d) != hipSuccess) { (void)hipGetLastError(); return rot; }
        pos_of[d] = -1;
        for (int i = 0; i < NR; ++i) if (kBus[i] == bus) pos_of[d] = i;
        if (pos_of[d] < 0 || dev_at[pos_of[d]] >= 0) return rot;
        dev_at[pos_of[d]] = d;
    }
    unsigned m = 0u;
    for (int g = 0; g < NR; ++g) m |= (unsigned)dev_at[kNear[pos_of[rank]][g]] << (4 * g);
    return m;
}

}

extern "C" void kernel_launch(int rank, int nranks, ncclComm_t comm,
                              hipStream_t stream,
                              void* const* d_in, const int* in_sizes, int n_in,
                              void* d_out, int out_size,
                              const int* in_dtypes, int out_dtype,
                              void* const* d_out_all,
                              void* const* d_ws_all, size_t ws_bytes) {
    (void)comm; (void)in_dtypes; (void)out_dtype; (void)d_out_all;
    if (nranks != NR || (unsigned)rank >= (unsigned)NR || n_in != 2 || ws_bytes < WS_NEED ||
        in_sizes[0] != 4096 * 512 || in_sizes[1] != 512 * 16384 || out_size != 512 * 16384) {
        fprintf(stderr, "unexpected problem geometry\n");
        return;
    }
    Params P;
    P.x = (const char*)d_in[0];
    P.w = (const char*)d_in[1];
    P.out = (char*)d_out;
    for (int p = 0; p < NR; ++p) P.ws[p] = (char*)d_ws_all[p];
    P.rank = rank;
    P.qmap = make_qmap(rank);
    fused_gemm_rs<<<GRID, 512, 0, stream>>>(P);
    HIP_CHECK(hipGetLastError());
}
```

```cpp
#include <hip/hip_runtime.h>
#include <rccl/rccl.h>
#include <cstdio>
#include <cstdint>

#define HIP_CHECK(expr) do { hipError_t _e = (expr); if (_e != hipSuccess) { \
    fprintf(stderr, "HIP error %s at %s:%d\n", hipGetErrorString(_e), __FILE__, __LINE__); \
    return; } } while (0)

#define LAS __attribute__((address_space(3)))
#define GAS __attribute__((address_space(1)))

namespace {

typedef __bf16 bf16x8 __attribute__((ext_vector_type(8)));
typedef __bf16 bf16x2 __attribute__((ext_vector_type(2)));
typedef short s16x4 __attribute__((ext_vector_type(4)));
typedef short s16x8 __attribute__((ext_vector_type(8)));
typedef float f32x4 __attribute__((ext_vector_type(4)));
typedef float f32x2 __attribute__((ext_vector_type(2)));
typedef unsigned u32x4 __attribute__((ext_vector_type(4)));
typedef unsigned u32x2 __attribute__((ext_vector_type(2)));

constexpr int NR = 8;
constexpr unsigned XROW = 1024u;
constexpr unsigned WROW = 32768u;
constexpr unsigned X_BYTES = 4096u * XROW;
constexpr unsigned W_BYTES = 512u * WROW;
constexpr unsigned OUT_BYTES = 512u * WROW;
#define NPUSH 4
constexpr int NPW = NPUSH;
constexpr unsigned NMOVER = NPW * 7u;
constexpr unsigned NBIN = 1408u, NBATCH = NBIN + 512u;
constexpr int NTILE = 26;
constexpr int GRID = 256;

#define LAG 5
#define MV_AUX 0
#define RX_SLEEP 2
#define BK_SLEEP 1
#define REV 1
#define MV_SLEEP 2
#define IN_SLEEP 4
#define NOLOOK 0
#define LOOKAHEAD 1
#define PLAN_AT 15
#define X_NOPIECE 0
#define X_NORD 0
#define X_NOBAR 0
#define PRIO 10
#define JIT 1
#define OUT_AUX 0
#define PUSH_D 3

constexpr unsigned OFF_CTR = 0u;
constexpr unsigned OFF_TMO = 32768u;
constexpr unsigned OFF_CNT = 40960u;
constexpr unsigned OFF_FLG = 65536u;
constexpr unsigned OFF_DBG = 262144u;
constexpr unsigned OFF_XS = 1u << 20;
constexpr unsigned XS_SRC = 4u << 20, XS_PAR = 8u * XS_SRC;
constexpr unsigned OFF_WS = OFF_XS + 2u * XS_PAR;
constexpr unsigned WS_SRC = 2u << 20, WS_PAR = 8u * WS_SRC;
constexpr unsigned Y_UNIT = 104u << 10, Y_PEER = 20u * Y_UNIT;
constexpr unsigned OFF_YP = OFF_WS + 2u * WS_PAR;
constexpr unsigned OFF_YL = OFF_YP + NR * Y_PEER;
constexpr unsigned OFF_SV = OFF_YL + NR * Y_PEER;
constexpr unsigned SV_WAVE = 20480u;
constexpr unsigned WS_NEED = OFF_SV + 7u * NTILE * 8u * SV_WAVE;

constexpr unsigned STAGE = 54272u;
constexpr unsigned LDS_CTL = 3u * STAGE;
constexpr unsigned LDS_BYTES = LDS_CTL + 64u;

constexpr unsigned long long DEADLINE_TICKS = 100000000ull;

struct Params {
    const char* x;
    const char* w;
    char* out;
    char* ws[NR];
    int rank;
    unsigned qmap;
};

__device__ __forceinline__ unsigned rfl(unsigned v) { return (unsigned)__builtin_amdgcn_readfirstlane((int)v); }

__device__ __forceinline__ unsigned fresh_lane() {
    unsigned l;
    asm volatile("v_mbcnt_lo_u32_b32 %0, -1, 0\n\tv_mbcnt_hi_u32_b32 %0, -1, %0" : "=v"(l));
    return l;
}
template <class T> __device__ __forceinline__ T sel3(unsigned k, T a, T b, T c) { return k == 0u ? a : (k == 1u ? b : c); }

__device__ __forceinline__ u32x4 mk_rsrc(const void* p, unsigned bytes) {
    const unsigned long long a = (unsigned long long)p;
    u32x4 r; r.x = (unsigned)a; r.y = (unsigned)(a >> 32) & 0xffffu; r.z = bytes; r.w = 0x00020000u; return r;
}
__device__ __forceinline__ __amdgpu_buffer_rsrc_t mk_brsrc(const void* p, unsigned bytes) {
    return __builtin_amdgcn_make_buffer_rsrc((void*)p, 0, (int)bytes, 0x00020000);
}

template <int OFF, bool SYS> __device__ __forceinline__ void dma16(u32x4 rs, unsigned voff, unsigned soff, unsigned ldsaddr) {
    if constexpr (SYS)
        asm volatile("s_mov_b32 m0, %0\n\ts_nop 0\n\tbuffer_load_dwordx4 %1, %2, %3 offen offset:%4 sc0 sc1 lds"
                     :: "s"(ldsaddr), "v"(voff), "s"(rs), "s"(soff), "i"(OFF) : "memory");
    else
        asm volatile("s_mov_b32 m0, %0\n\ts_nop 0\n\tbuffer_load_dwordx4 %1, %2, %3 offen offset:%4 lds"
                     :: "s"(ldsaddr), "v"(voff), "s"(rs), "s"(soff), "i"(OFF) : "memory");
}
template <int AUX> __device__ __forceinline__ void st128(u32x4 d, __amdgpu_buffer_rsrc_t rs, unsigned voff, unsigned soff) {
    __builtin_amdgcn_raw_buffer_store_b128(d, rs, voff, soff, AUX);
    asm volatile("s_nop 1" :: "v"(d) : "memory");
}
#define WAIT_V(n) asm volatile("s_waitcnt vmcnt(%0)" :: "i"(n) : "memory")
#define BAR() do { asm volatile("" ::: "memory"); __builtin_amdgcn_s_barrier(); asm volatile("" ::: "memory"); } while (0)

__device__ __forceinline__ unsigned ld_sys(const char* p) {
    return __hip_atomic_load((const GAS unsigned*)p, __ATOMIC_RELAXED, __HIP_MEMORY_SCOPE_SYSTEM);
}
__device__ __forceinline__ void st_sys(char* p, unsigned v) {
    __hip_atomic_store((GAS unsigned*)p, v, __ATOMIC_RELAXED, __HIP_MEMORY_SCOPE_SYSTEM);
}
__device__ __forceinline__ unsigned ld_agent(const char* p) {
    return __hip_atomic_load((const GAS unsigned*)p, __ATOMIC_RELAXED, __HIP_MEMORY_SCOPE_AGENT);
}
__device__ __forceinline__ void st_agent(char* p, unsigned v) {
    __hip_atomic_store((GAS unsigned*)p, v, __ATOMIC_RELAXED, __HIP_MEMORY_SCOPE_AGENT);
}

constexpr int PBASE = 3 * (8 - LAG);
__device__ __forceinline__ unsigned slice_of(unsigned round, unsigned h) { return (REV && h != 0u && round >= 8u - LAG) ? 15u - LAG - round : round; }
__device__ __forceinline__ unsigned word_of(unsigned k) {
    unsigned s, type;
    if (k < (unsigned)PBASE) { s = k / 3u; type = k - 3u * s; }
    else if (k < PBASE + 2u * LAG) { s = 8u - LAG + ((k - PBASE) >> 1); type = (k - PBASE) & 1u; }
    else { s = slice_of(min(8u - LAG + (k - PBASE - 2u * LAG), 7u), 1u); type = 2u; }
    return k < 32u ? 4u * s + (type < 2u ? 1u - type : 2u) : k;
}

__device__ __forceinline__ unsigned pk_bf16(float lo, float hi) {
    f32x2 v = {lo, hi};
    bf16x2 b = __builtin_convertvector(v, bf16x2);
    return __builtin_bit_cast(unsigned, b);
}
__device__ __forceinline__ float silu(float y) {
    const float c = __builtin_amdgcn_fmed3f(y, -60.0f, 60.0f);
    const float e = __builtin_amdgcn_exp2f(c * -1.44269504088896341f);
    return y * __builtin_amdgcn_rcpf(1.0f + e);
}

#define STAMP(i) do { } while (0)

struct Batch { unsigned piece, kind, so, dofs; };
__device__ __forceinline__ Batch batch_of(unsigned T, unsigned rank, unsigned q, unsigned par) {
    constexpr unsigned B0 = 176u * (8u - LAG), B1 = B0 + 120u * LAG;
    Batch b;
    if (T < NBIN) {
        unsigned s, r, type;
        if (T < B0)      { s = T / 176u; r = T - 176u * s; type = r < 64u ? 0u : (r < 120u ? 1u : 2u); b.piece = 3u * s + type; }
        else if (T < B1) { const unsigned u = (T - B0) / 120u; r = T - B0 - 120u * u; s = 8u - LAG + u; type = r < 64u ? 0u : 1u; b.piece = PBASE + 2u * u + type; }
        else             { const unsigned u = (T - B1) / 56u; r = T - B1 - 56u * u + 120u; s = slice_of(8u - LAG + u, 1u); type = 2u; b.piece = PBASE + 2u * LAG + u; }
        if (type == 0u) {
            b.kind = 0u;
            b.so = (s * 64u + (r & 1u) * 32u) * WROW + q * 4096u + (r >> 1) * 128u;
            b.dofs = OFF_WS + par * WS_PAR + rank * WS_SRC + s * (256u << 10) + (r >> 1) * 8192u + (r & 1u) * 2048u;
        } else {
            const unsigned e = r - (type == 1u ? 64u : 120u), blk = ((rank + 1u + (e >> 3)) & 7u) * 64u + (type - 1u) * 32u + (e & 7u) * 4u;
            b.kind = 1u;
            b.so = blk * (8u * XROW) + s * 128u;
            b.dofs = OFF_XS + par * XS_PAR + rank * XS_SRC + s * (512u << 10) + blk * 1024u;
        }
    } else {
        const unsigned e = T - NBIN, h = e >> 8, r = e & 255u;
        const unsigned u = r < 156u ? r / 26u : 6u + (r - 156u) / 25u, bi = r < 156u ? r - 26u * u : r - 156u - 25u * (u - 6u);
        const unsigned o = (h * 10u + u) * Y_UNIT + bi * 4096u;
        b.piece = 32u + h * 10u + u; b.kind = 2u + h;
        b.so = OFF_YL + q * Y_PEER + o; b.dofs = OFF_YP + rank * Y_PEER + o;
    }
    b.piece = rfl(b.piece); b.kind = rfl(b.kind); b.so = rfl(b.so); b.dofs = rfl(b.dofs);
    asm("" : "+s"(b.piece), "+s"(b.kind), "+s"(b.so), "+s"(b.dofs));
    return b;
}

__device__ __forceinline__ void pusher(const Params& P, unsigned q, unsigned j, LAS char* lds) {
    const unsigned lane = threadIdx.x & 63u, w = rfl(threadIdx.x >> 6);
    const unsigned rank = (unsigned)P.rank;
    char* const me = P.ws[rank];
    char* const peer = P.ws[q];
    const unsigned callv = ld_agent(me + OFF_CTR + blockIdx.x * 128u);
    volatile LAS unsigned* const lw = (volatile LAS unsigned*)lds;
    if (threadIdx.x < 72u) lw[threadIdx.x] = 0u;

    if (w == 0u) {
        const unsigned call = rfl(callv) + 1u;
        __syncthreads();
        if (lane == 0u) st_agent(me + OFF_CTR + blockIdx.x * 128u, call);
        GAS unsigned* const cnt = (GAS unsigned*)(peer + OFF_FLG + rank * 256u) + word_of(lane);
        unsigned seen = 0u, stops = 0u;
        for (;;) {
            const unsigned gone = rfl(lw[65]), st = rfl(lw[64]), v = lw[lane], d = v - seen;
            const bool later = ((__ballot(v != 0u) >> lane) >> 1) != 0ull;
            if (d != 0u && (later || st != stops)) {
                __hip_atomic_fetch_add(cnt, d, __ATOMIC_RELAXED, __HIP_MEMORY_SCOPE_SYSTEM);
                seen = v;
            }
            stops = st;
            if (gone == 7u) break;
            if (gone == 0u) __builtin_amdgcn_s_sleep(BK_SLEEP); else __builtin_amdgcn_s_sleep(1);
        }
        return;
    }

    const unsigned W = j * 7u + w - 1u;
    const unsigned vx = (lane >> 3) * XROW + (((lane & 7u) ^ (lane >> 4)) * 16u);
    const unsigned vw = (lane >> 2) * WROW + (((lane & 3u) ^ ((lane >> 5) << 1)) * 16u);
    const unsigned vd = lane * 16u;
    const unsigned long long pw = (unsigned long long)P.w, px = (unsigned long long)P.x, pm = (unsigned long long)me;
    auto load4 = [=](const Batch b, bool valid, u32x4 (&r)[4]) {
        const __amdgpu_buffer_rsrc_t rs = mk_brsrc((const void*)sel3(b.kind, pw, px, pm), valid ? sel3(b.kind, W_BYTES, X_BYTES, WS_NEED) : 0u);
        const unsigned v0 = sel3(b.kind, vw, vx, vd), v1 = sel3(b.kind, vw, vx ^ 64u, vd);
        const unsigned o1 = sel3(b.kind, 64u, 8u * XROW, 1024u), o2 = sel3(b.kind, 16u * WROW, 16u * XROW, 2048u);
#pragma unroll
        for (int i = 0; i < 4; ++i) r[i] = __builtin_amdgcn_raw_buffer_load_b128(rs, (i & 1) ? v1 : v0, b.so + (i & 1) * o1 + (i >> 1) * o2, MV_AUX);
    };
    u32x4 cur[4], nxt[4];
    load4(batch_of(W, rank, q, 0u), true, cur);
    const unsigned call = rfl(callv) + 1u, par = call & 1u;
    Batch b0 = batch_of(W, rank, q, par), b1 = batch_of(NMOVER + W, rank, q, par);
    bool v1 = true;
    __syncthreads();
    __builtin_amdgcn_s_waitcnt(0x0F70);
    const __amdgpu_buffer_rsrc_t rd = mk_brsrc(peer, WS_NEED);
    const __amdgpu_buffer_rsrc_t rc = mk_brsrc(me + OFF_CNT + q * 256u, 256u);
    const unsigned vc = lane == 0u ? 0u : 4096u;
    const unsigned tbase = (call - 1u) * NBATCH - 2u * NMOVER;
    constexpr int D = PUSH_D;
    static_assert(D >= 1 && D <= 4, "");
    unsigned pprev = 63u, rdy = 0u;
    bool near = false;
    auto bump = [&](unsigned word) { if (lane == 0u) __hip_atomic_fetch_add((LAS unsigned*)lds + word, 1u, __ATOMIC_RELAXED, __HIP_MEMORY_SCOPE_WORKGROUP); };
    const unsigned vr = lane < (unsigned)NTILE ? 128u + lane * 4u : 4096u;
    auto ready_bits = [&](unsigned r) { const unsigned m = (unsigned)__ballot(r == call);
                                        return ((m & 0x1555555u) == 0x1555555u ? 1u : 0u) | ((m & 0x2aaaaaau) == 0x2aaaaaau ? 2u : 0u); };
    STAMP(0);
    for (;;) {
        const bool ok1 = v1 && (b1.kind < 2u || ((rdy >> (b1.kind - 2u)) & 1u));
        const unsigned tk = (unsigned)__builtin_amdgcn_raw_ptr_buffer_atomic_add_i32(1, rc, vc, 0, 0);
        const bool look = !NOLOOK && near && rdy != 3u;
        const unsigned rv = __builtin_amdgcn_raw_buffer_load_b32(rc, look ? vr : 4096u, 0, 16);
        load4(b1, ok1, nxt);
        asm volatile("" ::: "memory");
        const unsigned o1 = b0.kind == 0u ? 4096u : 1024u, o2 = b0.kind == 0u ? 1024u : 2048u;
#pragma unroll
        for (int i = 0; i < 4; ++i) {
            st128<17>(cur[i], rd, vd, b0.dofs + (i & 1) * o1 + (i >> 1) * o2);
            if (i < D - 1) WAIT_V(D + 6); else WAIT_V(D);
            if (i == D - 1 && pprev != 63u) bump(pprev);
        }
        pprev = b0.piece;
        if (!v1) break;
        if (look) rdy = ready_bits(rv);
        if (!ok1) {
            if (!((rdy >> (b1.kind - 2u)) & 1u)) {
                WAIT_V(0);
                bump(pprev); pprev = 63u;
                bump(64u);
                STAMP(2 + b1.kind);
                const unsigned long long t0 = __builtin_amdgcn_s_memrealtime();
                bool dead = false;
                for (unsigned n = 1u;; ++n) {
                    asm volatile("" ::: "memory");
                    rdy = ready_bits(__builtin_amdgcn_raw_buffer_load_b32(rc, vr, 0, 16));
                    if ((rdy >> (b1.kind - 2u)) & 1u) break;
                    if ((n & 63u) == 0u && (rfl(ld_agent(me + OFF_TMO)) == call || __builtin_amdgcn_s_memrealtime() - t0 > DEADLINE_TICKS)) { dead = true; break; }
                    __builtin_amdgcn_s_sleep(MV_SLEEP);
                }
                if (dead) { if (lane == 0u) st_agent(me + OFF_TMO, call); break; }
            }
            load4(b1, true, nxt);
            __builtin_amdgcn_s_waitcnt(0x0F70);
        }
        const unsigned T = rfl(tk) - tbase;
#pragma unroll
        for (int i = 0; i < 4; ++i) cur[i] = nxt[i];
        b0 = b1; v1 = T < NBATCH; b1 = batch_of(v1 ? T : 0u, rank, q, par);
        near = T + 2u * NMOVER >= NBIN;
    }
    WAIT_V(0);
    if (pprev != 63u) bump(pprev);
    bump(64u);
    bump(65u);
    STAMP(1);
}

__device__ __forceinline__ void gemm_tile(const Params& P, unsigned q, unsigned tile, LAS char* lds) {
    unsigned lane = threadIdx.x & 63u;
    const unsigned w = rfl(threadIdx.x >> 6), wm = w >> 1, wn = w & 1u;
    const unsigned rank = (unsigned)P.rank, g = (q - rank - 1u) & 7u;
    const unsigned h = tile & 1u, c = tile >> 1;
    const bool own = g == 7u;
    char* const me = P.ws[rank];
    const unsigned call = rfl(ld_agent(me + OFF_CTR + blockIdx.x * 128u)) + 1u, par = call & 1u;
    const unsigned lds0 = (unsigned)(uintptr_t)lds;
    LAS unsigned* ctl = (LAS unsigned*)(lds + LDS_CTL);
    STAMP(0);

    const unsigned b0 = w < 4u ? 3u * w : 12u + 2u * (w - 4u);
    const unsigned cbmax = c == 12u ? 3u : 4u;
    const unsigned nraw = own ? 8u : 16u, nsrc = own ? 7u : 6u;
    const unsigned vF = (lane >> 3) * 256u + slice_of(lane & 7u, h) * 16u;
    u32x4 ra, rb, rx;
    unsigned za, zb, zx1, vP[7];
    const unsigned zx0 = w == 7u ? 2048u : 0u, mx = w < 4u ? ~0u : 0u;
    auto staging = [&](bool raw, unsigned lane) {
        ra = raw ? mk_rsrc(P.x, X_BYTES) : mk_rsrc(me + OFF_XS + par * XS_PAR, XS_PAR);
        rb = raw ? mk_rsrc(P.w, W_BYTES) : mk_rsrc(me + OFF_WS + par * WS_PAR, WS_PAR);
        rx = w < 4u ? rb : mk_rsrc(me + OFF_FLG, zx0);
        za = ra.z; zb = rb.z; zx1 = rx.z;
        const unsigned va = (lane >> 3) * XROW + (((lane & 7u) ^ (lane >> 4)) * 16u);
        const unsigned vb = (lane >> 2) * WROW + (((lane & 3u) ^ ((lane >> 5) << 1)) * 16u);
#pragma unroll
        for (int i = 0; i < 4; ++i) vP[i] = (raw ? (va ^ ((i & 1) * 64u)) + (4u * w + i) * (8u * XROW) : lane * 16u + (4u * w + i) * 1024u) - i * 1024u;
#pragma unroll
        for (int i = 0; i < 3; ++i) { const unsigned b = b0 + i, cb = min(b >> 2, cbmax);
            vP[4 + i] = raw ? vb + cb * 64u + (b & 3u) * (16u * WROW) : lane * 16u + cb * 4096u + (b & 3u) * 1024u; }
        if (w >= 4u) vP[6] = (lane >> 3) * 256u + slice_of(lane & 7u, h) * 16u;
    };
    staging(true, lane);
    unsigned tabA, tabB;
    {
        const bool extra = !own && lane < 8u;
        const unsigned s = lane & 7u, e = lane - nraw, r = own ? e / 7u : e / 6u, i = e - r * nsrc;
        const unsigned p = (rank + 1u + i + (i >= g ? 1u : 0u)) & 7u, sl = slice_of(r & 7u, h);
        tabA = lane < nraw ? ((extra ? rank : q) * 512u + h * 256u) * XROW + s * 128u : p * XS_SRC + sl * (512u << 10) + (q * 2u + h) * 32768u;
        tabB = lane < nraw ? s * 64u * WROW + (extra ? q : rank) * 4096u + c * 320u : p * WS_SRC + sl * (256u << 10) + c * 20480u;
    }
    const unsigned la0 = lds0 + w * 4096u, lb0 = lds0 + 32768u + b0 * 1024u;
    const unsigned lx0 = w < 4u ? lb0 + 2048u : (w == 7u ? lds0 + 53248u : lds0 + 0x30000u), kx = (w < 4u || w == 7u) ? ~0u : 0u;
    unsigned oa = 0u, ob = 0u;
    auto piece = [&](int i, unsigned so) {
        if (i == 0) dma16<0, false>(ra, vP[0], oa, la0 + so);
        if (i == 1) dma16<1024, false>(ra, vP[1], oa, la0 + so);
        if (i == 2) dma16<2048, false>(ra, vP[2], oa, la0 + so);
        if (i == 3) dma16<3072, false>(ra, vP[3], oa, la0 + so);
        if (i == 4) dma16<0, false>(rb, vP[4], ob, lb0 + so);
        if (i == 5) dma16<0, false>(rb, vP[5], ob, lb0 + so + 1024u);
        if (i == 6) dma16<0, true>(rx, vP[6], ob & mx, lx0 + (so & kx));
    };
    unsigned it = 0u;
    unsigned gm = 0u;
    auto planA = [&]() { oa = (unsigned)__builtin_amdgcn_readlane((int)tabA, (int)(it & 63u)); ra.z = za & gm; asm volatile("" : "+s"(oa), "+s"(ra)); };
    auto planB = [&]() { ob = (unsigned)__builtin_amdgcn_readlane((int)tabB, (int)(it & 63u)); rb.z = zb & gm; asm volatile("" : "+s"(ob), "+s"(rb)); };
    auto planX = [&]() { rx.z = (zx1 & gm) | zx0; it -= gm; asm volatile("" : "+s"(it), "+s"(rx)); };
    auto plan = [&]() { gm = ~0u; planA(); planB(); planX(); asm volatile("s_nop 4"); };
    const unsigned long long fmask = ~((0xffull << (8u * rank)) | (0xffull << (8u * q)));
    const unsigned need0 = (h ? 64u : 56u) * call, need1 = (h ? 56u : 64u) * call;
    auto rounds_in = [&](unsigned v0, unsigned v1) {
        const unsigned long long miss = __ballot((int)(v0 - need0) < 0 || (int)(v1 - need1) < 0) & fmask;
        unsigned m = (unsigned)miss | (unsigned)(miss >> 32);
        m |= m >> 16; m |= m >> 8;
        return (unsigned)__builtin_ctz(m | 0x100u);
    };
    const unsigned foff = 53248u + h * 4u + lane * 16u;

    unsigned g16 = lane >> 4;
    const unsigned aoff = wm * 8192u + (lane & 15u) * 128u + ((g16 ^ ((lane & 15u) >> 1)) * 16u);
    const unsigned lp = (g16 >> 1) * 1024u + (8u * (g16 & 1u) + ((lane & 15u) >> 2)) * 64u + (lane & 3u) * 8u;
    const unsigned be = 32768u + ((5u * wn) >> 1) * 4096u + lp + (((wn ^ g16) & 1u) * 32u);
    const unsigned bo = 32768u + ((5u * wn + 1u) >> 1) * 4096u + lp + (((wn ^ g16 ^ 1u) & 1u) * 32u);

    f32x4 acc[4][5];
#pragma unroll
    for (int m = 0; m < 4; ++m)
#pragma unroll
        for (int n = 0; n < 5; ++n) acc[m][n] = (f32x4){0.f, 0.f, 0.f, 0.f};
    char* const svp = me + OFF_SV + ((min(g, 6u) * NTILE + tile) * 8u + w) * SV_WAVE;
    s16x8 fa[2][4]; s16x4 fb[2][5][2];
    auto zero1 = [&]() {
#pragma unroll
        for (int m = 0; m < 4; ++m) fa[1][m] = (s16x8){0, 0, 0, 0, 0, 0, 0, 0};
#pragma unroll
        for (int n = 0; n < 5; ++n) { fb[1][n][0] = (s16x4){0, 0, 0, 0}; fb[1][n][1] = (s16x4){0, 0, 0, 0}; }
    };
    auto rd = [&](int kh, int r, LAS char* st) {
        if (r >= 2 && r < 6) fa[kh][r - 2] = *(const LAS s16x8*)(st + (kh ? (aoff ^ 64u) : aoff) + (r - 2) * 2048);
        else { const int n = r < 2 ? 0 : (r - 4) >> 1, hi = r & 1;
               fb[kh][n][hi] = __builtin_amdgcn_ds_read_tr16_b64_v4i16((LAS s16x4*)(st + ((n & 1) ? bo : be) + (n >> 1) * 4096 + kh * 2048 + hi * 256)); }
    };
    auto mm = [&](int kh, int i) {
        const int n = i >> 2, m = i & 3;
        acc[m][n] = __builtin_amdgcn_mfma_f32_16x16x32_bf16(
            __builtin_bit_cast(bf16x8, __builtin_shufflevector(fb[kh][n][0], fb[kh][n][1], 0, 1, 2, 3, 4, 5, 6, 7)),
            __builtin_bit_cast(bf16x8, fa[kh][m]), acc[m][n], 0, 0, 0);
    };
    zero1();

    unsigned t = 0u, ready = nraw, gain = 0u;
    unsigned s0 = 0u, s1 = STAGE, s2 = 2u * STAGE;
    bool bad = false;
    for (;;) {
        if (it == t) {
            plan();
#pragma unroll
            for (int i = 0; i < 7; ++i) piece(i, s0);
            plan();
#pragma unroll
            for (int i = 0; i < 7; ++i) piece(i, s1);
            plan();
        }
        const unsigned park = (!own && t < 8u) ? 8u : 64u;
        do {
            LAS char* st = lds + s0;
            __builtin_amdgcn_s_waitcnt(0xC07F);
            WAIT_V(7);
            if (!X_NOBAR) BAR();
            __builtin_amdgcn_sched_barrier(0);
#pragma unroll
            for (int i = 0; i < 20; ++i) {
                if (PRIO && i == 0) __builtin_amdgcn_s_setprio(3);
                if (PRIO && i == 10) __builtin_amdgcn_s_setprio(2);
                mm(1, i);
                if (i < 14 && (X_NORD == 0 || (X_NORD == 1 && (i < 2 || i >= 6)) || (X_NORD == 2 && i >= 2 && i < 6))) rd(0, i, st);
                if (!X_NOPIECE && i % 5 == 3) piece(i / 5, s2);
                if (i == 15 && __builtin_expect(it == ready && it == t + 3u, 0)) {
                    const LAS unsigned* fr = (const LAS unsigned*)(st + foff);
                    ready = max(ready, nraw + rounds_in(fr[0], fr[1]) * gain);
                }
                if (i == 16) asm volatile("s_cmp_lt_u32 %1, %2\n\ts_cselect_b32 %0, -1, 0" : "=s"(gm) : "s"(it), "s"(ready) : "scc");
                __builtin_amdgcn_sched_barrier(0);
            }
#pragma unroll
            for (int i = 0; i < 20; ++i) {
                if (PRIO && i == 0) __builtin_amdgcn_s_setprio(1);
                if (PRIO && i == PRIO) __builtin_amdgcn_s_setprio(0);
                mm(0, i);
                if (i < 14 && (X_NORD == 0 || (X_NORD == 1 && (i < 2 || i >= 6)) || (X_NORD == 2 && i >= 2 && i < 6))) rd(1, i, st);
                if (!X_NOPIECE && i % 5 == 3 && i < 15) piece(4 + i / 5, s2);
                if (i == 14) planA();
                if (i == 15) planB();
                if (i == 16) planX();
                __builtin_amdgcn_sched_barrier(0);
            }
            { const unsigned x = s0; s0 = s1; s1 = s2; s2 = x; }
            ++t;
        } while (t < min(it, park));
        if (t == 64u) break;
        if (t < it) {
#pragma unroll
            for (int i = 0; i < 20; ++i) mm(1, i);
            const __amdgpu_buffer_rsrc_t rv = mk_brsrc(svp, SV_WAVE);
#pragma unroll
            for (int m = 0; m < 4; ++m)
#pragma unroll
                for (int n = 0; n < 5; ++n) { st128<0>(__builtin_bit_cast(u32x4, acc[m][n]), rv, lane * 16u, (n * 4 + m) * 1024); acc[m][n] = (f32x4){0.f, 0.f, 0.f, 0.f}; }
            zero1();
            continue;
        }
        STAMP(1 + 2 * seg);
        if (t == nraw) {
            if (w == 0u && lane == 0u) st_agent(me + OFF_CTR + blockIdx.x * 128u, call);
            staging(false, lane);
            gain = LOOKAHEAD ? nsrc : 0u;
        }
        if (t == ready) {
            if (w == 0u) {
                const char* fp = me + OFF_FLG + vF + h * 4u;
                const unsigned long long t0 = __builtin_amdgcn_s_memrealtime();
                unsigned res;
                for (unsigned n = 1u;; ++n) {
                    const unsigned v0 = ld_sys(fp), v1 = ld_sys(fp + 4);
                    res = nraw + rounds_in(v0, v1) * nsrc;
                    if (res > t) break;
                    if ((n & 15u) == 0u && (rfl(ld_agent(me + OFF_TMO)) == call || __builtin_amdgcn_s_memrealtime() - t0 > DEADLINE_TICKS)) {
                        st_agent(me + OFF_TMO, call);
                        res = 0x100u + 64u; break;
                    }
                    __builtin_amdgcn_s_sleep(IN_SLEEP);
                }
                ctl[0] = res;
                asm volatile("s_waitcnt vmcnt(0) lgkmcnt(0)" ::: "memory");
            }
            BAR();
            const unsigned res = rfl(*(volatile LAS unsigned*)ctl);
            bad |= res > 0xffu;
            ready = res & 0xffu;
        }
        STAMP(2 + 2 * seg);
    }
#pragma unroll
    for (int i = 0; i < 20; ++i) mm(1, i);
    STAMP(20);

    lane = fresh_lane(); g16 = lane >> 4;
    const unsigned nu = (c == 12u && wn == 1u) ? 6u : 10u;
    const __amdgpu_buffer_rsrc_t ro = mk_brsrc(P.out, OUT_BYTES);
    const unsigned vo = ((g16 & 1u) * 16u + (lane & 15u)) * WROW + (g16 >> 1) * 16u;
    unsigned so0 = (h * 256u + wm * 64u) * WROW + q * 4096u + c * 320u + wn * 160u, so1 = so0 + 32u * WROW;
    asm("" : "+s"(so0), "+s"(so1));
    auto emit = [&](int u, const f32x4& v0, const f32x4& v1) {
        unsigned a0 = pk_bf16(silu(v0[0]), silu(v0[1])), a1 = pk_bf16(silu(v0[2]), silu(v0[3]));
        unsigned c0 = pk_bf16(silu(v1[0]), silu(v1[1])), c1 = pk_bf16(silu(v1[2]), silu(v1[3]));
        const auto s0 = __builtin_amdgcn_permlane16_swap(a0, c0, false, false);
        const auto s1 = __builtin_amdgcn_permlane16_swap(a1, c1, false, false);
        u32x4 d = {s0[0], s1[0], s0[1], s1[1]};
        if (bad) d = (u32x4){0x7fc07fc0u, 0x7fc07fc0u, 0x7fc07fc0u, 0x7fc07fc0u};
        st128<OUT_AUX>(d, ro, vo + (u >> 1) * 32u, (u & 1) ? so1 : so0);
    };
    if (own) {
        bad |= rfl(ld_agent(me + OFF_TMO)) == call;
#pragma unroll
        for (int u = 0; u < 10; ++u) if ((unsigned)u < nu) emit(u, acc[2 * (u & 1)][u >> 1], acc[2 * (u & 1) + 1][u >> 1]);
        STAMP(23);
        return;
    }
    const unsigned yo = h * 10u * Y_UNIT + ((c * 2u + wn) * 4u + wm) * 1024u;
    {
        const __amdgpu_buffer_rsrc_t ry = mk_brsrc(me + OFF_YL + q * Y_PEER, Y_PEER);
#pragma unroll
        for (int u = 0; u < 10; ++u) if ((unsigned)u < nu) {
            const f32x4 v0 = acc[2 * (u & 1)][u >> 1], v1 = acc[2 * (u & 1) + 1][u >> 1];
            u32x4 d; d.x = pk_bf16(v0[0], v0[1]); d.y = pk_bf16(v0[2], v0[3]); d.z = pk_bf16(v1[0], v1[1]); d.w = pk_bf16(v1[2], v1[3]);
            if (bad) d = (u32x4){0x7fc07fc0u, 0x7fc07fc0u, 0x7fc07fc0u, 0x7fc07fc0u};
            st128<16>(d, ry, lane * 16u, yo + u * Y_UNIT);
        }
        WAIT_V(0);
        BAR();
        if (w == 0u && lane == 0u) st_agent(me + OFF_CNT + q * 256u + 128u + tile * 4u, call);
    }
    STAMP(21);
    const __amdgpu_buffer_rsrc_t rv = mk_brsrc(svp, SV_WAVE);
    auto unpark = [&](int u) {
#pragma unroll
        for (int k = 0; k < 2; ++k) acc[2 * (u & 1) + k][u >> 1] = __builtin_bit_cast(f32x4, __builtin_amdgcn_raw_buffer_load_b128(rv, lane * 16u, ((u >> 1) * 4 + 2 * (u & 1) + k) * 1024, 0));
    };
    if (!JIT) {
#pragma unroll
        for (int u = 0; u < 10; ++u) unpark(u);
    }
    const char* const fp = lane < 10u ? me + OFF_FLG + q * 256u + (32u + h * 10u + lane) * 4u : me + OFF_TMO;
    const __amdgpu_buffer_rsrc_t rp = mk_brsrc(me + OFF_YP + q * Y_PEER, Y_PEER);
    const unsigned long long t0 = __builtin_amdgcn_s_memrealtime();
    const unsigned yneed = (lane < 6u ? 26u : 25u) * call;
    unsigned avail = 0u;
    u32x4 pv[10];
    auto finish = [&](int u) {
        const u32x4 d = pv[u];
        f32x4 v0 = acc[2 * (u & 1)][u >> 1], v1 = acc[2 * (u & 1) + 1][u >> 1];
        v0[0] += __builtin_bit_cast(float, d.x << 16); v0[1] += __builtin_bit_cast(float, d.x & 0xffff0000u);
        v0[2] += __builtin_bit_cast(float, d.y << 16); v0[3] += __builtin_bit_cast(float, d.y & 0xffff0000u);
        v1[0] += __builtin_bit_cast(float, d.z << 16); v1[1] += __builtin_bit_cast(float, d.z & 0xffff0000u);
        v1[2] += __builtin_bit_cast(float, d.w << 16); v1[3] += __builtin_bit_cast(float, d.w & 0xffff0000u);
        emit(u, v0, v1);
    };
#pragma unroll
    for (int u = 0; u <= 10; ++u) {
        if (u < 10 && (unsigned)u < nu) {
            while (avail <= (unsigned)u) {
                const unsigned v = ld_sys(fp);
                avail = (unsigned)__builtin_ctzll(__ballot(lane >= 10u || (int)(v - yneed) < 0));
                if (__builtin_amdgcn_readlane((int)v, 10) == (int)call) { bad = true; avail = 10u; }
                if (avail > (unsigned)u) break;
                if (__builtin_amdgcn_s_memrealtime() - t0 > DEADLINE_TICKS) {
                    if (lane == 0u) st_agent(me + OFF_TMO, call);
                    bad = true; avail = 10u;
                }
                __builtin_amdgcn_s_sleep(RX_SLEEP);
            }
            asm volatile("" ::: "memory");
            if (u == 0) STAMP(22);
            if (u == 9) STAMP(24);
            pv[u] = __builtin_amdgcn_raw_buffer_load_b128(rp, bad ? 0xfffffff0u : lane * 16u, yo + u * Y_UNIT, 17);
            if (JIT) { if (u == 0) unpark(0); if (u < 9) unpark(u + 1); }
            asm volatile("" ::: "memory");
        }
        if (u > 0 && (unsigned)(u - 1) < nu) finish(u - 1);
    }
    STAMP(23);
}

__global__ void __launch_bounds__(512, 2) fused_gemm_rs(Params P) {
    __shared__ __attribute__((aligned(16))) char smem[LDS_BYTES];
    LAS char* lds = (LAS char*)smem;
    const unsigned q = (P.qmap >> ((blockIdx.x & 7u) * 4u)) & 7u, slot = blockIdx.x >> 3;
    if (slot < (unsigned)NPW) { if (q != (unsigned)P.rank) pusher(P, q, slot, lds); }
    else if (slot < (unsigned)(NPW + NTILE)) gemm_tile(P, q, slot - NPW, lds);
}

unsigned make_qmap(int rank) {
    static const unsigned char kNear[NR][NR] = {
        {6, 0, 2, 3, 1, 5, 4, 7}, {7, 1, 4, 5, 0, 3, 6, 2}, {3, 2, 0, 6, 4, 5, 7, 1}, {2, 3, 6, 0, 5, 1, 7, 4},
        {5, 4, 1, 7, 2, 0, 6, 3}, {4, 5, 7, 1, 3, 2, 0, 6}, {0, 6, 3, 2, 7, 4, 1, 5}, {1, 7, 5, 4, 6, 3, 2, 0}};
    unsigned m = 0u;
    for (int g = 0; g < NR; ++g) m |= (unsigned)kNear[rank][g] << (4 * g);
    return m;
}

}

extern "C" void kernel_launch(int rank, int nranks, ncclComm_t comm,
                              hipStream_t stream,
                              void* const* d_in, const int* in_sizes, int n_in,
                              void* d_out, int out_size,
                              const int* in_dtypes, int out_dtype,
                              void* const* d_out_all,
                              void* const* d_ws_all, size_t ws_bytes) {
    (void)comm; (void)in_dtypes; (void)out_dtype; (void)d_out_all;
    if (nranks != NR || (unsigned)rank >= (unsigned)NR || n_in != 2 || ws_bytes < (size_t)WS_NEED ||
        in_sizes[0] != 4096 * 512 || in_sizes[1] != 512 * 16384 || out_size != 512 * 16384) {
        fprintf(stderr, "unexpected problem geometry\n");
        return;
    }
    Params P;
    P.x = (const char*)d_in[0];
    P.w = (const char*)d_in[1];
    P.out = (char*)d_out;
    for (int p = 0; p < NR; ++p) P.ws[p] = (char*)d_ws_all[p];
    P.rank = rank;
    P.qmap = make_qmap(rank);
    fused_gemm_rs<<<GRID, 512, 0, stream>>>(P);
    HIP_CHECK(hipGetLastError());
}
```

```cpp
#include <hip/hip_runtime.h>
#include <rccl/rccl.h>
#include <cstdio>
#include <cstdint>

#define HIP_CHECK(expr) do { hipError_t _e = (expr); if (_e != hipSuccess) { \
    fprintf(stderr, "HIP error %s at %s:%d\n", hipGetErrorString(_e), __FILE__, __LINE__); \
    return; } } while (0)

#define LAS __attribute__((address_space(3)))
#define GAS __attribute__((address_space(1)))

namespace {

typedef __bf16 bf16x8 __attribute__((ext_vector_type(8)));
typedef __bf16 bf16x2 __attribute__((ext_vector_type(2)));
typedef short s16x4 __attribute__((ext_vector_type(4)));
typedef short s16x8 __attribute__((ext_vector_type(8)));
typedef float f32x4 __attribute__((ext_vector_type(4)));
typedef float f32x2 __attribute__((ext_vector_type(2)));
typedef unsigned u32x4 __attribute__((ext_vector_type(4)));
typedef unsigned u32x2 __attribute__((ext_vector_type(2)));

constexpr int NR = 8;
constexpr unsigned XROW = 1024u;
constexpr unsigned WROW = 32768u;
constexpr unsigned X_BYTES = 4096u * XROW;
constexpr unsigned W_BYTES = 512u * WROW;
constexpr unsigned OUT_BYTES = 512u * WROW;
#define NPUSH 4
constexpr int NPW = NPUSH;
constexpr unsigned NMOVER = NPW * 7u;
constexpr unsigned NBIN = 1408u, NBATCH = NBIN + 512u;
constexpr int NTILE = 26;
constexpr int GRID = 256;

#define LAG 5
#define MV_AUX 0
#define RX_SLEEP 2
#define BK_SLEEP 1
#define REV 1
#define MV_SLEEP 2
#define IN_SLEEP 4
#define NOLOOK 0
#define LOOKAHEAD 1
#define PLAN_AT 15
#define X_NOPIECE 0
#define X_NORD 0
#define X_NOBAR 0
#define PRIO 10
#define JIT 1
#define GUESS_MASK 7u
#define OUT_AUX 0
#define PUSH_D 3

constexpr unsigned OFF_CTR = 0u;
constexpr unsigned OFF_TMO = 32768u;
constexpr unsigned OFF_CNT = 40960u;
constexpr unsigned OFF_FLG = 65536u;
constexpr unsigned OFF_CLM = 98304u;
constexpr unsigned OFF_DBG = 262144u;
constexpr unsigned OFF_XS = 1u << 20;
constexpr unsigned XS_SRC = 4u << 20, XS_PAR = 8u * XS_SRC;
constexpr unsigned OFF_WS = OFF_XS + 2u * XS_PAR;
constexpr unsigned WS_SRC = 2u << 20, WS_PAR = 8u * WS_SRC;
constexpr unsigned Y_UNIT = 104u << 10, Y_PEER = 20u * Y_UNIT;
constexpr unsigned OFF_YP = OFF_WS + 2u * WS_PAR;
constexpr unsigned OFF_YL = OFF_YP + NR * Y_PEER;
constexpr unsigned OFF_SV = OFF_YL + NR * Y_PEER;
constexpr unsigned SV_WAVE = 20480u;
constexpr unsigned WS_NEED = OFF_SV + 7u * NTILE * 8u * SV_WAVE;

constexpr unsigned STAGE = 54272u;
constexpr unsigned LDS_CTL = 3u * STAGE;
constexpr unsigned LDS_BYTES = LDS_CTL + 64u;

constexpr unsigned long long DEADLINE_TICKS = 100000000ull;

struct Params {
    const char* x;
    const char* w;
    char* out;
    char* me;
    char* ws[NR];
    int rank;
    unsigned qmap;
};

__device__ __forceinline__ unsigned rfl(unsigned v) { return (unsigned)__builtin_amdgcn_readfirstlane((int)v); }

__device__ __forceinline__ unsigned fresh_lane() {
    unsigned l;
    asm volatile("v_mbcnt_lo_u32_b32 %0, -1, 0\n\tv_mbcnt_hi_u32_b32 %0, -1, %0" : "=v"(l));
    return l;
}
template <class T> __device__ __forceinline__ T sel3(unsigned k, T a, T b, T c) { return k == 0u ? a : (k == 1u ? b : c); }

__device__ __forceinline__ u32x4 mk_rsrc(const void* p, unsigned bytes) {
    const unsigned long long a = (unsigned long long)p;
    u32x4 r; r.x = (unsigned)a; r.y = (unsigned)(a >> 32) & 0xffffu; r.z = bytes; r.w = 0x00020000u; return r;
}
__device__ __forceinline__ __amdgpu_buffer_rsrc_t mk_brsrc(const void* p, unsigned bytes) {
    return __builtin_amdgcn_make_buffer_rsrc((void*)p, 0, (int)bytes, 0x00020000);
}

template <int OFF, bool SYS> __device__ __forceinline__ void dma16(u32x4 rs, unsigned voff, unsigned soff, unsigned ldsaddr) {
    if constexpr (SYS)
        asm volatile("s_mov_b32 m0, %0\n\ts_nop 0\n\tbuffer_load_dwordx4 %1, %2, %3 offen offset:%4 sc0 sc1 lds"
                     :: "s"(ldsaddr), "v"(voff), "s"(rs), "s"(soff), "i"(OFF) : "memory");
    else
        asm volatile("s_mov_b32 m0, %0\n\ts_nop 0\n\tbuffer_load_dwordx4 %1, %2, %3 offen offset:%4 lds"
                     :: "s"(ldsaddr), "v"(voff), "s"(rs), "s"(soff), "i"(OFF) : "memory");
}
template <int AUX> __device__ __forceinline__ void st128(u32x4 d, __amdgpu_buffer_rsrc_t rs, unsigned voff, unsigned soff) {
    __builtin_amdgcn_raw_buffer_store_b128(d, rs, voff, soff, AUX);
    asm volatile("s_nop 1" :: "v"(d) : "memory");
}
#define WAIT_V(n) asm volatile("s_waitcnt vmcnt(%0)" :: "i"(n) : "memory")
#define BAR() do { asm volatile("" ::: "memory"); __builtin_amdgcn_s_barrier(); asm volatile("" ::: "memory"); } while (0)

__device__ __forceinline__ unsigned ld_sys(const char* p) {
    return __hip_atomic_load((const GAS unsigned*)p, __ATOMIC_RELAXED, __HIP_MEMORY_SCOPE_SYSTEM);
}
__device__ __forceinline__ void st_sys(char* p, unsigned v) {
    __hip_atomic_store((GAS unsigned*)p, v, __ATOMIC_RELAXED, __HIP_MEMORY_SCOPE_SYSTEM);
}
__device__ __forceinline__ unsigned ld_agent(const char* p) {
    return __hip_atomic_load((const GAS unsigned*)p, __ATOMIC_RELAXED, __HIP_MEMORY_SCOPE_AGENT);
}
__device__ __forceinline__ void st_agent(char* p, unsigned v) {
    __hip_atomic_store((GAS unsigned*)p, v, __ATOMIC_RELAXED, __HIP_MEMORY_SCOPE_AGENT);
}

constexpr int PBASE = 3 * (8 - LAG);
__device__ __forceinline__ unsigned slice_of(unsigned round, unsigned h) { return (REV && h != 0u && round >= 8u - LAG) ? 15u - LAG - round : round; }
__device__ __forceinline__ unsigned word_of(unsigned k) {
    unsigned s, type;
    if (k < (unsigned)PBASE) { s = k / 3u; type = k - 3u * s; }
    else if (k < PBASE + 2u * LAG) { s = 8u - LAG + ((k - PBASE) >> 1); type = (k - PBASE) & 1u; }
    else { s = slice_of(min(8u - LAG + (k - PBASE - 2u * LAG), 7u), 1u); type = 2u; }
    return k < 32u ? 4u * s + (type < 2u ? 1u - type : 2u) : k;
}

__device__ __forceinline__ unsigned pk_bf16(float lo, float hi) {
    f32x2 v = {lo, hi};
    bf16x2 b = __builtin_convertvector(v, bf16x2);
    return __builtin_bit_cast(unsigned, b);
}
__device__ __forceinline__ float silu(float y) {
    const float c = __builtin_amdgcn_fmed3f(y, -60.0f, 60.0f);
    const float e = __builtin_amdgcn_exp2f(c * -1.44269504088896341f);
    return y * __builtin_amdgcn_rcpf(1.0f + e);
}

#define STAMP(i) do { } while (0)

struct Batch { unsigned piece, kind, so, dofs; };
__device__ __forceinline__ Batch batch_of(unsigned T, unsigned rank, unsigned q, unsigned par) {
    constexpr unsigned B0 = 176u * (8u - LAG), B1 = B0 + 120u * LAG;
    Batch b;
    if (T < NBIN) {
        unsigned s, r, type;
        if (T < B0)      { s = T / 176u; r = T - 176u * s; type = r < 64u ? 0u : (r < 120u ? 1u : 2u); b.piece = 3u * s + type; }
        else if (T < B1) { const unsigned u = (T - B0) / 120u; r = T - B0 - 120u * u; s = 8u - LAG + u; type = r < 64u ? 0u : 1u; b.piece = PBASE + 2u * u + type; }
        else             { const unsigned u = (T - B1) / 56u; r = T - B1 - 56u * u + 120u; s = slice_of(8u - LAG + u, 1u); type = 2u; b.piece = PBASE + 2u * LAG + u; }
        if (type == 0u) {
            b.kind = 0u;
            b.so = (s * 64u + (r & 1u) * 32u) * WROW + q * 4096u + (r >> 1) * 128u;
            b.dofs = OFF_WS + par * WS_PAR + rank * WS_SRC + s * (256u << 10) + (r >> 1) * 8192u + (r & 1u) * 2048u;
        } else {
            const unsigned e = r - (type == 1u ? 64u : 120u), blk = ((rank + 1u + (e >> 3)) & 7u) * 64u + (type - 1u) * 32u + (e & 7u) * 4u;
            b.kind = 1u;
            b.so = blk * (8u * XROW) + s * 128u;
            b.dofs = OFF_XS + par * XS_PAR + rank * XS_SRC + s * (512u << 10) + blk * 1024u;
        }
    } else {
        const unsigned e = T - NBIN, h = e >> 8, r = e & 255u;
        const unsigned u = r < 156u ? r / 26u : 6u + (r - 156u) / 25u, bi = r < 156u ? r - 26u * u : r - 156u - 25u * (u - 6u);
        const unsigned o = (h * 10u + u) * Y_UNIT + bi * 4096u;
        b.piece = 32u + h * 10u + u; b.kind = 2u + h;
        b.so = OFF_YL + q * Y_PEER + o; b.dofs = OFF_YP + rank * Y_PEER + o;
    }
    b.piece = rfl(b.piece); b.kind = rfl(b.kind); b.so = rfl(b.so); b.dofs = rfl(b.dofs);
    asm("" : "+s"(b.piece), "+s"(b.kind), "+s"(b.so), "+s"(b.dofs));
    return b;
}

static_assert(NMOVER <= 64u && NPW + NTILE <= 31, "");
__device__ __forceinline__ void first_loads(const Params& P, unsigned q, unsigned W, u32x4 (&r)[4]) {
    const unsigned lane = threadIdx.x & 63u;
    const __amdgpu_buffer_rsrc_t rs = mk_brsrc(P.w, W_BYTES);
    const unsigned vw = (lane >> 2) * WROW + (((lane & 3u) ^ ((lane >> 5) << 1)) * 16u), so = (W & 1u) * 32u * WROW + q * 4096u + (W >> 1) * 128u;
#pragma unroll
    for (int i = 0; i < 4; ++i) r[i] = __builtin_amdgcn_raw_buffer_load_b128(rs, vw, so + (i & 1) * 64u + (i >> 1) * (16u * WROW), MV_AUX);
}

__device__ __forceinline__ void pusher(const Params& P, unsigned q, unsigned j, unsigned call, u32x4 (&cur)[4], LAS char* lds, unsigned dbg_role) {
    const unsigned lane = threadIdx.x & 63u, w = rfl(threadIdx.x >> 6);
    const unsigned rank = (unsigned)P.rank;
    char* const me = P.me;
    char* const peer = P.ws[q];
    volatile LAS unsigned* const lw = (volatile LAS unsigned*)lds;

    if (w == 0u) {
        GAS unsigned* const cnt = (GAS unsigned*)(peer + OFF_FLG + rank * 256u) + word_of(lane);
        unsigned seen = 0u, stops = 0u;
        for (;;) {
            const unsigned gone = rfl(lw[65]), st = rfl(lw[64]), v = lw[lane], d = v - seen;
            const bool later = ((__ballot(v != 0u) >> lane) >> 1) != 0ull;
            if (d != 0u && (later || st != stops)) {
                __hip_atomic_fetch_add(cnt, d, __ATOMIC_RELAXED, __HIP_MEMORY_SCOPE_SYSTEM);
                seen = v;
            }
            stops = st;
            if (gone == 7u) break;
            if (gone == 0u) __builtin_amdgcn_s_sleep(BK_SLEEP); else __builtin_amdgcn_s_sleep(1);
        }
        return;
    }

    const unsigned W = j * 7u + w - 1u;
    const unsigned vx = (lane >> 3) * XROW + (((lane & 7u) ^ (lane >> 4)) * 16u);
    const unsigned vw = (lane >> 2) * WROW + (((lane & 3u) ^ ((lane >> 5) << 1)) * 16u);
    const unsigned vd = lane * 16u;
    const unsigned long long pw = (unsigned long long)P.w, px = (unsigned long long)P.x, pm = (unsigned long long)me;
    auto load4 = [=](const Batch b, bool valid, u32x4 (&r)[4]) {
        const __amdgpu_buffer_rsrc_t rs = mk_brsrc((const void*)sel3(b.kind, pw, px, pm), valid ? sel3(b.kind, W_BYTES, X_BYTES, WS_NEED) : 0u);
        const unsigned v0 = sel3(b.kind, vw, vx, vd), v1 = sel3(b.kind, vw, vx ^ 64u, vd);
        const unsigned o1 = sel3(b.kind, 64u, 8u * XROW, 1024u), o2 = sel3(b.kind, 16u * WROW, 16u * XROW, 2048u);
#pragma unroll
        for (int i = 0; i < 4; ++i) r[i] = __builtin_amdgcn_raw_buffer_load_b128(rs, (i & 1) ? v1 : v0, b.so + (i & 1) * o1 + (i >> 1) * o2, MV_AUX);
    };
    u32x4 nxt[4];
    const unsigned par = call & 1u;
    Batch b0 = {0u, 0u, 0u, OFF_WS + par * WS_PAR + rank * WS_SRC + (W >> 1) * 8192u + (W & 1u) * 2048u}, b1 = batch_of(NMOVER + W, rank, q, par);
    bool v1 = true;
    __builtin_amdgcn_s_waitcnt(0x0F70);
    const __amdgpu_buffer_rsrc_t rd = mk_brsrc(peer, WS_NEED);
    const __amdgpu_buffer_rsrc_t rc = mk_brsrc(me + OFF_CNT + q * 256u, 256u);
    const unsigned vc = lane == 0u ? 0u : 4096u;
    const unsigned tbase = (call - 1u) * NBATCH - 2u * NMOVER;
    constexpr int D = PUSH_D;
    static_assert(D >= 1 && D <= 4, "");
    unsigned pprev = 63u, rdy = 0u;
    bool near = false;
    auto bump = [&](unsigned word) { if (lane == 0u) __hip_atomic_fetch_add((LAS unsigned*)lds + word, 1u, __ATOMIC_RELAXED, __HIP_MEMORY_SCOPE_WORKGROUP); };
    const unsigned vr = lane < (unsigned)NTILE ? 128u + lane * 4u : 4096u;
    auto ready_bits = [&](unsigned r) { const unsigned m = (unsigned)__ballot(r == call);
                                        return ((m & 0x1555555u) == 0x1555555u ? 1u : 0u) | ((m & 0x2aaaaaau) == 0x2aaaaaau ? 2u : 0u); };
    STAMP(0);
    for (;;) {
        const bool ok1 = v1 && (b1.kind < 2u || ((rdy >> (b1.kind - 2u)) & 1u));
        const unsigned tk = (unsigned)__builtin_amdgcn_raw_ptr_buffer_atomic_add_i32(1, rc, vc, 0, 0);
        const bool look = !NOLOOK && near && rdy != 3u;
        const unsigned rv = __builtin_amdgcn_raw_buffer_load_b32(rc, look ? vr : 4096u, 0, 16);
        load4(b1, ok1, nxt);
        asm volatile("" ::: "memory");
        const unsigned o1 = b0.kind == 0u ? 4096u : 1024u, o2 = b0.kind == 0u ? 1024u : 2048u;
#pragma unroll
        for (int i = 0; i < 4; ++i) {
            st128<17>(cur[i], rd, vd, b0.dofs + (i & 1) * o1 + (i >> 1) * o2);
            if (i < D - 1) WAIT_V(D + 6); else WAIT_V(D);
            if (i == D - 1 && pprev != 63u) bump(pprev);
        }
        pprev = b0.piece;
        if (!v1) break;
        if (look) rdy = ready_bits(rv);
        if (!ok1) {
            if (!((rdy >> (b1.kind - 2u)) & 1u)) {
                WAIT_V(0);
                bump(pprev); pprev = 63u;
                bump(64u);
                STAMP(2 + b1.kind);
                const unsigned long long t0 = __builtin_amdgcn_s_memrealtime();
                bool dead = false;
                for (unsigned n = 1u;; ++n) {
                    asm volatile("" ::: "memory");
                    rdy = ready_bits(__builtin_amdgcn_raw_buffer_load_b32(rc, vr, 0, 16));
                    if ((rdy >> (b1.kind - 2u)) & 1u) break;
                    if ((n & 63u) == 0u && (rfl(ld_agent(me + OFF_TMO)) == call || __builtin_amdgcn_s_memrealtime() - t0 > DEADLINE_TICKS)) { dead = true; break; }
                    __builtin_amdgcn_s_sleep(MV_SLEEP);
                }
                if (dead) { if (lane == 0u) st_agent(me + OFF_TMO, call); break; }
            }
            load4(b1, true, nxt);
            __builtin_amdgcn_s_waitcnt(0x0F70);
        }
        const unsigned T = rfl(tk) - tbase;
#pragma unroll
        for (int i = 0; i < 4; ++i) cur[i] = nxt[i];
        b0 = b1; v1 = T < NBATCH; b1 = batch_of(v1 ? T : 0u, rank, q, par);
        near = T + 2u * NMOVER >= NBIN;
    }
    WAIT_V(0);
    if (pprev != 63u) bump(pprev);
    bump(64u);
    bump(65u);
    STAMP(1);
}

__device__ __forceinline__ void gemm_tile(const Params& P, unsigned q, unsigned tile, unsigned call, LAS char* lds, unsigned dbg_role) {
    unsigned lane = threadIdx.x & 63u;
    const unsigned w = rfl(threadIdx.x >> 6), wm = w >> 1, wn = w & 1u;
    const unsigned rank = (unsigned)P.rank, g = (q - rank - 1u) & 7u;
    const unsigned h = tile & 1u, c = tile >> 1;
    const bool own = g == 7u;
    char* const me = P.me;
    const unsigned par = call & 1u;
    const unsigned lds0 = (unsigned)(uintptr_t)lds;
    LAS unsigned* ctl = (LAS unsigned*)(lds + LDS_CTL);
    STAMP(0);

    const unsigned b0 = w < 4u ? 3u * w : 12u + 2u * (w - 4u);
    const unsigned cbmax = c == 12u ? 3u : 4u;
    const unsigned nraw = own ? 8u : 16u, nsrc = own ? 7u : 6u;
    const unsigned vF = (lane >> 3) * 256u + slice_of(lane & 7u, h) * 16u;
    u32x4 ra, rb, rx;
    unsigned za, zb, zx1, vP[7];
    const unsigned zx0 = w == 7u ? 2048u : 0u, mx = w < 4u ? ~0u : 0u;
    auto staging = [&](bool raw, unsigned lane) {
        ra = raw ? mk_rsrc(P.x, X_BYTES) : mk_rsrc(me + OFF_XS + par * XS_PAR, XS_PAR);
        rb = raw ? mk_rsrc(P.w, W_BYTES) : mk_rsrc(me + OFF_WS + par * WS_PAR, WS_PAR);
        rx = w < 4u ? rb : mk_rsrc(me + OFF_FLG, zx0);
        za = ra.z; zb = rb.z; zx1 = rx.z;
        const unsigned va = (lane >> 3) * XROW + (((lane & 7u) ^ (lane >> 4)) * 16u);
        const unsigned vb = (lane >> 2) * WROW + (((lane & 3u) ^ ((lane >> 5) << 1)) * 16u);
#pragma unroll
        for (int i = 0; i < 4; ++i) vP[i] = (raw ? (va ^ ((i & 1) * 64u)) + (4u * w + i) * (8u * XROW) : lane * 16u + (4u * w + i) * 1024u) - i * 1024u;
#pragma unroll
        for (int i = 0; i < 3; ++i) { const unsigned b = b0 + i, cb = min(b >> 2, cbmax);
            vP[4 + i] = raw ? vb + cb * 64u + (b & 3u) * (16u * WROW) : lane * 16u + cb * 4096u + (b & 3u) * 1024u; }
        if (w >= 4u) vP[6] = (lane >> 3) * 256u + slice_of(lane & 7u, h) * 16u;
    };
    staging(true, lane);
    unsigned tabA, tabB;
    {
        const bool extra = !own && lane < 8u;
        const unsigned s = lane & 7u, e = lane - nraw, r = own ? e / 7u : e / 6u, i = e - r * nsrc;
        const unsigned p = (rank + 1u + i + (i >= g ? 1u : 0u)) & 7u, sl = slice_of(r & 7u, h);
        tabA = lane < nraw ? ((extra ? rank : q) * 512u + h * 256u) * XROW + s * 128u : p * XS_SRC + sl * (512u << 10) + (q * 2u + h) * 32768u;
        tabB = lane < nraw ? s * 64u * WROW + (extra ? q : rank) * 4096u + c * 320u : p * WS_SRC + sl * (256u << 10) + c * 20480u;
    }
    const unsigned la0 = lds0 + w * 4096u, lb0 = lds0 + 32768u + b0 * 1024u;
    const unsigned lx0 = w < 4u ? lb0 + 2048u : (w == 7u ? lds0 + 53248u : lds0 + 0x30000u), kx = (w < 4u || w == 7u) ? ~0u : 0u;
    unsigned oa = 0u, ob = 0u;
    auto piece = [&](int i, unsigned so) {
        if (i == 0) dma16<0, false>(ra, vP[0], oa, la0 + so);
        if (i == 1) dma16<1024, false>(ra, vP[1], oa, la0 + so);
        if (i == 2) dma16<2048, false>(ra, vP[2], oa, la0 + so);
        if (i == 3) dma16<3072, false>(ra, vP[3], oa, la0 + so);
        if (i == 4) dma16<0, false>(rb, vP[4], ob, lb0 + so);
        if (i == 5) dma16<0, false>(rb, vP[5], ob, lb0 + so + 1024u);
        if (i == 6) dma16<0, true>(rx, vP[6], ob & mx, lx0 + (so & kx));
    };
    unsigned it = 0u;
    unsigned gm = 0u;
    auto planA = [&]() { oa = (unsigned)__builtin_amdgcn_readlane((int)tabA, (int)(it & 63u)); ra.z = za & gm; asm volatile("" : "+s"(oa), "+s"(ra)); };
    auto planB = [&]() { ob = (unsigned)__builtin_amdgcn_readlane((int)tabB, (int)(it & 63u)); rb.z = zb & gm; asm volatile("" : "+s"(ob), "+s"(rb)); };
    auto planX = [&]() { rx.z = (zx1 & gm) | zx0; it -= gm; asm volatile("" : "+s"(it), "+s"(rx)); };
    auto plan = [&]() { gm = ~0u; planA(); planB(); planX(); asm volatile("s_nop 4"); };
    const unsigned long long fmask = ~((0xffull << (8u * rank)) | (0xffull << (8u * q)));
    const unsigned need0 = (h ? 64u : 56u) * call, need1 = (h ? 56u : 64u) * call;
    auto rounds_in = [&](unsigned v0, unsigned v1) {
        const unsigned long long miss = __ballot((int)(v0 - need0) < 0 || (int)(v1 - need1) < 0) & fmask;
        unsigned m = (unsigned)miss | (unsigned)(miss >> 32);
        m |= m >> 16; m |= m >> 8;
        return (unsigned)__builtin_ctz(m | 0x100u);
    };
    const unsigned foff = 53248u + h * 4u + lane * 16u;

    unsigned g16 = lane >> 4;
    const unsigned aoff = wm * 8192u + (lane & 15u) * 128u + ((g16 ^ ((lane & 15u) >> 1)) * 16u);
    const unsigned lp = (g16 >> 1) * 1024u + (8u * (g16 & 1u) + ((lane & 15u) >> 2)) * 64u + (lane & 3u) * 8u;
    const unsigned be = 32768u + ((5u * wn) >> 1) * 4096u + lp + (((wn ^ g16) & 1u) * 32u);
    const unsigned bo = 32768u + ((5u * wn + 1u) >> 1) * 4096u + lp + (((wn ^ g16 ^ 1u) & 1u) * 32u);

    f32x4 acc[4][5];
#pragma unroll
    for (int m = 0; m < 4; ++m)
#pragma unroll
        for (int n = 0; n < 5; ++n) acc[m][n] = (f32x4){0.f, 0.f, 0.f, 0.f};
    char* const svp = me + OFF_SV + ((min(g, 6u) * NTILE + tile) * 8u + w) * SV_WAVE;
    s16x8 fa[2][4]; s16x4 fb[2][5][2];
    auto zero1 = [&]() {
#pragma unroll
        for (int m = 0; m < 4; ++m) fa[1][m] = (s16x8){0, 0, 0, 0, 0, 0, 0, 0};
#pragma unroll
        for (int n = 0; n < 5; ++n) { fb[1][n][0] = (s16x4){0, 0, 0, 0}; fb[1][n][1] = (s16x4){0, 0, 0, 0}; }
    };
    auto rd = [&](int kh, int r, LAS char* st) {
        if (r >= 2 && r < 6) fa[kh][r - 2] = *(const LAS s16x8*)(st + (kh ? (aoff ^ 64u) : aoff) + (r - 2) * 2048);
        else { const int n = r < 2 ? 0 : (r - 4) >> 1, hi = r & 1;
               fb[kh][n][hi] = __builtin_amdgcn_ds_read_tr16_b64_v4i16((LAS s16x4*)(st + ((n & 1) ? bo : be) + (n >> 1) * 4096 + kh * 2048 + hi * 256)); }
    };
    auto mm = [&](int kh, int i) {
        const int n = i >> 2, m = i & 3;
        acc[m][n] = __builtin_amdgcn_mfma_f32_16x16x32_bf16(
            __builtin_bit_cast(bf16x8, __builtin_shufflevector(fb[kh][n][0], fb[kh][n][1], 0, 1, 2, 3, 4, 5, 6, 7)),
            __builtin_bit_cast(bf16x8, fa[kh][m]), acc[m][n], 0, 0, 0);
    };
    zero1();

    unsigned t = 0u, ready = nraw, gain = 0u;
    unsigned s0 = 0u, s1 = STAGE, s2 = 2u * STAGE;
    bool bad = false;
    for (;;) {
        if (it == t) {
            plan();
#pragma unroll
            for (int i = 0; i < 7; ++i) piece(i, s0);
            plan();
#pragma unroll
            for (int i = 0; i < 7; ++i) piece(i, s1);
            plan();
        }
        const unsigned park = (!own && t < 8u) ? 8u : 64u;
        do {
            LAS char* st = lds + s0;
            __builtin_amdgcn_s_waitcnt(0xC07F);
            WAIT_V(7);
            if (!X_NOBAR) BAR();
            __builtin_amdgcn_sched_barrier(0);
#pragma unroll
            for (int i = 0; i < 20; ++i) {
                if (PRIO && i == 0) __builtin_amdgcn_s_setprio(3);
                if (PRIO && i == 10) __builtin_amdgcn_s_setprio(2);
                mm(1, i);
                if (i < 14 && (X_NORD == 0 || (X_NORD == 1 && (i < 2 || i >= 6)) || (X_NORD == 2 && i >= 2 && i < 6))) rd(0, i, st);
                if (!X_NOPIECE && i % 5 == 3) piece(i / 5, s2);
                if (i == 15 && __builtin_expect(it == ready && it == t + 3u, 0)) {
                    const LAS unsigned* fr = (const LAS unsigned*)(st + foff);
                    ready = max(ready, nraw + rounds_in(fr[0], fr[1]) * gain);
                }
                if (i == 16) asm volatile("s_cmp_lt_u32 %1, %2\n\ts_cselect_b32 %0, -1, 0" : "=s"(gm) : "s"(it), "s"(ready) : "scc");
                __builtin_amdgcn_sched_barrier(0);
            }
#pragma unroll
            for (int i = 0; i < 20; ++i) {
                if (PRIO && i == 0) __builtin_amdgcn_s_setprio(1);
                if (PRIO && i == PRIO) __builtin_amdgcn_s_setprio(0);
                mm(0, i);
                if (i < 14 && (X_NORD == 0 || (X_NORD == 1 && (i < 2 || i >= 6)) || (X_NORD == 2 && i >= 2 && i < 6))) rd(1, i, st);
                if (!X_NOPIECE && i % 5 == 3 && i < 15) piece(4 + i / 5, s2);
                if (i == 14) planA();
                if (i == 15) planB();
                if (i == 16) planX();
                __builtin_amdgcn_sched_barrier(0);
            }
            { const unsigned x = s0; s0 = s1; s1 = s2; s2 = x; }
            ++t;
        } while (t < min(it, park));
        if (t == 64u) break;
        if (t < it) {
#pragma unroll
            for (int i = 0; i < 20; ++i) mm(1, i);
            const __amdgpu_buffer_rsrc_t rv = mk_brsrc(svp, SV_WAVE);
#pragma unroll
            for (int m = 0; m < 4; ++m)
#pragma unroll
                for (int n = 0; n < 5; ++n) { st128<0>(__builtin_bit_cast(u32x4, acc[m][n]), rv, lane * 16u, (n * 4 + m) * 1024); acc[m][n] = (f32x4){0.f, 0.f, 0.f, 0.f}; }
            zero1();
            continue;
        }
        STAMP(1 + 2 * seg);
        if (t == nraw) {
            staging(false, lane);
            gain = LOOKAHEAD ? nsrc : 0u;
        }
        if (t == ready) {
            if (w == 0u) {
                const char* fp = me + OFF_FLG + vF + h * 4u;
                const unsigned long long t0 = __builtin_amdgcn_s_memrealtime();
                unsigned res;
                for (unsigned n = 1u;; ++n) {
                    const unsigned v0 = ld_sys(fp), v1 = ld_sys(fp + 4);
                    res = nraw + rounds_in(v0, v1) * nsrc;
                    if (res > t) break;
                    if ((n & 15u) == 0u && (rfl(ld_agent(me + OFF_TMO)) == call || __builtin_amdgcn_s_memrealtime() - t0 > DEADLINE_TICKS)) {
                        st_agent(me + OFF_TMO, call);
                        res = 0x100u + 64u; break;
                    }
                    __builtin_amdgcn_s_sleep(IN_SLEEP);
                }
                ctl[0] = res;
                asm volatile("s_waitcnt vmcnt(0) lgkmcnt(0)" ::: "memory");
            }
            BAR();
            const unsigned res = rfl(*(volatile LAS unsigned*)ctl);
            bad |= res > 0xffu;
            ready = res & 0xffu;
        }
        STAMP(2 + 2 * seg);
    }
#pragma unroll
    for (int i = 0; i < 20; ++i) mm(1, i);
    STAMP(20);

    lane = fresh_lane(); g16 = lane >> 4;
    const unsigned nu = (c == 12u && wn == 1u) ? 6u : 10u;
    const __amdgpu_buffer_rsrc_t ro = mk_brsrc(P.out, OUT_BYTES);
    const unsigned vo = ((g16 & 1u) * 16u + (lane & 15u)) * WROW + (g16 >> 1) * 16u;
    unsigned so0 = (h * 256u + wm * 64u) * WROW + q * 4096u + c * 320u + wn * 160u, so1 = so0 + 32u * WROW;
    asm("" : "+s"(so0), "+s"(so1));
    auto emit = [&](int u, const f32x4& v0, const f32x4& v1) {
        unsigned a0 = pk_bf16(silu(v0[0]), silu(v0[1])), a1 = pk_bf16(silu(v0[2]), silu(v0[3]));
        unsigned c0 = pk_bf16(silu(v1[0]), silu(v1[1])), c1 = pk_bf16(silu(v1[2]), silu(v1[3]));
        const auto s0 = __builtin_amdgcn_permlane16_swap(a0, c0, false, false);
        const auto s1 = __builtin_amdgcn_permlane16_swap(a1, c1, false, false);
        u32x4 d = {s0[0], s1[0], s0[1], s1[1]};
        if (bad) d = (u32x4){0x7fc07fc0u, 0x7fc07fc0u, 0x7fc07fc0u, 0x7fc07fc0u};
        st128<OUT_AUX>(d, ro, vo + (u >> 1) * 32u, (u & 1) ? so1 : so0);
    };
    if (own) {
        bad |= rfl(ld_agent(me + OFF_TMO)) == call;
#pragma unroll
        for (int u = 0; u < 10; ++u) if ((unsigned)u < nu) emit(u, acc[2 * (u & 1)][u >> 1], acc[2 * (u & 1) + 1][u >> 1]);
        STAMP(23);
        return;
    }
    const unsigned yo = h * 10u * Y_UNIT + ((c * 2u + wn) * 4u + wm) * 1024u;
    {
        const __amdgpu_buffer_rsrc_t ry = mk_brsrc(me + OFF_YL + q * Y_PEER, Y_PEER);
#pragma unroll
        for (int u = 0; u < 10; ++u) if ((unsigned)u < nu) {
            const f32x4 v0 = acc[2 * (u & 1)][u >> 1], v1 = acc[2 * (u & 1) + 1][u >> 1];
            u32x4 d; d.x = pk_bf16(v0[0], v0[1]); d.y = pk_bf16(v0[2], v0[3]); d.z = pk_bf16(v1[0], v1[1]); d.w = pk_bf16(v1[2], v1[3]);
            if (bad) d = (u32x4){0x7fc07fc0u, 0x7fc07fc0u, 0x7fc07fc0u, 0x7fc07fc0u};
            st128<16>(d, ry, lane * 16u, yo + u * Y_UNIT);
        }
        WAIT_V(0);
        BAR();
        if (w == 0u && lane == 0u) st_agent(me + OFF_CNT + q * 256u + 128u + tile * 4u, call);
    }
    STAMP(21);
    const __amdgpu_buffer_rsrc_t rv = mk_brsrc(svp, SV_WAVE);
    auto unpark = [&](int u) {
#pragma unroll
        for (int k = 0; k < 2; ++k) acc[2 * (u & 1) + k][u >> 1] = __builtin_bit_cast(f32x4, __builtin_amdgcn_raw_buffer_load_b128(rv, lane * 16u, ((u >> 1) * 4 + 2 * (u & 1) + k) * 1024, 0));
    };
    if (!JIT) {
#pragma unroll
        for (int u = 0; u < 10; ++u) unpark(u);
    }
    const char* const fp = lane < 10u ? me + OFF_FLG + q * 256u + (32u + h * 10u + lane) * 4u : me + OFF_TMO;
    const __amdgpu_buffer_rsrc_t rp = mk_brsrc(me + OFF_YP + q * Y_PEER, Y_PEER);
    const unsigned long long t0 = __builtin_amdgcn_s_memrealtime();
    const unsigned yneed = (lane < 6u ? 26u : 25u) * call;
    unsigned avail = 0u;
    u32x4 pv[10];
    auto finish = [&](int u) {
        const u32x4 d = pv[u];
        f32x4 v0 = acc[2 * (u & 1)][u >> 1], v1 = acc[2 * (u & 1) + 1][u >> 1];
        v0[0] += __builtin_bit_cast(float, d.x << 16); v0[1] += __builtin_bit_cast(float, d.x & 0xffff0000u);
        v0[2] += __builtin_bit_cast(float, d.y << 16); v0[3] += __builtin_bit_cast(float, d.y & 0xffff0000u);
        v1[0] += __builtin_bit_cast(float, d.z << 16); v1[1] += __builtin_bit_cast(float, d.z & 0xffff0000u);
        v1[2] += __builtin_bit_cast(float, d.w << 16); v1[3] += __builtin_bit_cast(float, d.w & 0xffff0000u);
        emit(u, v0, v1);
    };
#pragma unroll
    for (int u = 0; u <= 10; ++u) {
        if (u < 10 && (unsigned)u < nu) {
            while (avail <= (unsigned)u) {
                const unsigned v = ld_sys(fp);
                avail = (unsigned)__builtin_ctzll(__ballot(lane >= 10u || (int)(v - yneed) < 0));
                if (__builtin_amdgcn_readlane((int)v, 10) == (int)call) { bad = true; avail = 10u; }
                if (avail > (unsigned)u) break;
                if (__builtin_amdgcn_s_memrealtime() - t0 > DEADLINE_TICKS) {
                    if (lane == 0u) st_agent(me + OFF_TMO, call);
                    bad = true; avail = 10u;
                }
                __builtin_amdgcn_s_sleep(RX_SLEEP);
            }
            asm volatile("" ::: "memory");
            if (u == 0) STAMP(22);
            if (u == 9) STAMP(24);
            pv[u] = __builtin_amdgcn_raw_buffer_load_b128(rp, bad ? 0xfffffff0u : lane * 16u, yo + u * Y_UNIT, 17);
            if (JIT) { if (u == 0) unpark(0); if (u < 9) unpark(u + 1); }
            asm volatile("" ::: "memory");
        }
        if (u > 0 && (unsigned)(u - 1) < nu) finish(u - 1);
    }
    STAMP(23);
}

__global__ void __launch_bounds__(512, 2) fused_gemm_rs(Params P) {
    __shared__ __attribute__((aligned(16))) char smem[LDS_BYTES];
    LAS char* lds = (LAS char*)smem;
    volatile LAS unsigned* const lw = (volatile LAS unsigned*)lds;
    const unsigned w = rfl(threadIdx.x >> 6), rank = (unsigned)P.rank;
    const unsigned guess = (blockIdx.x & ~7u) | ((unsigned)__builtin_amdgcn_s_getreg((3 << 11) | 20) & GUESS_MASK);
    u32x4 cur[4];
    auto prefetch = [&](unsigned role) {
        const unsigned q = (P.qmap >> ((role & 7u) * 4u)) & 7u, slot = role >> 3;
        if (slot < (unsigned)NPW && w != 0u) first_loads(P, q, slot * 7u + w - 1u, cur);
    };
    if (w == 0u) {
        char* const me = P.me;
        const __amdgpu_buffer_rsrc_t rc = mk_brsrc(me + OFF_CLM, 32768u);
        const unsigned v0 = threadIdx.x == 0u ? 0u : 65536u;
        unsigned role = guess, old = (unsigned)__builtin_amdgcn_raw_ptr_buffer_atomic_add_i32(1, rc, v0, role * 128u, 0);
        const unsigned done = rfl(ld_agent(me + OFF_CTR + blockIdx.x * 128u));
        lw[threadIdx.x] = 0u;
        while (rfl(old) != done) {
            __builtin_amdgcn_raw_ptr_buffer_atomic_add_i32(-1, rc, v0, role * 128u, 0);
            role = (role + 1u) & 255u;
            if (role == guess) { role = 255u; st_agent(me + OFF_TMO, done + 1u); break; }
            old = (unsigned)__builtin_amdgcn_raw_ptr_buffer_atomic_add_i32(1, rc, v0, role * 128u, 0);
        }
        if (threadIdx.x < 2u) lw[64u + threadIdx.x] = 0u;
        if (threadIdx.x == 0u) { lw[LDS_CTL / 4u + 1u] = role; lw[LDS_CTL / 4u + 2u] = done + 1u; st_agent(me + OFF_CTR + blockIdx.x * 128u, done + 1u); }
        asm volatile("s_waitcnt lgkmcnt(0)" ::: "memory");
    } else prefetch(guess);
    BAR();
    const unsigned role = rfl(lw[LDS_CTL / 4u + 1u]), call = rfl(lw[LDS_CTL / 4u + 2u]);
    if (role != guess) prefetch(role);
    const unsigned q = (P.qmap >> ((role & 7u) * 4u)) & 7u, slot = role >> 3;
    if (slot < (unsigned)NPW) { if (q != rank) pusher(P, q, slot, call, cur, lds, role); }
    else if (slot < (unsigned)(NPW + NTILE)) gemm_tile(P, q, slot - NPW, call, lds, role);
}

unsigned make_qmap(int rank) {
    static const unsigned char kNear[NR][NR] = {
        {6, 0, 2, 3, 1, 5, 4, 7}, {7, 1, 4, 5, 0, 3, 6, 2}, {3, 2, 0, 6, 4, 5, 7, 1}, {2, 3, 6, 0, 5, 1, 7, 4},
        {5, 4, 1, 7, 2, 0, 6, 3}, {4, 5, 7, 1, 3, 2, 0, 6}, {0, 6, 3, 2, 7, 4, 1, 5}, {1, 7, 5, 4, 6, 3, 2, 0}};
    unsigned m = 0u;
    for (int g = 0; g < NR; ++g) m |= (unsigned)kNear[rank][g] << (4 * g);
    return m;
}

}

extern "C" void kernel_launch(int rank, int nranks, ncclComm_t comm,
                              hipStream_t stream,
                              void* const* d_in, const int* in_sizes, int n_in,
                              void* d_out, int out_size,
                              const int* in_dtypes, int out_dtype,
                              void* const* d_out_all,
                              void* const* d_ws_all, size_t ws_bytes) {
    (void)comm; (void)in_dtypes; (void)out_dtype; (void)d_out_all;
    if (nranks != NR || (unsigned)rank >= (unsigned)NR || n_in != 2 || ws_bytes < (size_t)WS_NEED ||
        in_sizes[0] != 4096 * 512 || in_sizes[1] != 512 * 16384 || out_size != 512 * 16384) {
        fprintf(stderr, "unexpected problem geometry\n");
        return;
    }
    Params P;
    P.x = (const char*)d_in[0];
    P.w = (const char*)d_in[1];
    P.out = (char*)d_out;
    for (int p = 0; p < NR; ++p) P.ws[p] = (char*)d_ws_all[p];
    P.me = P.ws[rank];
    P.rank = rank;
    P.qmap = make_qmap(rank);
    fused_gemm_rs<<<GRID, 512, 0, stream>>>(P);
    HIP_CHECK(hipGetLastError());
}
```

```cpp
#include <hip/hip_runtime.h>
#include <rccl/rccl.h>
#include <cstdio>
#include <cstdint>

#define HIP_CHECK(expr) do { hipError_t _e = (expr); if (_e != hipSuccess) { \
    fprintf(stderr, "HIP error %s at %s:%d\n", hipGetErrorString(_e), __FILE__, __LINE__); \
    return; } } while (0)

#define LAS __attribute__((address_space(3)))
#define GAS __attribute__((address_space(1)))

namespace {

typedef __bf16 bf16x8 __attribute__((ext_vector_type(8)));
typedef __bf16 bf16x2 __attribute__((ext_vector_type(2)));
typedef short s16x4 __attribute__((ext_vector_type(4)));
typedef short s16x8 __attribute__((ext_vector_type(8)));
typedef float f32x4 __attribute__((ext_vector_type(4)));
typedef float f32x2 __attribute__((ext_vector_type(2)));
typedef unsigned u32x4 __attribute__((ext_vector_type(4)));
typedef unsigned u32x2 __attribute__((ext_vector_type(2)));

constexpr int NR = 8;
constexpr unsigned XROW = 1024u;
constexpr unsigned WROW = 32768u;
constexpr unsigned X_BYTES = 4096u * XROW;
constexpr unsigned W_BYTES = 512u * WROW;
constexpr unsigned OUT_BYTES = 512u * WROW;
#define NPUSH 4
constexpr int NPW = NPUSH;
constexpr unsigned NMOVER = NPW * 7u;
constexpr unsigned NBATCH = 1408u;
constexpr int NTILE = 26;
constexpr int GRID = 256;

#define LAG 4
#define MV_AUX 0
#define RX_SLEEP 2
#define BK_SLEEP 1
#define REV 1
#define MV_SLEEP 2
#define IN_SLEEP 4
#define NOLOOK 0
#define LOOKAHEAD 1
#define PLAN_AT 15
#define X_NOPIECE 0
#define X_NORD 0
#define X_NOBAR 0
#define PRIO 10
#define JIT 1
#define GUESS_MASK 7u
#define EARLY 2
#define BAR_AT 2
#define R1_AT 0
#define RD2 0
#define D0 4
#define D1 9
#define D2 14
#define D3 19
#define D4 24
#define D5 29
#define D6 34
#define BK_POLL 3u
#define GO_AT 24
#define CHUNK 1
#define RPOLL 1
#define TOK0 4
#define TOK1 4
#define SV_AUX 0
#define OUT_AUX 16
#define PUSH_D 3

constexpr unsigned OFF_CTR = 0u;
constexpr unsigned OFF_TMO = 32768u;
constexpr unsigned OFF_CNT = 40960u;
constexpr unsigned OFF_FLG = 65536u;
constexpr unsigned OFF_CLM = 98304u;
constexpr unsigned OFF_YF = 131072u;
constexpr unsigned OFF_DBG = 262144u;
constexpr unsigned OFF_XS = 1u << 20;
constexpr unsigned XS_SRC = 4u << 20, XS_PAR = 8u * XS_SRC;
constexpr unsigned OFF_WS = OFF_XS + 2u * XS_PAR;
constexpr unsigned WS_SRC = 2u << 20, WS_PAR = 8u * WS_SRC;
constexpr unsigned Y_WAVE = 10240u, Y_PEER = NTILE * 8u * Y_WAVE, Y_PAR = NR * Y_PEER;
constexpr unsigned OFF_YL = OFF_WS + 2u * WS_PAR;
constexpr unsigned OFF_SV = OFF_YL + 2u * Y_PAR;
constexpr unsigned SV_WAVE = 20480u;
constexpr unsigned WS_NEED = OFF_SV + 7u * NTILE * 8u * SV_WAVE;

constexpr unsigned STAGE = 54272u;
constexpr unsigned LDS_CTL = 3u * STAGE;
constexpr unsigned LDS_BYTES = LDS_CTL + 64u;

constexpr unsigned long long DEADLINE_TICKS = 100000000ull;

struct Params {
    const char* x;
    const char* w;
    char* out;
    char* me;
    char* ws[NR];
    int rank;
    unsigned qmap;
};

__device__ __forceinline__ unsigned rfl(unsigned v) { return (unsigned)__builtin_amdgcn_readfirstlane((int)v); }

__device__ __forceinline__ unsigned fresh_lane() {
    unsigned l;
    asm volatile("v_mbcnt_lo_u32_b32 %0, -1, 0\n\tv_mbcnt_hi_u32_b32 %0, -1, %0" : "=v"(l));
    return l;
}
template <class T> __device__ __forceinline__ T sel3(unsigned k, T a, T b, T c) { return k == 0u ? a : (k == 1u ? b : c); }

__device__ __forceinline__ u32x4 mk_rsrc(const void* p, unsigned bytes) {
    const unsigned long long a = (unsigned long long)p;
    u32x4 r; r.x = (unsigned)a; r.y = (unsigned)(a >> 32) & 0xffffu; r.z = bytes; r.w = 0x00020000u; return r;
}
__device__ __forceinline__ __amdgpu_buffer_rsrc_t mk_brsrc(const void* p, unsigned bytes) {
    return __builtin_amdgcn_make_buffer_rsrc((void*)p, 0, (int)bytes, 0x00020000);
}

template <int OFF, bool SYS> __device__ __forceinline__ void dma16(u32x4 rs, unsigned voff, unsigned soff, unsigned ldsaddr) {
    if constexpr (SYS)
        asm volatile("s_mov_b32 m0, %0\n\ts_nop 0\n\tbuffer_load_dwordx4 %1, %2, %3 offen offset:%4 sc0 sc1 lds"
                     :: "s"(ldsaddr), "v"(voff), "s"(rs), "s"(soff), "i"(OFF) : "memory");
    else
        asm volatile("s_mov_b32 m0, %0\n\ts_nop 0\n\tbuffer_load_dwordx4 %1, %2, %3 offen offset:%4 lds"
                     :: "s"(ldsaddr), "v"(voff), "s"(rs), "s"(soff), "i"(OFF) : "memory");
}
template <int AUX> __device__ __forceinline__ void st128(u32x4 d, __amdgpu_buffer_rsrc_t rs, unsigned voff, unsigned soff) {
    __builtin_amdgcn_raw_buffer_store_b128(d, rs, voff, soff, AUX);
    asm volatile("s_nop 1" :: "v"(d) : "memory");
}
#define WAIT_V(n) asm volatile("s_waitcnt vmcnt(%0)" :: "i"(n) : "memory")
#define BAR() do { asm volatile("" ::: "memory"); __builtin_amdgcn_s_barrier(); asm volatile("" ::: "memory"); } while (0)

__device__ __forceinline__ unsigned ld_sys(const char* p) {
    return __hip_atomic_load((const GAS unsigned*)p, __ATOMIC_RELAXED, __HIP_MEMORY_SCOPE_SYSTEM);
}
__device__ __forceinline__ void st_sys(char* p, unsigned v) {
    __hip_atomic_store((GAS unsigned*)p, v, __ATOMIC_RELAXED, __HIP_MEMORY_SCOPE_SYSTEM);
}
__device__ __forceinline__ unsigned ld_agent(const char* p) {
    return __hip_atomic_load((const GAS unsigned*)p, __ATOMIC_RELAXED, __HIP_MEMORY_SCOPE_AGENT);
}
__device__ __forceinline__ void st_agent(char* p, unsigned v) {
    __hip_atomic_store((GAS unsigned*)p, v, __ATOMIC_RELAXED, __HIP_MEMORY_SCOPE_AGENT);
}

constexpr int PBASE = 3 * (8 - LAG);
__device__ __forceinline__ unsigned slice_of(unsigned round, unsigned h) { return (REV && h != 0u && round >= 8u - LAG) ? 15u - LAG - round : round; }
__device__ __forceinline__ unsigned word_of(unsigned k) {
    unsigned s, type;
    if (k < (unsigned)PBASE) { s = k / 3u; type = k - 3u * s; }
    else if (k < PBASE + 2u * LAG) { s = 8u - LAG + ((k - PBASE) >> 1); type = (k - PBASE) & 1u; }
    else { s = slice_of(min(8u - LAG + (k - PBASE - 2u * LAG), 7u), 1u); type = 2u; }
    return 4u * s + (type < 2u ? 1u - type : 2u);
}

__device__ __forceinline__ unsigned pk_bf16(float lo, float hi) {
    f32x2 v = {lo, hi};
    bf16x2 b = __builtin_convertvector(v, bf16x2);
    return __builtin_bit_cast(unsigned, b);
}
__device__ __forceinline__ float silu(float y) {
    const float c = __builtin_amdgcn_fmed3f(y, -60.0f, 60.0f);
    const float e = __builtin_amdgcn_exp2f(c * -1.44269504088896341f);
    return y * __builtin_amdgcn_rcpf(1.0f + e);
}

#define STAMP(i) do { } while (0)

struct Batch { unsigned piece, isx, so, dofs; };
__device__ __forceinline__ Batch batch_of(unsigned T, unsigned rank, unsigned q, unsigned par) {
    constexpr unsigned B0 = 176u * (8u - LAG), B1 = B0 + 120u * LAG;
    Batch b;
    unsigned s, r, type;
    if (T < B0)      { s = T / 176u; r = T - 176u * s; type = r < 64u ? 0u : (r < 120u ? 1u : 2u); b.piece = 3u * s + type; }
    else if (T < B1) { const unsigned u = (T - B0) / 120u; r = T - B0 - 120u * u; s = 8u - LAG + u; type = r < 64u ? 0u : 1u; b.piece = PBASE + 2u * u + type; }
    else             { const unsigned u = (T - B1) / 56u; r = T - B1 - 56u * u + 120u; s = slice_of(8u - LAG + u, 1u); type = 2u; b.piece = PBASE + 2u * LAG + u; }
    if (type == 0u) {
        b.isx = 0u;
        b.so = (s * 64u + (r & 1u) * 32u) * WROW + q * 4096u + (r >> 1) * 128u;
        b.dofs = OFF_WS + par * WS_PAR + rank * WS_SRC + s * (256u << 10) + (r >> 1) * 8192u + (r & 1u) * 2048u;
    } else {
        const unsigned e = r - (type == 1u ? 64u : 120u), blk = ((rank + 1u + (e >> 3)) & 7u) * 64u + (type - 1u) * 32u + (e & 7u) * 4u;
        b.isx = 1u;
        b.so = blk * (8u * XROW) + s * 128u;
        b.dofs = OFF_XS + par * XS_PAR + rank * XS_SRC + s * (512u << 10) + blk * 1024u;
    }
    b.piece = rfl(b.piece); b.isx = rfl(b.isx); b.so = rfl(b.so); b.dofs = rfl(b.dofs);
    asm("" : "+s"(b.piece), "+s"(b.isx), "+s"(b.so), "+s"(b.dofs));
    return b;
}

static_assert(NMOVER <= 64u && NPW + NTILE <= 31, "");
__device__ __forceinline__ void first_loads(const Params& P, unsigned q, unsigned W, u32x4 (&r)[4]) {
    const unsigned lane = threadIdx.x & 63u;
    const __amdgpu_buffer_rsrc_t rs = mk_brsrc(P.w, W_BYTES);
    const unsigned vw = (lane >> 2) * WROW + (((lane & 3u) ^ ((lane >> 5) << 1)) * 16u), so = (W & 1u) * 32u * WROW + q * 4096u + (W >> 1) * 128u;
#pragma unroll
    for (int i = 0; i < 4; ++i) r[i] = __builtin_amdgcn_raw_buffer_load_b128(rs, vw, so + (i & 1) * 64u + (i >> 1) * (16u * WROW), 0);
}

__device__ __forceinline__ void pusher(const Params& P, unsigned q, unsigned j, unsigned call, u32x4 (&cur)[4], LAS char* lds, unsigned dbg_role) {
    const unsigned lane = threadIdx.x & 63u, w = rfl(threadIdx.x >> 6);
    const unsigned rank = (unsigned)P.rank;
    char* const me = P.me;
    char* const peer = P.ws[q];
    volatile LAS unsigned* const lw = (volatile LAS unsigned*)lds;

    if (w == 0u) {
        GAS unsigned* const cnt = (GAS unsigned*)(peer + OFF_FLG + rank * 256u) + word_of(lane);
        unsigned seen = 0u;
        for (;;) {
            const unsigned gone = rfl(lw[63]), v = lw[lane], d = v - seen;
            const unsigned long long busy = __ballot(v != 0u);
            if (lane < 32u && d != 0u && ((busy >> lane) >> 1) != 0ull) {
                __hip_atomic_fetch_add(cnt, d, __ATOMIC_RELAXED, __HIP_MEMORY_SCOPE_SYSTEM);
                seen = v;
            }
            if (gone == 7u) break;
            __builtin_amdgcn_s_sleep(1);
        }
        return;
    }

    const unsigned W = j * 7u + w - 1u;
    const unsigned vx = (lane >> 3) * XROW + (((lane & 7u) ^ (lane >> 4)) * 16u);
    const unsigned vw = (lane >> 2) * WROW + (((lane & 3u) ^ ((lane >> 5) << 1)) * 16u);
    const unsigned vd = lane * 16u;
    const unsigned long long pw = (unsigned long long)P.w, px = (unsigned long long)P.x;
    u32x4 nxt[4];
    const unsigned par = call & 1u;
    Batch b0 = {0u, 0u, 0u, OFF_WS + par * WS_PAR + rank * WS_SRC + (W >> 1) * 8192u + (W & 1u) * 2048u}, b1 = batch_of(NMOVER + W, rank, q, par);
    bool v1 = true;
    __builtin_amdgcn_s_waitcnt(0x0F70);
    const __amdgpu_buffer_rsrc_t rd = mk_brsrc(peer, OFF_YL);
    const __amdgpu_buffer_rsrc_t rc = mk_brsrc(me + OFF_CNT + q * 256u, 256u);
    const unsigned vc = lane == 0u ? 0u : 4096u;
    const unsigned tbase = (call - 1u) * NBATCH - 2u * NMOVER;
    constexpr int D = PUSH_D;
    static_assert(D >= 1 && D <= 4 && GO_AT < 2 * (int)NMOVER, "");
    unsigned pprev = 62u;
    auto bump = [&](unsigned word) { if (lane == 0u) __hip_atomic_fetch_add((LAS unsigned*)lds + word, 1u, __ATOMIC_RELAXED, __HIP_MEMORY_SCOPE_WORKGROUP); };
    STAMP(0);
    for (;;) {
        const unsigned tk = (unsigned)__builtin_amdgcn_raw_ptr_buffer_atomic_add_i32(1, rc, vc, 0, 0);
        {
            const __amdgpu_buffer_rsrc_t rs = mk_brsrc((const void*)(b1.isx ? px : pw), v1 ? (b1.isx ? X_BYTES : W_BYTES) : 0u);
            const unsigned v0 = b1.isx ? vx : vw, v1_ = b1.isx ? (vx ^ 64u) : vw;
            const unsigned o1 = b1.isx ? 8u * XROW : 64u, o2 = b1.isx ? 16u * XROW : 16u * WROW;
#pragma unroll
            for (int i = 0; i < 4; ++i) nxt[i] = __builtin_amdgcn_raw_buffer_load_b128(rs, (i & 1) ? v1_ : v0, b1.so + (i & 1) * o1 + (i >> 1) * o2, 0);
        }
        asm volatile("" ::: "memory");
        const unsigned o1 = b0.isx ? 1024u : 4096u, o2 = b0.isx ? 2048u : 1024u;
#pragma unroll
        for (int i = 0; i < 4; ++i) {
            st128<17>(cur[i], rd, vd, b0.dofs + (i & 1) * o1 + (i >> 1) * o2);
            if (i < D - 1) WAIT_V(D + 5); else WAIT_V(D);
            if (i == D - 1 && pprev != 62u) bump(pprev);
        }
        pprev = b0.piece;
        const unsigned T = rfl(tk) - tbase;
        if (T == NBATCH + GO_AT && lane == 0u) st_sys(peer + OFF_FLG + rank * 256u + 128u, call);
        if (!v1) break;
#pragma unroll
        for (int i = 0; i < 4; ++i) cur[i] = nxt[i];
        b0 = b1; v1 = T < NBATCH; b1 = batch_of(v1 ? T : 0u, rank, q, par);
    }
    WAIT_V(0);
    bump(pprev);
    bump(63u);
    STAMP(1);
}

__device__ __forceinline__ void gemm_tile(const Params& P, unsigned q, unsigned tile, unsigned call, LAS char* lds, unsigned dbg_role) {
    unsigned lane = threadIdx.x & 63u;
    const unsigned w = rfl(threadIdx.x >> 6), wm = w >> 1, wn = w & 1u;
    const unsigned rank = (unsigned)P.rank, g = (q - rank - 1u) & 7u;
    const unsigned h = tile & 1u, c = tile >> 1;
    const bool own = g == 7u;
    char* const me = P.me;
    const unsigned par = call & 1u;
    const unsigned lds0 = (unsigned)(uintptr_t)lds;
    LAS unsigned* ctl = (LAS unsigned*)(lds + LDS_CTL);
    STAMP(0);

    const unsigned b0 = w < 4u ? 3u * w : 12u + 2u * (w - 4u);
    const unsigned cbmax = c == 12u ? 3u : 4u;
    const unsigned nraw = own ? 8u : 16u, nsrc = own ? 7u : 6u;
    const unsigned vF = (lane >> 3) * 256u + slice_of(lane & 7u, h) * 16u;
    u32x4 ra, rb, rx;
    unsigned za, zb, zx1, vP[7];
    const unsigned zx0 = w == 7u ? 2048u : 0u, mx = w < 4u ? ~0u : 0u;
    auto staging = [&](bool raw, unsigned lane) {
        ra = raw ? mk_rsrc(P.x, X_BYTES) : mk_rsrc(me + OFF_XS + par * XS_PAR, XS_PAR);
        rb = raw ? mk_rsrc(P.w, W_BYTES) : mk_rsrc(me + OFF_WS + par * WS_PAR, WS_PAR);
        rx = w < 4u ? rb : mk_rsrc(me + OFF_FLG, zx0);
        za = ra.z; zb = rb.z; zx1 = rx.z;
        const unsigned va = (lane >> 3) * XROW + (((lane & 7u) ^ (lane >> 4)) * 16u);
        const unsigned vb = (lane >> 2) * WROW + (((lane & 3u) ^ ((lane >> 5) << 1)) * 16u);
#pragma unroll
        for (int i = 0; i < 4; ++i) vP[i] = (raw ? (va ^ ((i & 1) * 64u)) + (4u * w + i) * (8u * XROW) : lane * 16u + (4u * w + i) * 1024u) - i * 1024u;
#pragma unroll
        for (int i = 0; i < 3; ++i) { const unsigned b = b0 + i, cb = min(b >> 2, cbmax);
            vP[4 + i] = raw ? vb + cb * 64u + (b & 3u) * (16u * WROW) : lane * 16u + cb * 4096u + (b & 3u) * 1024u; }
        if (w >= 4u) vP[6] = (lane >> 3) * 256u + slice_of(lane & 7u, h) * 16u;
    };
    staging(true, lane);
    unsigned tabA, tabB;
    {
        const bool extra = !own && lane < 8u;
        const unsigned s = lane & 7u, e = lane - nraw, r = own ? e / 7u : e / 6u, i = e - r * nsrc;
        const unsigned p = (rank + 1u + i + (i >= g ? 1u : 0u)) & 7u, sl = slice_of(r & 7u, h);
        tabA = lane < nraw ? ((extra ? rank : q) * 512u + h * 256u) * XROW + s * 128u : p * XS_SRC + sl * (512u << 10) + (q * 2u + h) * 32768u;
        tabB = lane < nraw ? s * 64u * WROW + (extra ? q : rank) * 4096u + c * 320u : p * WS_SRC + sl * (256u << 10) + c * 20480u;
    }
    const unsigned la0 = lds0 + w * 4096u, lb0 = lds0 + 32768u + b0 * 1024u;
    const unsigned lx0 = w < 4u ? lb0 + 2048u : (w == 7u ? lds0 + 53248u : lds0 + 0x30000u), kx = (w < 4u || w == 7u) ? ~0u : 0u;
    unsigned oa = 0u, ob = 0u;
    auto piece = [&](int i, unsigned so) {
        if (i == 0) dma16<0, false>(ra, vP[0], oa, la0 + so);
        if (i == 1) dma16<1024, false>(ra, vP[1], oa, la0 + so);
        if (i == 2) dma16<2048, false>(ra, vP[2], oa, la0 + so);
        if (i == 3) dma16<3072, false>(ra, vP[3], oa, la0 + so);
        if (i == 4) dma16<0, false>(rb, vP[4], ob, lb0 + so);
        if (i == 5) dma16<0, false>(rb, vP[5], ob, lb0 + so + 1024u);
        if (i == 6) dma16<0, true>(rx, vP[6], ob & mx, lx0 + (so & kx));
    };
    unsigned it = 0u;
    unsigned gm = 0u;
    auto planA = [&]() { oa = (unsigned)__builtin_amdgcn_readlane((int)tabA, (int)(it & 63u)); ra.z = za & gm; asm volatile("" : "+s"(oa), "+s"(ra)); };
    auto planB = [&]() { ob = (unsigned)__builtin_amdgcn_readlane((int)tabB, (int)(it & 63u)); rb.z = zb & gm; asm volatile("" : "+s"(ob), "+s"(rb)); };
    auto planX = [&]() { rx.z = (zx1 & gm) | zx0; it -= gm; asm volatile("" : "+s"(it), "+s"(rx)); };
    auto plan = [&]() { gm = ~0u; planA(); planB(); planX(); asm volatile("s_nop 4"); };
    const unsigned long long fmask = ~((0xffull << (8u * rank)) | (0xffull << (8u * q)));
    const unsigned need0 = (h ? 64u : 56u) * call, need1 = (h ? 56u : 64u) * call;
    auto rounds_in = [&](unsigned v0, unsigned v1) {
        const unsigned long long miss = __ballot((int)(v0 - need0) < 0 || (int)(v1 - need1) < 0) & fmask;
        unsigned m = (unsigned)miss | (unsigned)(miss >> 32);
        m |= m >> 16; m |= m >> 8;
        return (unsigned)__builtin_ctz(m | 0x100u);
    };
    const unsigned foff = 53248u + h * 4u + lane * 16u;

    unsigned g16 = lane >> 4;
    const unsigned aoff = wm * 8192u + (lane & 15u) * 128u + ((g16 ^ ((lane & 15u) >> 1)) * 16u);
    const unsigned lp = (g16 >> 1) * 1024u + (8u * (g16 & 1u) + ((lane & 15u) >> 2)) * 64u + (lane & 3u) * 8u;
    const unsigned be = 32768u + ((5u * wn) >> 1) * 4096u + lp + (((wn ^ g16) & 1u) * 32u);
    const unsigned bo = 32768u + ((5u * wn + 1u) >> 1) * 4096u + lp + (((wn ^ g16 ^ 1u) & 1u) * 32u);

    f32x4 acc[4][5];
#pragma unroll
    for (int m = 0; m < 4; ++m)
#pragma unroll
        for (int n = 0; n < 5; ++n) acc[m][n] = (f32x4){0.f, 0.f, 0.f, 0.f};
    char* const svp = me + OFF_SV + ((min(g, 6u) * NTILE + tile) * 8u + w) * SV_WAVE;
    s16x8 fa[2][4]; s16x4 fb[2][5][2];
    auto zero1 = [&]() {
#pragma unroll
        for (int m = 0; m < 4; ++m) fa[1][m] = (s16x8){0, 0, 0, 0, 0, 0, 0, 0};
#pragma unroll
        for (int n = 0; n < 5; ++n) { fb[1][n][0] = (s16x4){0, 0, 0, 0}; fb[1][n][1] = (s16x4){0, 0, 0, 0}; }
    };
    auto rd = [&](int kh, int r, LAS char* st) {
        if (r >= 2 && r < 6) fa[kh][r - 2] = *(const LAS s16x8*)(st + (kh ? (aoff ^ 64u) : aoff) + (r - 2) * 2048);
        else { const int n = r < 2 ? 0 : (r - 4) >> 1, hi = r & 1;
               fb[kh][n][hi] = __builtin_amdgcn_ds_read_tr16_b64_v4i16((LAS s16x4*)(st + ((n & 1) ? bo : be) + (n >> 1) * 4096 + kh * 2048 + hi * 256)); }
    };
    auto mm = [&](int kh, int i) {
        const int n = i >> 2, m = i & 3;
        acc[m][n] = __builtin_amdgcn_mfma_f32_16x16x32_bf16(
            __builtin_bit_cast(bf16x8, __builtin_shufflevector(fb[kh][n][0], fb[kh][n][1], 0, 1, 2, 3, 4, 5, 6, 7)),
            __builtin_bit_cast(bf16x8, fa[kh][m]), acc[m][n], 0, 0, 0);
    };
    zero1();

    unsigned t = 0u, ready = nraw, gain = 0u;
    unsigned s0 = 0u, s1 = STAGE, s2 = 2u * STAGE;
    bool bad = false;
    for (;;) {
        if (it == t) {
            plan();
#pragma unroll
            for (int i = 0; i < 7; ++i) piece(i, s0);
            plan();
#pragma unroll
            for (int i = 0; i < 7; ++i) piece(i, s1);
            plan();
        }
        const unsigned park = (!own && t < 8u) ? 8u : 64u;
        do {
            LAS char* st = lds + s0;
#pragma unroll
            for (int i = 0; i < 20; ++i) {
                if (i == BAR_AT) {
                    __builtin_amdgcn_s_waitcnt(0xC07F);
                    WAIT_V(7);
                    BAR();
                    __builtin_amdgcn_sched_barrier(0);
                    if (PRIO) __builtin_amdgcn_s_setprio(3);
                }
                if (PRIO && i == BAR_AT + 10) __builtin_amdgcn_s_setprio(2);
                mm(1, i);
                if (RD2 == 0 && i >= BAR_AT && i < BAR_AT + 14) rd(0, i - BAR_AT, st);
                if (RD2 == 1 && i >= BAR_AT && i < BAR_AT + 7) { rd(0, 2 * (i - BAR_AT), st); rd(0, 2 * (i - BAR_AT) + 1, st); }
                if (i == D0) piece(0, s2);
                if (i == D1) piece(1, s2);
                if (i == D2) piece(2, s2);
                if (i == D3) piece(3, s2);
                if (i == 17 && __builtin_expect(it == ready && it == t + 3u, 0)) {
                    const LAS unsigned* fr = (const LAS unsigned*)(st + foff);
                    ready = max(ready, nraw + rounds_in(fr[0], fr[1]) * gain);
                }
                if (i == 18) asm volatile("s_cmp_lt_u32 %1, %2\n\ts_cselect_b32 %0, -1, 0" : "=s"(gm) : "s"(it), "s"(ready) : "scc");
                __builtin_amdgcn_sched_barrier(0);
            }
#pragma unroll
            for (int i = 0; i < 20; ++i) {
                if (PRIO && i == BAR_AT) __builtin_amdgcn_s_setprio(1);
                if (PRIO && i == BAR_AT + 10) __builtin_amdgcn_s_setprio(0);
                mm(0, i);
                if (RD2 == 0 && i >= R1_AT && i < R1_AT + 14) rd(1, i - R1_AT, st);
                if (RD2 == 1 && i >= R1_AT && i < R1_AT + 7) { rd(1, 2 * (i - R1_AT), st); rd(1, 2 * (i - R1_AT) + 1, st); }
                if (i + 20 == D3) piece(3, s2);
                if (i + 20 == D4) piece(4, s2);
                if (i + 20 == D5) piece(5, s2);
                if (i + 20 == D6) piece(6, s2);
                if (i == 17) planA();
                if (i == 18) planB();
                if (i == 19) planX();
                __builtin_amdgcn_sched_barrier(0);
            }
            { const unsigned x = s0; s0 = s1; s1 = s2; s2 = x; }
            ++t;
        } while (t < min(it, park));
        if (t == 64u) break;
        if (t < it) {
#pragma unroll
            for (int i = 0; i < 20; ++i) mm(1, i);
            const __amdgpu_buffer_rsrc_t rv = mk_brsrc(svp, SV_WAVE);
#pragma unroll
            for (int m = 0; m < 4; ++m)
#pragma unroll
                for (int n = 0; n < 5; ++n) { st128<SV_AUX>(__builtin_bit_cast(u32x4, acc[m][n]), rv, lane * 16u, (n * 4 + m) * 1024); acc[m][n] = (f32x4){0.f, 0.f, 0.f, 0.f}; }
            zero1();
            continue;
        }
        STAMP(1 + 2 * seg);
        if (t == nraw) {
            staging(false, lane);
            gain = LOOKAHEAD ? nsrc : 0u;
        }
        if (t == ready) {
            if (w == 0u) {
                const char* fp = me + OFF_FLG + vF + h * 4u;
                const unsigned long long t0 = __builtin_amdgcn_s_memrealtime();
                unsigned res;
                for (unsigned n = 1u;; ++n) {
                    const unsigned v0 = ld_sys(fp), v1 = ld_sys(fp + 4);
                    res = nraw + rounds_in(v0, v1) * nsrc;
                    if (res > t) break;
                    if ((n & 15u) == 0u && (rfl(ld_agent(me + OFF_TMO)) == call || __builtin_amdgcn_s_memrealtime() - t0 > DEADLINE_TICKS)) {
                        st_agent(me + OFF_TMO, call);
                        res = 0x100u + 64u; break;
                    }
                    __builtin_amdgcn_s_sleep(IN_SLEEP);
                }
                ctl[0] = res;
                asm volatile("s_waitcnt vmcnt(0) lgkmcnt(0)" ::: "memory");
            }
            BAR();
            const unsigned res = rfl(*(volatile LAS unsigned*)ctl);
            bad |= res > 0xffu;
            ready = res & 0xffu;
        }
        STAMP(2 + 2 * seg);
    }
#pragma unroll
    for (int i = 0; i < 20; ++i) mm(1, i);
    STAMP(20);

    lane = fresh_lane(); g16 = lane >> 4;
    const unsigned nu = (c == 12u && wn == 1u) ? 6u : 10u;
    const __amdgpu_buffer_rsrc_t ro = mk_brsrc(P.out, OUT_BYTES);
    const unsigned vo = ((g16 & 1u) * 16u + (lane & 15u)) * WROW + (g16 >> 1) * 16u;
    const unsigned vo2 = nu == 10u ? vo : 0x80000000u;
    unsigned so0 = (h * 256u + wm * 64u) * WROW + q * 4096u + c * 320u + wn * 160u, so1 = so0 + 32u * WROW;
    asm("" : "+s"(so0), "+s"(so1));
    auto emit = [&](int u, const f32x4& v0, const f32x4& v1) {
        unsigned a0 = pk_bf16(silu(v0[0]), silu(v0[1])), a1 = pk_bf16(silu(v0[2]), silu(v0[3]));
        unsigned c0 = pk_bf16(silu(v1[0]), silu(v1[1])), c1 = pk_bf16(silu(v1[2]), silu(v1[3]));
        const auto s0 = __builtin_amdgcn_permlane16_swap(a0, c0, false, false);
        const auto s1 = __builtin_amdgcn_permlane16_swap(a1, c1, false, false);
        u32x4 d = {s0[0], s1[0], s0[1], s1[1]};
        if (bad) d = (u32x4){0x7fc07fc0u, 0x7fc07fc0u, 0x7fc07fc0u, 0x7fc07fc0u};
        st128<OUT_AUX>(d, ro, (u < 6 ? vo : vo2) + (u >> 1) * 32u, (u & 1) ? so1 : so0);
    };
    if (own) {
        bad |= rfl(ld_agent(me + OFF_TMO)) == call;
#pragma unroll
        for (int u = 0; u < 10; ++u) emit(u, acc[2 * (u & 1)][u >> 1], acc[2 * (u & 1) + 1][u >> 1]);
        STAMP(23);
        return;
    }
    char* const peer = P.ws[q];
    const unsigned yo = OFF_YL + par * Y_PAR + (tile * 8u + w) * Y_WAVE;
    {
        const __amdgpu_buffer_rsrc_t ry = mk_brsrc(me + yo + q * Y_PEER, Y_WAVE);
#pragma unroll
        for (int u = 0; u < 10; ++u) if ((unsigned)u < nu) {
            const f32x4 v0 = acc[2 * (u & 1)][u >> 1], v1 = acc[2 * (u & 1) + 1][u >> 1];
            u32x4 d; d.x = pk_bf16(v0[0], v0[1]); d.y = pk_bf16(v0[2], v0[3]); d.z = pk_bf16(v1[0], v1[1]); d.w = pk_bf16(v1[2], v1[3]);
            if (bad) d = (u32x4){0x7fc07fc0u, 0x7fc07fc0u, 0x7fc07fc0u, 0x7fc07fc0u};
            st128<17>(d, ry, lane * 16u, u * 1024);
        }
        WAIT_V(0);
        BAR();
        if (w == 0u && lane == 0u) st_sys((RPOLL ? me + q * 4096u : peer + rank * 4096u) + OFF_YF + tile * 128u, call);
    }
    STAMP(21);
    const __amdgpu_buffer_rsrc_t rv = mk_brsrc(svp, SV_WAVE);
    auto unpark = [&](int u) {
#pragma unroll
        for (int k = 0; k < 2; ++k) acc[2 * (u & 1) + k][u >> 1] = __builtin_bit_cast(f32x4, __builtin_amdgcn_raw_buffer_load_b128(rv, lane * 16u, ((u >> 1) * 4 + 2 * (u & 1) + k) * 1024, 0));
    };
    if (w == 0u) {
        const char* const fy = (RPOLL ? peer + rank * 4096u : me + q * 4096u) + OFF_YF + tile * 128u;
        const char* fp = lane == 0u ? fy : (lane == 1u ? me + OFF_FLG + q * 256u + 128u : me + OFF_TMO);
        const unsigned long long t0 = __builtin_amdgcn_s_memrealtime();
        bool have = false;
        for (;;) {
            const unsigned v = ld_sys(fp);
            if (__builtin_amdgcn_readlane((int)v, 2) == (int)call) { bad = true; break; }
            const unsigned m = (unsigned)__ballot((int)(v - call) >= 0);
            if (!have && (m & 1u)) { have = true; if (lane == 0u) fp = me + OFF_TMO; }
            if (have && (m & 2u)) break;
            if (__builtin_amdgcn_s_memrealtime() - t0 > DEADLINE_TICKS) {
                if (lane == 0u) st_agent(me + OFF_TMO, call);
                bad = true; break;
            }
            __builtin_amdgcn_s_sleep(RX_SLEEP);
        }
        if (lane == 0u) ctl[0] = bad ? 1u : 0u;
        asm volatile("s_waitcnt lgkmcnt(0)" ::: "memory");
    }
    BAR();
    bad |= rfl(*(volatile LAS unsigned*)ctl) != 0u;
    STAMP(22);
    const __amdgpu_buffer_rsrc_t rp = mk_brsrc(peer + yo + rank * Y_PEER, bad ? 0u : nu * 1024u);
    u32x4 pv[10];
    auto finish = [&](int u) {
        const u32x4 d = pv[u];
        f32x4 v0 = acc[2 * (u & 1)][u >> 1], v1 = acc[2 * (u & 1) + 1][u >> 1];
        v0[0] += __builtin_bit_cast(float, d.x << 16); v0[1] += __builtin_bit_cast(float, d.x & 0xffff0000u);
        v0[2] += __builtin_bit_cast(float, d.y << 16); v0[3] += __builtin_bit_cast(float, d.y & 0xffff0000u);
        v1[0] += __builtin_bit_cast(float, d.z << 16); v1[1] += __builtin_bit_cast(float, d.z & 0xffff0000u);
        v1[2] += __builtin_bit_cast(float, d.w << 16); v1[3] += __builtin_bit_cast(float, d.w & 0xffff0000u);
        emit(u, v0, v1);
    };
    constexpr int C = CHUNK;
    static_assert(10 % C == 0 && C <= TOK0 && C <= TOK1, "");
    const unsigned tokens = h ? TOK1 : TOK0;
    auto fetch = [&](int k) {
        unsigned n = 0u;
        if (lane == 0u) n = __hip_atomic_fetch_add(ctl + 4, (unsigned)C, __ATOMIC_RELAXED, __HIP_MEMORY_SCOPE_WORKGROUP);
        n = rfl(n);
        while (n + C > rfl(*(volatile LAS unsigned*)(ctl + 5)) + tokens) __builtin_amdgcn_s_sleep(1);
#pragma unroll
        for (int u = C * k; u < C * k + C; ++u) pv[u] = __builtin_amdgcn_raw_buffer_load_b128(rp, lane * 16u + u * 1024u, 0, 17);
        asm volatile("" ::: "memory");
#pragma unroll
        for (int u = C * k; u < C * k + C; ++u) unpark(u);
        asm volatile("" ::: "memory");
    };
    auto landed = [&](int u) {
        asm volatile("" : "+v"(pv[u]));
        if (lane == 0u) __hip_atomic_fetch_add(ctl + 5, 1u, __ATOMIC_RELAXED, __HIP_MEMORY_SCOPE_WORKGROUP);
    };
    fetch(0);
#pragma unroll
    for (int k = 0; k < 10 / C; ++k) {
#pragma unroll
        for (int u = C * k; u < C * k + C; ++u) landed(u);
        if (k + 1 < 10 / C) fetch(k + 1);
        __builtin_amdgcn_sched_barrier(0);
#pragma unroll
        for (int u = C * k; u < C * k + C; ++u) finish(u);
        __builtin_amdgcn_sched_barrier(0);
    }
    STAMP(23);
}

__global__ void __launch_bounds__(512, 2) fused_gemm_rs(Params P) {
    __shared__ __attribute__((aligned(16))) char smem[LDS_BYTES];
    LAS char* lds = (LAS char*)smem;
    volatile LAS unsigned* const lw = (volatile LAS unsigned*)lds;
    const unsigned w = rfl(threadIdx.x >> 6), rank = (unsigned)P.rank;
    const unsigned guess = (blockIdx.x & ~7u) | ((unsigned)__builtin_amdgcn_s_getreg((3 << 11) | 20) & GUESS_MASK);
    u32x4 cur[4];
    auto prefetch = [&](unsigned role) {
        const unsigned q = (P.qmap >> ((role & 7u) * 4u)) & 7u, slot = role >> 3;
        if (slot < (unsigned)NPW && w != 0u) first_loads(P, q, slot * 7u + w - 1u, cur);
    };
    if (w == 0u) {
        char* const me = P.me;
        const __amdgpu_buffer_rsrc_t rc = mk_brsrc(me + OFF_CLM, 32768u);
        const unsigned v0 = threadIdx.x == 0u ? 0u : 65536u;
        unsigned role = guess, old = (unsigned)__builtin_amdgcn_raw_ptr_buffer_atomic_add_i32(1, rc, v0, role * 128u, 0);
        const unsigned done = rfl(ld_agent(me + OFF_CTR + blockIdx.x * 128u));
        lw[threadIdx.x] = 0u;
        while (rfl(old) != done) {
            __builtin_amdgcn_raw_ptr_buffer_atomic_add_i32(-1, rc, v0, role * 128u, 0);
            role = (role + 1u) & 255u;
            if (role == guess) { role = 255u; st_agent(me + OFF_TMO, done + 1u); break; }
            old = (unsigned)__builtin_amdgcn_raw_ptr_buffer_atomic_add_i32(1, rc, v0, role * 128u, 0);
        }
        if (threadIdx.x == 0u) { lw[LDS_CTL / 4u + 1u] = role; lw[LDS_CTL / 4u + 2u] = done + 1u; lw[LDS_CTL / 4u + 4u] = 0u; lw[LDS_CTL / 4u + 5u] = 0u; st_agent(me + OFF_CTR + blockIdx.x * 128u, done + 1u); }
        asm volatile("s_waitcnt lgkmcnt(0)" ::: "memory");
    } else prefetch(guess);
    BAR();
    const unsigned role = rfl(lw[LDS_CTL / 4u + 1u]), call = rfl(lw[LDS_CTL / 4u + 2u]);
    if (role != guess) prefetch(role);
    const unsigned q = (P.qmap >> ((role & 7u) * 4u)) & 7u, slot = role >> 3;
    if (slot < (unsigned)NPW) { if (q != rank) pusher(P, q, slot, call, cur, lds, role); }
    else if (slot < (unsigned)(NPW + NTILE)) gemm_tile(P, q, slot - NPW, call, lds, role);
}

unsigned make_qmap(int rank) {
    static const unsigned char kNear[NR][NR] = {
        {6, 0, 2, 3, 1, 5, 4, 7}, {7, 1, 4, 5, 0, 3, 6, 2}, {3, 2, 0, 6, 4, 5, 7, 1}, {2, 3, 6, 0, 5, 1, 7, 4},
        {5, 4, 1, 7, 2, 0, 6, 3}, {4, 5, 7, 1, 3, 2, 0, 6}, {0, 6, 3, 2, 7, 4, 1, 5}, {1, 7, 5, 4, 6, 3, 2, 0}};
    unsigned m = 0u;
    for (int g = 0; g < NR; ++g) m |= (unsigned)kNear[rank][g] << (4 * g);
    return m;
}

}

extern "C" void kernel_launch(int rank, int nranks, ncclComm_t comm,
                              hipStream_t stream,
                              void* const* d_in, const int* in_sizes, int n_in,
                              void* d_out, int out_size,
                              const int* in_dtypes, int out_dtype,
                              void* const* d_out_all,
                              void* const* d_ws_all, size_t ws_bytes) {
    (void)comm; (void)in_dtypes; (void)out_dtype; (void)d_out_all;
    if (nranks != NR || (unsigned)rank >= (unsigned)NR || n_in != 2 || ws_bytes < (size_t)WS_NEED ||
        in_sizes[0] != 4096 * 512 || in_sizes[1] != 512 * 16384 || out_size != 512 * 16384) {
        fprintf(stderr, "unexpected problem geometry\n");
        return;
    }
    Params P;
    P.x = (const char*)d_in[0];
    P.w = (const char*)d_in[1];
    P.out = (char*)d_out;
    for (int p = 0; p < NR; ++p) P.ws[p] = (char*)d_ws_all[p];
    P.me = P.ws[rank];
    P.rank = rank;
    P.qmap = make_qmap(rank);
    fused_gemm_rs<<<GRID, 512, 0, stream>>>(P);
    HIP_CHECK(hipGetLastError());
}
```

```cpp
#include <hip/hip_runtime.h>
#include <rccl/rccl.h>
#include <cstdio>
#include <cstdint>

#define HIP_CHECK(expr) do { hipError_t _e = (expr); if (_e != hipSuccess) { \
    fprintf(stderr, "HIP error %s at %s:%d\n", hipGetErrorString(_e), __FILE__, __LINE__); \
    return; } } while (0)

#define LAS __attribute__((address_space(3)))
#define GAS __attribute__((address_space(1)))

#define LAG 4
#define FILL 3
#define GO_AT 0
#define TOK 4
#define DMA_SC "sc1"
#define GUESS_MASK 7u

namespace {

typedef __bf16 bf16x8 __attribute__((ext_vector_type(8)));
typedef __bf16 bf16x2 __attribute__((ext_vector_type(2)));
typedef short s16x4 __attribute__((ext_vector_type(4)));
typedef short s16x8 __attribute__((ext_vector_type(8)));
typedef float f32x4 __attribute__((ext_vector_type(4)));
typedef float f32x2 __attribute__((ext_vector_type(2)));
typedef unsigned u32x4 __attribute__((ext_vector_type(4)));

constexpr int NR = 8;
constexpr unsigned XROW = 1024u;
constexpr unsigned WROW = 32768u;
constexpr unsigned X_BYTES = 4096u * XROW, W_BYTES = 512u * WROW, OUT_BYTES = 512u * WROW;
constexpr unsigned NSW = 2u, NPW = 4u, NTILE = 26u;
constexpr unsigned NMOVER = NPW * 7u;
constexpr unsigned NSTAGE = 8u * NSW + NPW;
constexpr unsigned NPULL = 1344u;
constexpr int GRID = 256;
static_assert(NSW + NPW + NTILE == 32u && NSTAGE * 8u * 4u == 640u && NSTAGE * 7u >= 128u, "");

constexpr unsigned OFF_CTR = 0u;
constexpr unsigned OFF_TMO = 32768u;
constexpr unsigned OFF_CNT = 40960u;
constexpr unsigned OFF_STG = 49152u;
constexpr unsigned OFF_GO = 53248u;
constexpr unsigned OFF_FLG = 65536u;
constexpr unsigned OFF_CLM = 98304u;
constexpr unsigned OFF_YF = 131072u;
constexpr unsigned OFF_DBG = 262144u;
constexpr unsigned OFF_XS = 1u << 20;
constexpr unsigned XS_SRC = 4u << 20;
constexpr unsigned OFF_WS = OFF_XS + 8u * XS_SRC;
constexpr unsigned WS_SRC = 9u * (256u << 10);
constexpr unsigned OFF_SX = OFF_WS + 8u * WS_SRC;
constexpr unsigned OFF_SW = OFF_SX + 2u * X_BYTES;
constexpr unsigned Y_WAVE = 10240u, Y_PEER = NTILE * 8u * Y_WAVE, Y_PAR = NR * Y_PEER;
constexpr unsigned OFF_YL = OFF_SW + 2u * W_BYTES;
constexpr unsigned OFF_SV = OFF_YL + 2u * Y_PAR;
constexpr unsigned SV_WAVE = 20480u;
constexpr unsigned WS_NEED = OFF_SV + 7u * NTILE * 8u * SV_WAVE;

constexpr unsigned STAGE = 54272u;
constexpr unsigned LDS_CTL = 3u * STAGE;
constexpr unsigned LDS_BYTES = LDS_CTL + 64u;

constexpr unsigned long long DEADLINE_TICKS = 100000000ull;

struct Params {
    const char* x;
    const char* w;
    char* out;
    char* me;
    char* wsg[NR];
    int rank;
    unsigned qmap;
};

__device__ __forceinline__ unsigned rfl(unsigned v) { return (unsigned)__builtin_amdgcn_readfirstlane((int)v); }
__device__ __forceinline__ unsigned rdl(unsigned v, unsigned l) { return (unsigned)__builtin_amdgcn_readlane((int)v, (int)l); }

__device__ __forceinline__ u32x4 mk_rsrc(const void* p, unsigned bytes) {
    const unsigned long long a = (unsigned long long)p;
    u32x4 r; r.x = (unsigned)a; r.y = (unsigned)(a >> 32) & 0xffffu; r.z = bytes; r.w = 0x00020000u; return r;
}
__device__ __forceinline__ __amdgpu_buffer_rsrc_t mk_brsrc(const void* p, unsigned bytes) {
    return __builtin_amdgcn_make_buffer_rsrc((void*)p, 0, (int)bytes, 0x00020000);
}

template <int OFF, bool SYS> __device__ __forceinline__ void dma16(u32x4 rs, unsigned voff, unsigned soff, unsigned ldsaddr) {
    if constexpr (SYS)
        asm volatile("s_mov_b32 m0, %0\n\ts_nop 0\n\tbuffer_load_dwordx4 %1, %2, %3 offen offset:%4 sc0 sc1 lds"
                     :: "s"(ldsaddr), "v"(voff), "s"(rs), "s"(soff), "i"(OFF) : "memory");
    else
        asm volatile("s_mov_b32 m0, %0\n\ts_nop 0\n\tbuffer_load_dwordx4 %1, %2, %3 offen offset:%4 " DMA_SC " lds"
                     :: "s"(ldsaddr), "v"(voff), "s"(rs), "s"(soff), "i"(OFF) : "memory");
}
template <int AUX> __device__ __forceinline__ void st128(u32x4 d, __amdgpu_buffer_rsrc_t rs, unsigned voff, unsigned soff) {
    __builtin_amdgcn_raw_buffer_store_b128(d, rs, voff, soff, AUX);
    asm volatile("s_nop 1" :: "v"(d) : "memory");
}
#define WAIT_V(n) asm volatile("s_waitcnt vmcnt(%0)" :: "i"(n) : "memory")
#define BAR() do { asm volatile("" ::: "memory"); __builtin_amdgcn_s_barrier(); asm volatile("" ::: "memory"); } while (0)

__device__ __forceinline__ unsigned ld_sys(const char* p) {
    return __hip_atomic_load((const GAS unsigned*)p, __ATOMIC_RELAXED, __HIP_MEMORY_SCOPE_SYSTEM);
}
__device__ __forceinline__ void st_sys(char* p, unsigned v) {
    __hip_atomic_store((GAS unsigned*)p, v, __ATOMIC_RELAXED, __HIP_MEMORY_SCOPE_SYSTEM);
}
__device__ __forceinline__ unsigned ld_agent(const char* p) {
    return __hip_atomic_load((const GAS unsigned*)p, __ATOMIC_RELAXED, __HIP_MEMORY_SCOPE_AGENT);
}
__device__ __forceinline__ void st_agent(char* p, unsigned v) {
    __hip_atomic_store((GAS unsigned*)p, v, __ATOMIC_RELAXED, __HIP_MEMORY_SCOPE_AGENT);
}

__device__ __forceinline__ unsigned pk_bf16(float lo, float hi) {
    f32x2 v = {lo, hi};
    return __builtin_bit_cast(unsigned, __builtin_convertvector(v, bf16x2));
}
__device__ __forceinline__ float silu(float y) {
    const float c = __builtin_amdgcn_fmed3f(y, -60.0f, 60.0f);
    const float e = __builtin_amdgcn_exp2f(c * -1.44269504088896341f);
    return y * __builtin_amdgcn_rcpf(1.0f + e);
}

__device__ __forceinline__ unsigned lane_x(unsigned lane) { return (lane >> 3) * XROW + (((lane & 7u) ^ (lane >> 4)) * 16u); }
__device__ __forceinline__ unsigned lane_w(unsigned lane) { return (lane >> 2) * WROW + (((lane & 3u) ^ ((lane >> 5) << 1)) * 16u); }

#define STAMP(i) do { } while (0)

__device__ __forceinline__ void stage_head(const Params& P, unsigned V) {
    const unsigned lane = threadIdx.x & 63u, vx = lane_x(lane);
    const unsigned done = ld_agent(P.me + OFF_CTR + blockIdx.x * 128u);
    const __amdgpu_buffer_rsrc_t rs = mk_brsrc(P.x, X_BYTES), rd = mk_brsrc(P.me, OFF_YL);
    u32x4 r[4];
#pragma unroll
    for (int i = 0; i < 4; ++i) r[i] = __builtin_amdgcn_raw_buffer_load_b128(rs, (i & 1) ? (vx ^ 64u) : vx, (V * 4u + i) * (8u * XROW), 0);
    asm volatile("" ::: "memory");
    const unsigned dofs = OFF_SX + ((rfl(done) + 1u) & 1u) * X_BYTES + V * 4096u;
#pragma unroll
    for (int i = 0; i < 4; ++i) st128<17>(r[i], rd, lane * 16u, dofs + i * 1024u);
}
__device__ __forceinline__ void push_head(const Params& P, unsigned g, unsigned q, unsigned W) {
    const unsigned lane = threadIdx.x & 63u;
    const unsigned done = ld_agent(P.me + OFF_CTR + blockIdx.x * 128u);
    const __amdgpu_buffer_rsrc_t rs = mk_brsrc(P.w, W_BYTES);
    const unsigned so = (W & 1u) * 32u * WROW + q * 4096u + (W >> 1) * 128u;
    u32x4 r[4];
#pragma unroll
    for (int i = 0; i < 4; ++i) r[i] = __builtin_amdgcn_raw_buffer_load_b128(rs, lane_w(lane), so + (i & 1) * 64u + (i >> 1) * (16u * WROW), 0);
    asm volatile("" ::: "memory");
    const __amdgpu_buffer_rsrc_t rd = mk_brsrc(P.wsg[g], OFF_SX);
    const unsigned dofs = OFF_WS + (unsigned)P.rank * WS_SRC + ((rfl(done) + 1u) & 1u) * (2u << 20) + (W >> 1) * 8192u + (W & 1u) * 2048u;
#pragma unroll
    for (int i = 0; i < 4; ++i) { if (i >= 2) WAIT_V(1); st128<17>(r[i], rd, lane * 16u, dofs + (i & 1) * 4096u + (i >> 1) * 1024u); }
}

__device__ __forceinline__ void stager(const Params& P, unsigned idx, unsigned call, unsigned dbg_role) {
    const unsigned lane = threadIdx.x & 63u, w = rfl(threadIdx.x >> 6), V = idx * 8u + w, par = call & 1u;
    const unsigned vx = lane_x(lane), vw = lane_w(lane);
    const unsigned long long pw = (unsigned long long)P.w, px = (unsigned long long)P.x;
    const __amdgpu_buffer_rsrc_t rd = mk_brsrc(P.me, OFF_YL);
    char* const flag = P.wsg[lane & 7u] + OFF_STG + (unsigned)P.rank * 128u + idx * 4u;
    for (unsigned s = 1u;; ++s) {
        WAIT_V(0);
        BAR();
        if (threadIdx.x < 8u) st_sys(flag, call * 16u + s);
        STAMP(s - 1u);
        if (s == 8u) break;
        u32x4 r[4][4];
#pragma unroll
        for (int k = 0; k < 4; ++k) {
            const unsigned bi = V + k * (NSTAGE * 8u);
            const bool x = bi >= 512u;
            const unsigned so = x ? (bi - 512u) * (32u * XROW) + s * 128u : (s * 64u + (bi & 1u) * 32u) * WROW + (bi >> 1) * 128u;
            const __amdgpu_buffer_rsrc_t rs = mk_brsrc((const void*)(x ? px : pw), x ? X_BYTES : W_BYTES);
            const unsigned v0 = x ? vx : vw, v1 = x ? (vx ^ 64u) : vw, o1 = x ? 8u * XROW : 64u, o2 = x ? 16u * XROW : 16u * WROW;
#pragma unroll
            for (int i = 0; i < 4; ++i) r[k][i] = __builtin_amdgcn_raw_buffer_load_b128(rs, (i & 1) ? v1 : v0, so + (i & 1) * o1 + (i >> 1) * o2, 0);
        }
#pragma unroll
        for (int k = 0; k < 4; ++k) {
            const unsigned bi = V + k * (NSTAGE * 8u);
            const bool x = bi >= 512u;
            const unsigned dofs = x ? OFF_SX + par * X_BYTES + s * (512u << 10) + (bi - 512u) * 4096u
                                    : OFF_SW + par * W_BYTES + s * (2u << 20) + (bi >> 1) * 8192u + (bi & 1u) * 2048u;
            const unsigned o1 = x ? 1024u : 4096u, o2 = x ? 2048u : 1024u;
#pragma unroll
            for (int i = 0; i < 4; ++i) st128<17>(r[k][i], rd, lane * 16u, dofs + (i & 1) * o1 + (i >> 1) * o2);
        }
    }
}

struct Batch { unsigned piece, so, dofs; };
__device__ __forceinline__ void mover(const Params& P, unsigned g, unsigned q, unsigned j, unsigned call, LAS char* lds, unsigned dbg_role) {
    const unsigned lane = threadIdx.x & 63u, w = rfl(threadIdx.x >> 6);
    const unsigned rank = (unsigned)P.rank;
    char* const me = P.me;
    char* const peer = P.wsg[g];
    volatile LAS unsigned* const lw = (volatile LAS unsigned*)lds;
    const unsigned ps = (lane >> 2) & 7u, pt = lane & 3u;
    const unsigned size = (lane >= 32u || lane == 1u) ? 0u : (pt == 1u ? 64u : (pt == 3u ? 16u : 48u));
    const unsigned x0 = 112u * ps + 48u * min(ps, 8u - LAG) + 16u * min(ps, 8u - FILL);
    const unsigned start = pt < 2u ? x0 - 64u * pt
                         : (pt == 2u ? (ps < 8u - LAG ? x0 + 48u : NPULL - 16u * FILL - 48u * (8u - ps))
                                     : (ps < 8u - FILL ? x0 + (ps < 8u - LAG ? 96u : 48u) : NPULL - 16u * (8u - ps)));

    if (w == 0u) {
        GAS unsigned* const cnt = (GAS unsigned*)(me + OFF_FLG + q * 256u) + lane;
        const u32x4 rf = mk_rsrc(me + OFF_STG + q * 128u, NSTAGE * 4u);
        const unsigned long long t0 = __builtin_amdgcn_s_memrealtime();
        unsigned seen = 0u, rounds = 0u;
        for (unsigned n = 0u;; ++n) {
            const unsigned gone = rfl(lw[63]), v = lw[lane], d = v - seen;
            if (lane < 32u && d != 0u && (rdl(v, 60u) >= start + size || gone != 0u)) {
                __hip_atomic_fetch_add(cnt, d, __ATOMIC_RELAXED, __HIP_MEMORY_SCOPE_SYSTEM);
                seen = v;
            }
            if (gone == 7u || ((n & 4095u) == 4095u && __builtin_amdgcn_s_memrealtime() - t0 > 2ull * DEADLINE_TICKS - 1000000ull)) break;
            if (rounds < 8u) {
                const unsigned sv = lw[64u + lane];
                while (rounds < 8u && __ballot(lane < NSTAGE && (int)(sv - (call * 16u + rounds + 1u)) < 0) == 0ull) ++rounds;
                if (lane == 0u) lw[62] = rounds;
                if ((n & (rounds < 2u ? 1u : 7u)) == 0u)
                    asm volatile("s_mov_b32 m0, %0\n\ts_nop 0\n\tbuffer_load_dword %1, %2, 0 offen sc0 sc1 lds\n\ts_waitcnt vmcnt(24)"
                                 :: "s"((unsigned)(uintptr_t)lds + 256u), "v"(lane * 4u), "s"(rf) : "memory");
            }
            __builtin_amdgcn_s_sleep(1);
        }
        return;
    }

    const unsigned par = call & 1u, vd = lane * 16u;
    const unsigned lo = min(rank, q), hi = max(rank, q);
    auto batch_of = [&](unsigned U) {
        Batch b;
        b.piece = (unsigned)__builtin_ctz((unsigned)__ballot(U - start < size) | 0x80000000u);
        const unsigned r = U - rdl(start, b.piece), s = b.piece >> 2, t = b.piece & 3u;
        if (t == 1u) {
            b.so = OFF_SW + par * W_BYTES + s * (2u << 20) + rank * (256u << 10) + r * 4096u;
            b.dofs = OFF_WS + q * WS_SRC + s * (256u << 10) + r * 4096u;
        } else {
            unsigned mb = r >> 3; mb += mb >= lo ? 1u : 0u; mb += mb >= hi ? 1u : 0u;
            const unsigned o = s * (512u << 10) + (t == 3u ? rank * 64u + r * 4u : mb * 64u + t * 16u + (r & 7u) * 4u) * 1024u;
            b.so = OFF_SX + par * X_BYTES + o;
            b.dofs = OFF_XS + q * XS_SRC + o;
        }
        b.piece = rfl(b.piece); b.so = rfl(b.so); b.dofs = rfl(b.dofs);
        asm("" : "+s"(b.piece), "+s"(b.so), "+s"(b.dofs));
        return b;
    };
    const __amdgpu_buffer_rsrc_t rd = mk_brsrc(me, OFF_SX);
    const __amdgpu_buffer_rsrc_t rc = mk_brsrc(me + OFF_CNT + q * 256u, 256u);
    const unsigned vc = lane == 0u ? 0u : 4096u;
    const unsigned tbase = (call - 1u) * NPULL - NMOVER;
    static_assert(GO_AT < (int)NMOVER && LAG <= 7 && FILL <= 8, "");
    auto bump = [&](unsigned word) { if (lane == 0u) __hip_atomic_fetch_add((LAS unsigned*)lds + word, 1u, __ATOMIC_RELAXED, __HIP_MEMORY_SCOPE_WORKGROUP); };
    unsigned known = 0u;
    auto staged = [&](unsigned round) {
        if (round < known) return true;
        const unsigned long long t0 = __builtin_amdgcn_s_memrealtime();
        for (unsigned n = 1u; (known = rfl(lw[62])) <= round; ++n) {
            if ((n & 1023u) == 0u && (rfl(ld_agent(me + OFF_TMO)) == call || __builtin_amdgcn_s_memrealtime() - t0 > DEADLINE_TICKS)) {
                if (lane == 0u) st_agent(me + OFF_TMO, call);
                return false;
            }
            __builtin_amdgcn_s_sleep(1);
        }
        return true;
    };
    auto pull = [&](unsigned so, bool valid) { return __builtin_amdgcn_raw_buffer_load_b128(mk_brsrc(peer, valid ? OFF_YL : 0u), vd, so, 17); };
    STAMP(0);
    Batch b0 = batch_of(j * 7u + w - 1u);
    unsigned pprev = 61u, tprev = 0u, t0 = 0u;
    if (staged(b0.piece >> 2)) {
        STAMP(6);
        u32x4 r0 = pull(b0.so, true);
        unsigned tk = (unsigned)__builtin_amdgcn_raw_ptr_buffer_atomic_add_i32(1, rc, vc, 0, 0);
        u32x4 r1 = pull(b0.so + 1024u, true);
        for (;;) {
            st128<16>(r0, rd, vd, b0.dofs);          r0 = pull(b0.so + 2048u, true); asm volatile("" ::: "memory");
            st128<16>(r1, rd, vd, b0.dofs + 1024u);  r1 = pull(b0.so + 3072u, true); asm volatile("" ::: "memory");
            const unsigned T = rfl(tk) - tbase;
            if (T == NPULL + GO_AT && lane == 0u) st_agent(me + OFF_GO + q * 128u, call);
            bool v1 = T < NPULL;
            const Batch b1 = batch_of(v1 ? T : 0u);
            WAIT_V(2);
            bump(pprev);
            if (lane == 0u) __hip_atomic_fetch_max((LAS unsigned*)lds + 60, tprev, __ATOMIC_RELAXED, __HIP_MEMORY_SCOPE_WORKGROUP);
            pprev = b0.piece; tprev = t0; t0 = T;
            v1 = v1 && staged(b1.piece >> 2);
            st128<16>(r0, rd, vd, b0.dofs + 2048u);  r0 = pull(b1.so, v1); asm volatile("" ::: "memory");
            tk = (unsigned)__builtin_amdgcn_raw_ptr_buffer_atomic_add_i32(1, rc, v1 ? vc : 4096u, 0, 0);
            st128<16>(r1, rd, vd, b0.dofs + 3072u);  r1 = pull(b1.so + 1024u, v1); asm volatile("" ::: "memory");
            if (!v1) break;
            b0 = b1;
        }
    }
    WAIT_V(0);
    bump(pprev);
    bump(63u);
    STAMP(1);
}

__device__ __forceinline__ void gemm_tile(const Params& P, unsigned grp, unsigned q, unsigned tile, unsigned call, LAS char* lds, unsigned dbg_role) {
    const unsigned lane = threadIdx.x & 63u, g16 = lane >> 4;
    const unsigned w = rfl(threadIdx.x >> 6), wm = w >> 1, wn = w & 1u;
    const unsigned rank = (unsigned)P.rank, g = (q - rank - 1u) & 7u;
    const unsigned h = tile & 1u, c = tile >> 1;
    const bool own = g == 7u;
    char* const me = P.me;
    char* const peer = P.wsg[grp];
    const unsigned par = call & 1u;
    const unsigned lds0 = (unsigned)(uintptr_t)lds;
    LAS unsigned* ctl = (LAS unsigned*)(lds + LDS_CTL);
    STAMP(0);
    if (!own && w != 0u && tile * 7u + w - 1u < 64u) {
        WAIT_V(0);
        if (lane == 0u) __hip_atomic_fetch_add((GAS unsigned*)(peer + OFF_FLG + rank * 256u) + 1, 1u, __ATOMIC_RELAXED, __HIP_MEMORY_SCOPE_SYSTEM);
    }

    const unsigned nraw = own ? 8u : 16u, nsrc = own ? 7u : 6u;
    unsigned tabA, tabB, vF;
    unsigned long long gend;
    {
        const bool extra = !own && lane < 8u, raw = lane < nraw;
        const unsigned e = lane - nraw, s = raw ? lane & 7u : (own ? e / 7u : e / 6u), i = e - s * nsrc;
        const unsigned p = (rank + 1u + i + (i >= g ? 1u : 0u)) & 7u;
        tabA = raw ? ((extra ? rank : q) * 512u + h * 256u) * XROW + s * 128u : p * XS_SRC + s * (512u << 10) + (q * 2u + h) * 32768u;
        tabB = raw ? s * 64u * WROW + (extra ? q : rank) * 4096u + c * 320u : p * WS_SRC + (s ? s : 8u * par) * (256u << 10) + c * 20480u;
        vF = raw ? rank * 256u : p * 256u + s * 16u;
        gend = __ballot(raw ? lane == nraw - 1u : i == nsrc - 1u);
    }
    const unsigned b0 = w < 4u ? 3u * w : 12u + 2u * (w - 4u);
    const unsigned cbmax = c == 12u ? 3u : 4u;
    u32x4 ra, rb, rx;
    unsigned za, zb, zx1, vP[7];
    const unsigned zx0 = w == 7u ? 2048u : 0u, mx = w < 4u ? ~0u : 0u;
    auto staging = [&](bool raw) {
        ra = raw ? mk_rsrc(P.x, X_BYTES) : mk_rsrc(me + OFF_XS, 8u * XS_SRC);
        rb = raw ? mk_rsrc(P.w, W_BYTES) : mk_rsrc(me + OFF_WS, 8u * WS_SRC);
        rx = w < 4u ? rb : mk_rsrc(me + OFF_FLG, zx0);
        za = ra.z; zb = rb.z; zx1 = rx.z;
#pragma unroll
        for (int i = 0; i < 4; ++i) vP[i] = (raw ? (lane_x(lane) ^ ((i & 1) * 64u)) + (4u * w + i) * (8u * XROW) : lane * 16u + (4u * w + i) * 1024u) - i * 1024u;
#pragma unroll
        for (int i = 0; i < 3; ++i) { const unsigned b = b0 + i, cb = min(b >> 2, cbmax);
            vP[4 + i] = raw ? lane_w(lane) + cb * 64u + (b & 3u) * (16u * WROW) : lane * 16u + cb * 4096u + (b & 3u) * 1024u; }
        if (w >= 4u) vP[6] = vF;
    };
    staging(true);
    const unsigned la0 = lds0 + w * 4096u, lb0 = lds0 + 32768u + b0 * 1024u;
    const unsigned lx0 = w < 4u ? lb0 + 2048u : (w == 7u ? lds0 + 53248u : lds0 + 0x30000u), kx = (w < 4u || w == 7u) ? ~0u : 0u;
    unsigned oa = 0u, ob = 0u;
    auto piece = [&](int i, unsigned so) {
        if (i == 0) dma16<0, false>(ra, vP[0], oa, la0 + so);
        if (i == 1) dma16<1024, false>(ra, vP[1], oa, la0 + so);
        if (i == 2) dma16<2048, false>(ra, vP[2], oa, la0 + so);
        if (i == 3) dma16<3072, false>(ra, vP[3], oa, la0 + so);
        if (i == 4) dma16<0, false>(rb, vP[4], ob, lb0 + so);
        if (i == 5) dma16<0, false>(rb, vP[5], ob, lb0 + so + 1024u);
        if (i == 6) dma16<0, true>(rx, vP[6], ob & mx, lx0 + (so & kx));
    };
    unsigned it = 0u, ready = nraw;
    unsigned gm = 0u;
    auto decide = [&]() { asm volatile("s_cmp_lt_u32 %1, %2\n\ts_cselect_b32 %0, -1, 0" : "=s"(gm) : "s"(it), "s"(ready) : "scc"); };
    auto planA = [&]() { oa = rdl(tabA, it & 63u); ra.z = za & gm; asm volatile("" : "+s"(oa), "+s"(ra)); };
    auto planB = [&]() { ob = rdl(tabB, it & 63u); rb.z = zb & gm; asm volatile("" : "+s"(ob), "+s"(rb)); };
    auto planX = [&]() { rx.z = zx1 & gm & (it + 3u == ready ? ~0u : mx); it -= gm; asm volatile("" : "+s"(it), "+s"(rx)); };
    auto plan = [&]() { decide(); planA(); planB(); planX(); asm volatile("s_nop 4"); };
    const unsigned xw = own ? 12u : 8u * h, needx = (own ? 16u : 48u) * call, needw = 64u * call;
    auto arrived = [&](unsigned vw, unsigned vx) {
        const unsigned long long miss = __ballot((int)(vw - needw) < 0 || (int)(vx - needx) < 0) >> nraw << nraw;
        return 64u - (unsigned)__builtin_clzll(gend & ((miss & (0ull - miss)) - 1ull));
    };

    const unsigned aoff = wm * 8192u + (lane & 15u) * 128u + ((g16 ^ ((lane & 15u) >> 1)) * 16u);
    const unsigned lp = (g16 >> 1) * 1024u + (8u * (g16 & 1u) + ((lane & 15u) >> 2)) * 64u + (lane & 3u) * 8u;
    const unsigned be = 32768u + ((5u * wn) >> 1) * 4096u + lp + (((wn ^ g16) & 1u) * 32u);
    const unsigned bo = 32768u + ((5u * wn + 1u) >> 1) * 4096u + lp + (((wn ^ g16 ^ 1u) & 1u) * 32u);

    f32x4 acc[4][5];
#pragma unroll
    for (int m = 0; m < 4; ++m)
#pragma unroll
        for (int n = 0; n < 5; ++n) acc[m][n] = (f32x4){0.f, 0.f, 0.f, 0.f};
    const __amdgpu_buffer_rsrc_t rv = mk_brsrc(me + OFF_SV + ((min(g, 6u) * NTILE + tile) * 8u + w) * SV_WAVE, SV_WAVE);
    s16x8 fa[2][4]; s16x4 fb[2][5][2];
    auto zero1 = [&]() {
#pragma unroll
        for (int m = 0; m < 4; ++m) fa[1][m] = (s16x8){0, 0, 0, 0, 0, 0, 0, 0};
#pragma unroll
        for (int n = 0; n < 5; ++n) { fb[1][n][0] = (s16x4){0, 0, 0, 0}; fb[1][n][1] = (s16x4){0, 0, 0, 0}; }
    };
    auto rd = [&](int kh, int r, LAS char* st) {
        if (r >= 2 && r < 6) fa[kh][r - 2] = *(const LAS s16x8*)(st + (kh ? (aoff ^ 64u) : aoff) + (r - 2) * 2048);
        else { const int n = r < 2 ? 0 : (r - 4) >> 1, hi = r & 1;
               fb[kh][n][hi] = __builtin_amdgcn_ds_read_tr16_b64_v4i16((LAS s16x4*)(st + ((n & 1) ? bo : be) + (n >> 1) * 4096 + kh * 2048 + hi * 256)); }
    };
    auto mm = [&](int kh, int i) {
        const int n = i >> 2, m = i & 3;
        acc[m][n] = __builtin_amdgcn_mfma_f32_16x16x32_bf16(
            __builtin_bit_cast(bf16x8, __builtin_shufflevector(fb[kh][n][0], fb[kh][n][1], 0, 1, 2, 3, 4, 5, 6, 7)),
            __builtin_bit_cast(bf16x8, fa[kh][m]), acc[m][n], 0, 0, 0);
    };
    zero1();

    unsigned t = 0u;
    unsigned s0 = 0u, s1 = STAGE, s2 = 2u * STAGE;
    bool bad = false;
    for (;;) {
        if (it == t) {
            plan();
#pragma unroll
            for (int i = 0; i < 7; ++i) piece(i, s0);
            plan();
#pragma unroll
            for (int i = 0; i < 7; ++i) piece(i, s1);
            plan();
        }
        const unsigned park = (!own && t < 8u) ? 8u : 64u;
        do {
            LAS char* st = lds + s0;
#pragma unroll
            for (int i = 0; i < 20; ++i) {
                if (i == 2) {
                    __builtin_amdgcn_s_waitcnt(0xC07F);
                    WAIT_V(7);
                    BAR();
                    __builtin_amdgcn_sched_barrier(0);
                    __builtin_amdgcn_s_setprio(3);
                }
                if (i == 12) __builtin_amdgcn_s_setprio(2);
                mm(1, i);
                if (i >= 2 && i < 16) rd(0, i - 2, st);
                if (i % 5 == 4) piece(i / 5, s2);
                if (i == 17 && __builtin_expect(it == ready && it == t + 3u && t >= nraw, 0)) {
                    const LAS char* fr = st + 53248u + lane * 16u;
                    ready = max(ready, arrived(*(const LAS unsigned*)(fr + 4), *(const LAS unsigned*)(fr + xw)));
                }
                if (i == 18) decide();
                __builtin_amdgcn_sched_barrier(0);
            }
#pragma unroll
            for (int i = 0; i < 20; ++i) {
                if (i == 2) __builtin_amdgcn_s_setprio(1);
                if (i == 12) __builtin_amdgcn_s_setprio(0);
                mm(0, i);
                if (i < 14) rd(1, i, st);
                if (i % 5 == 4 && i < 15) piece(4 + i / 5, s2);
                if (i == 17) planA();
                if (i == 18) planB();
                if (i == 19) planX();
                __builtin_amdgcn_sched_barrier(0);
            }
            { const unsigned x = s0; s0 = s1; s1 = s2; s2 = x; }
            ++t;
        } while (t < min(it, park));
        if (t == 64u) break;
        if (t < it) {
#pragma unroll
            for (int i = 0; i < 20; ++i) mm(1, i);
#pragma unroll
            for (int m = 0; m < 4; ++m)
#pragma unroll
                for (int n = 0; n < 5; ++n) { st128<0>(__builtin_bit_cast(u32x4, acc[m][n]), rv, lane * 16u, (n * 4 + m) * 1024); acc[m][n] = (f32x4){0.f, 0.f, 0.f, 0.f}; }
            zero1();
            continue;
        }
        STAMP(1 + 2 * (seg < 8u ? seg : 8u));
        if (t == nraw) staging(false);
        if (w == 0u) {
            const char* fp = me + OFF_FLG + vF;
            const unsigned long long t0 = __builtin_amdgcn_s_memrealtime();
            unsigned res;
            for (unsigned n = 1u;; ++n) {
                const unsigned vw = ld_sys(fp + 4), vx = ld_sys(fp + xw);
                res = arrived(vw, vx);
                if (res > t) break;
                if ((n & 15u) == 0u && (rfl(ld_agent(me + OFF_TMO)) == call || __builtin_amdgcn_s_memrealtime() - t0 > DEADLINE_TICKS)) {
                    st_agent(me + OFF_TMO, call);
                    res = 0x100u + 64u; break;
                }
                __builtin_amdgcn_s_sleep(4);
            }
            ctl[0] = res;
            asm volatile("s_waitcnt vmcnt(0) lgkmcnt(0)" ::: "memory");
        }
        BAR();
        const unsigned res = rfl(*(volatile LAS unsigned*)ctl);
        bad |= res > 0xffu;
        ready = res & 0xffu;
        STAMP(2 + 2 * (seg < 8u ? seg : 8u));
    }
#pragma unroll
    for (int i = 0; i < 20; ++i) mm(1, i);
    STAMP(20);

    const unsigned nu = (c == 12u && wn == 1u) ? 6u : 10u;
    const __amdgpu_buffer_rsrc_t ro = mk_brsrc(P.out, OUT_BYTES);
    const unsigned vo = ((g16 & 1u) * 16u + (lane & 15u)) * WROW + (g16 >> 1) * 16u;
    const unsigned vo2 = nu == 10u ? vo : 0x80000000u;
    unsigned so0 = (h * 256u + wm * 64u) * WROW + q * 4096u + c * 320u + wn * 160u, so1 = so0 + 32u * WROW;
    asm("" : "+s"(so0), "+s"(so1));
    auto emit = [&](int u, const f32x4& v0, const f32x4& v1) {
        const unsigned a0 = pk_bf16(silu(v0[0]), silu(v0[1])), a1 = pk_bf16(silu(v0[2]), silu(v0[3]));
        const unsigned c0 = pk_bf16(silu(v1[0]), silu(v1[1])), c1 = pk_bf16(silu(v1[2]), silu(v1[3]));
        const auto x0 = __builtin_amdgcn_permlane16_swap(a0, c0, false, false);
        const auto x1 = __builtin_amdgcn_permlane16_swap(a1, c1, false, false);
        u32x4 d = {x0[0], x1[0], x0[1], x1[1]};
        if (bad) d = (u32x4){0x7fc07fc0u, 0x7fc07fc0u, 0x7fc07fc0u, 0x7fc07fc0u};
        st128<16>(d, ro, (u < 6 ? vo : vo2) + (u >> 1) * 32u, (u & 1) ? so1 : so0);
    };
    if (own) {
        bad |= rfl(ld_agent(me + OFF_TMO)) == call;
#pragma unroll
        for (int u = 0; u < 10; ++u) emit(u, acc[2 * (u & 1)][u >> 1], acc[2 * (u & 1) + 1][u >> 1]);
        STAMP(23);
        return;
    }
    const unsigned yo = OFF_YL + par * Y_PAR + (tile * 8u + w) * Y_WAVE;
    {
        const __amdgpu_buffer_rsrc_t ry = mk_brsrc(me + yo + q * Y_PEER, Y_WAVE);
#pragma unroll
        for (int u = 0; u < 10; ++u) {
            const f32x4 v0 = acc[2 * (u & 1)][u >> 1], v1 = acc[2 * (u & 1) + 1][u >> 1];
            u32x4 d = {pk_bf16(v0[0], v0[1]), pk_bf16(v0[2], v0[3]), pk_bf16(v1[0], v1[1]), pk_bf16(v1[2], v1[3])};
            if (bad) d = (u32x4){0x7fc07fc0u, 0x7fc07fc0u, 0x7fc07fc0u, 0x7fc07fc0u};
            st128<17>(d, ry, lane * 16u, u * 1024);
        }
        WAIT_V(0);
        BAR();
        if (threadIdx.x == 0u) st_sys(me + OFF_YF + q * 4096u + tile * 128u, call);
    }
    STAMP(21);
    if (w == 0u) {
        const char* fp = lane == 0u ? peer + OFF_YF + rank * 4096u + tile * 128u : (lane == 1u ? me + OFF_GO + q * 128u : me + OFF_TMO);
        const unsigned long long t0 = __builtin_amdgcn_s_memrealtime();
        bool have = false;
        for (;;) {
            const unsigned v = ld_sys(fp);
            if (rdl(v, 2u) == call) { bad = true; break; }
            const unsigned m = (unsigned)__ballot((int)(v - call) >= 0);
            if (!have && (m & 1u)) { have = true; if (lane == 0u) fp = me + OFF_TMO; }
            if (have && (m & 2u)) break;
            if (__builtin_amdgcn_s_memrealtime() - t0 > DEADLINE_TICKS) {
                if (lane == 0u) st_agent(me + OFF_TMO, call);
                bad = true; break;
            }
            __builtin_amdgcn_s_sleep(2);
        }
        if (lane == 0u) ctl[0] = bad ? 1u : 0u;
        asm volatile("s_waitcnt lgkmcnt(0)" ::: "memory");
    }
    BAR();
    bad |= rfl(*(volatile LAS unsigned*)ctl) != 0u;
    STAMP(22);
    const __amdgpu_buffer_rsrc_t rp = mk_brsrc(peer + yo + rank * Y_PEER, bad ? 0u : nu * 1024u);
    u32x4 pv[10];
    auto fetch = [&](int u) {
        unsigned n = 0u;
        if (lane == 0u) n = __hip_atomic_fetch_add(ctl + 4, 1u, __ATOMIC_RELAXED, __HIP_MEMORY_SCOPE_WORKGROUP);
        n = rfl(n);
        for (unsigned k = 0u; n >= rfl(*(volatile LAS unsigned*)(ctl + 5)) + TOK && k < (1u << 20); ++k) __builtin_amdgcn_s_sleep(1);
        pv[u] = __builtin_amdgcn_raw_buffer_load_b128(rp, lane * 16u + u * 1024u, 0, 17);
        asm volatile("" ::: "memory");
#pragma unroll
        for (int k = 0; k < 2; ++k) acc[2 * (u & 1) + k][u >> 1] = __builtin_bit_cast(f32x4, __builtin_amdgcn_raw_buffer_load_b128(rv, lane * 16u, ((u >> 1) * 4 + 2 * (u & 1) + k) * 1024, 0));
        asm volatile("" ::: "memory");
    };
    fetch(0);
#pragma unroll
    for (int u = 0; u < 10; ++u) {
        asm volatile("" : "+v"(pv[u]));
        if (lane == 0u) __hip_atomic_fetch_add(ctl + 5, 1u, __ATOMIC_RELAXED, __HIP_MEMORY_SCOPE_WORKGROUP);
        if (u < 9) fetch(u + 1);
        __builtin_amdgcn_sched_barrier(0);
        const u32x4 d = pv[u];
        f32x4 v0 = acc[2 * (u & 1)][u >> 1], v1 = acc[2 * (u & 1) + 1][u >> 1];
        v0[0] += __builtin_bit_cast(float, d.x << 16); v0[1] += __builtin_bit_cast(float, d.x & 0xffff0000u);
        v0[2] += __builtin_bit_cast(float, d.y << 16); v0[3] += __builtin_bit_cast(float, d.y & 0xffff0000u);
        v1[0] += __builtin_bit_cast(float, d.z << 16); v1[1] += __builtin_bit_cast(float, d.z & 0xffff0000u);
        v1[2] += __builtin_bit_cast(float, d.w << 16); v1[3] += __builtin_bit_cast(float, d.w & 0xffff0000u);
        emit(u, v0, v1);
        __builtin_amdgcn_sched_barrier(0);
    }
    STAMP(23);
}

__global__ void __launch_bounds__(512, 2) fused_gemm_rs(Params P) {
    __shared__ __attribute__((aligned(16))) char smem[LDS_BYTES];
    LAS char* lds = (LAS char*)smem;
    volatile LAS unsigned* const lw = (volatile LAS unsigned*)lds;
    const unsigned w = rfl(threadIdx.x >> 6), rank = (unsigned)P.rank;
    const unsigned guess = (blockIdx.x & ~7u) | ((unsigned)__builtin_amdgcn_s_getreg((3 << 11) | 20) & GUESS_MASK);
    auto head = [&](unsigned role) {
        const unsigned slot = role >> 3, q = (P.qmap >> ((role & 7u) * 4u)) & 7u;
        if (w == 0u || slot >= 32u) return;
        if (slot >= NSW + NPW) { const unsigned W = (slot - NSW - NPW) * 7u + w - 1u; if (W < 64u && q != rank) push_head(P, role & 7u, q, W); }
        else if (slot < NSW || q == rank) { const unsigned V = (slot < NSW ? role : 8u * NSW + slot - NSW) * 7u + w - 1u; if (V < 128u) stage_head(P, V); }
    };
    if (w == 0u) {
        char* const me = P.me;
        const __amdgpu_buffer_rsrc_t rc = mk_brsrc(me + OFF_CLM, 32768u);
        const unsigned v0 = threadIdx.x == 0u ? 0u : 65536u;
        unsigned role = guess, old = (unsigned)__builtin_amdgcn_raw_ptr_buffer_atomic_add_i32(1, rc, v0, role * 128u, 0);
        const unsigned done = rfl(ld_agent(me + OFF_CTR + blockIdx.x * 128u));
        lw[threadIdx.x] = 0u; lw[64u + threadIdx.x] = 0u;
        while (rfl(old) != done) {
            __builtin_amdgcn_raw_ptr_buffer_atomic_add_i32(-1, rc, v0, role * 128u, 0);
            role = (role + 1u) & 255u;
            if (role == guess) { role = 256u; st_agent(me + OFF_TMO, done + 1u); break; }
            old = (unsigned)__builtin_amdgcn_raw_ptr_buffer_atomic_add_i32(1, rc, v0, role * 128u, 0);
        }
        if (threadIdx.x == 0u) { lw[LDS_CTL / 4u + 1u] = role; lw[LDS_CTL / 4u + 2u] = done + 1u; lw[LDS_CTL / 4u + 4u] = 0u; lw[LDS_CTL / 4u + 5u] = 0u; }
        __builtin_amdgcn_s_waitcnt(0x0070);
    } else head(guess);
    BAR();
    const unsigned role = rfl(lw[LDS_CTL / 4u + 1u]), call = rfl(lw[LDS_CTL / 4u + 2u]);
    if (role != guess) { head(role); BAR(); }
    if (threadIdx.x == 0u) st_agent(P.me + OFF_CTR + blockIdx.x * 128u, call);
    const unsigned q = (P.qmap >> ((role & 7u) * 4u)) & 7u, slot = role >> 3;
    if (slot >= 32u) return;
    if (slot >= NSW + NPW) gemm_tile(P, role & 7u, q, slot - NSW - NPW, call, lds, role);
    else if (slot >= NSW && q != rank) mover(P, role & 7u, q, slot - NSW, call, lds, role);
    else stager(P, slot < NSW ? role : 8u * NSW + slot - NSW, call, role);
}

unsigned make_qmap(int rank) {
    static const unsigned char kNear[NR][NR] = {
        {6, 0, 2, 3, 1, 5, 4, 7}, {7, 1, 4, 5, 0, 3, 6, 2}, {3, 2, 0, 6, 4, 5, 7, 1}, {2, 3, 6, 0, 5, 1, 7, 4},
        {5, 4, 1, 7, 2, 0, 6, 3}, {4, 5, 7, 1, 3, 2, 0, 6}, {0, 6, 3, 2, 7, 4, 1, 5}, {1, 7, 5, 4, 6, 3, 2, 0}};
    unsigned m = 0u;
    for (int g = 0; g < NR; ++g) m |= (unsigned)kNear[rank][g] << (4 * g);
    return m;
}

}

extern "C" void kernel_launch(int rank, int nranks, ncclComm_t comm,
                              hipStream_t stream,
                              void* const* d_in, const int* in_sizes, int n_in,
                              void* d_out, int out_size,
                              const int* in_dtypes, int out_dtype,
                              void* const* d_out_all,
                              void* const* d_ws_all, size_t ws_bytes) {
    (void)comm; (void)in_dtypes; (void)out_dtype; (void)d_out_all;
    if (nranks != NR || (unsigned)rank >= (unsigned)NR || n_in != 2 || ws_bytes < (size_t)WS_NEED ||
        in_sizes[0] != 4096 * 512 || in_sizes[1] != 512 * 16384 || out_size != 512 * 16384) {
        fprintf(stderr, "unexpected problem geometry\n");
        return;
    }
    Params P;
    P.x = (const char*)d_in[0];
    P.w = (const char*)d_in[1];
    P.out = (char*)d_out;
    P.me = (char*)d_ws_all[rank];
    P.rank = rank;
    P.qmap = make_qmap(rank);
    for (int g = 0; g < NR; ++g) P.wsg[g] = (char*)d_ws_all[(P.qmap >> (4 * g)) & 7u];
    fused_gemm_rs<<<GRID, 512, 0, stream>>>(P);
    HIP_CHECK(hipGetLastError());
}
```

```cpp
#include <hip/hip_runtime.h>
#include <rccl/rccl.h>
#include <cstdio>
#include <cstdint>

#define HIP_CHECK(expr) do { hipError_t _e = (expr); if (_e != hipSuccess) { \
    fprintf(stderr, "HIP error %s at %s:%d\n", hipGetErrorString(_e), __FILE__, __LINE__); \
    return; } } while (0)

#define LAS __attribute__((address_space(3)))
#define GAS __attribute__((address_space(1)))

#define LAG 4
#define FILL 3
#define GO_AT -24
#define TOK 4
#define NPUSH 28
#define HS 2
#define MW 6u
#define DMA_SC "sc1"
#define GUESS_MASK 7u

namespace {

typedef __bf16 bf16x8 __attribute__((ext_vector_type(8)));
typedef __bf16 bf16x2 __attribute__((ext_vector_type(2)));
typedef short s16x4 __attribute__((ext_vector_type(4)));
typedef short s16x8 __attribute__((ext_vector_type(8)));
typedef float f32x4 __attribute__((ext_vector_type(4)));
typedef float f32x2 __attribute__((ext_vector_type(2)));
typedef unsigned u32x4 __attribute__((ext_vector_type(4)));

constexpr int NR = 8;
constexpr unsigned XROW = 1024u;
constexpr unsigned WROW = 32768u;
constexpr unsigned X_BYTES = 4096u * XROW, W_BYTES = 512u * WROW, OUT_BYTES = 512u * WROW;
constexpr unsigned NSW = 2u, NPW = 4u, NTILE = 26u;
constexpr unsigned NMOVER = NPW * MW;
constexpr unsigned NSTAGE = 8u * NSW + NPW;
constexpr unsigned NPULL = 1408u - NPUSH, WREST = 64u - NPUSH, NHEAD = 19u;
constexpr int GRID = 256;
static_assert(NSW + NPW + NTILE == 32u && NSTAGE * 8u * 4u == 640u && NSTAGE * 8u * 2u >= 8u * WREST && NHEAD * 7u >= 128u, "");

constexpr unsigned OFF_CTR = 0u;
constexpr unsigned OFF_TMO = 32768u;
constexpr unsigned OFF_CNT = 40960u;
constexpr unsigned OFF_STG = 49152u;
constexpr unsigned OFF_GO = 53248u;
constexpr unsigned OFF_FLG = 65536u;
constexpr unsigned OFF_CLM = 98304u;
constexpr unsigned OFF_YF = 131072u;
constexpr unsigned OFF_DBG = 262144u;
constexpr unsigned OFF_XS = 1u << 20;
constexpr unsigned XS_SRC = 4u << 20;
constexpr unsigned OFF_WS = OFF_XS + 8u * XS_SRC;
constexpr unsigned WS_SRC = 9u * (256u << 10);
constexpr unsigned OFF_SX = OFF_WS + 8u * WS_SRC;
constexpr unsigned OFF_SW = OFF_SX + 2u * X_BYTES;
constexpr unsigned Y_WAVE = 10240u, Y_PEER = NTILE * 8u * Y_WAVE, Y_PAR = NR * Y_PEER;
constexpr unsigned OFF_YL = OFF_SW + 2u * W_BYTES;
constexpr unsigned OFF_SV = OFF_YL + 2u * Y_PAR;
constexpr unsigned SV_WAVE = 20480u;
constexpr unsigned WS_NEED = OFF_SV + 7u * NTILE * 8u * SV_WAVE;

constexpr unsigned STAGE = 54272u;
constexpr unsigned LDS_CTL = 3u * STAGE;
constexpr unsigned LDS_BYTES = LDS_CTL + 64u;

constexpr unsigned long long DEADLINE_TICKS = 100000000ull;

struct Params {
    const char* x;
    const char* w;
    char* out;
    char* me;
    char* wsg[NR];
    int rank;
    unsigned qmap;
};

__device__ __forceinline__ unsigned rfl(unsigned v) { return (unsigned)__builtin_amdgcn_readfirstlane((int)v); }
__device__ __forceinline__ unsigned rdl(unsigned v, unsigned l) { return (unsigned)__builtin_amdgcn_readlane((int)v, (int)l); }

__device__ __forceinline__ u32x4 mk_rsrc(const void* p, unsigned bytes) {
    const unsigned long long a = (unsigned long long)p;
    u32x4 r; r.x = (unsigned)a; r.y = (unsigned)(a >> 32) & 0xffffu; r.z = bytes; r.w = 0x00020000u; return r;
}
__device__ __forceinline__ __amdgpu_buffer_rsrc_t mk_brsrc(const void* p, unsigned bytes) {
    return __builtin_amdgcn_make_buffer_rsrc((void*)p, 0, (int)bytes, 0x00020000);
}

template <int OFF, bool SYS> __device__ __forceinline__ void dma16(u32x4 rs, unsigned voff, unsigned soff, unsigned ldsaddr) {
    if constexpr (SYS)
        asm volatile("s_mov_b32 m0, %0\n\ts_nop 0\n\tbuffer_load_dwordx4 %1, %2, %3 offen offset:%4 sc0 sc1 lds"
                     :: "s"(ldsaddr), "v"(voff), "s"(rs), "s"(soff), "i"(OFF) : "memory");
    else
        asm volatile("s_mov_b32 m0, %0\n\ts_nop 0\n\tbuffer_load_dwordx4 %1, %2, %3 offen offset:%4 " DMA_SC " lds"
                     :: "s"(ldsaddr), "v"(voff), "s"(rs), "s"(soff), "i"(OFF) : "memory");
}
template <int AUX> __device__ __forceinline__ void st128(u32x4 d, __amdgpu_buffer_rsrc_t rs, unsigned voff, unsigned soff) {
    __builtin_amdgcn_raw_buffer_store_b128(d, rs, voff, soff, AUX);
    asm volatile("s_nop 3" :: "v"(d) : "memory");
}
#define WAIT_V(n) asm volatile("s_waitcnt vmcnt(%0)" :: "i"(n) : "memory")
#define BAR() do { asm volatile("" ::: "memory"); __builtin_amdgcn_s_barrier(); asm volatile("" ::: "memory"); } while (0)

__device__ __forceinline__ unsigned ld_sys(const char* p) {
    return __hip_atomic_load((const GAS unsigned*)p, __ATOMIC_RELAXED, __HIP_MEMORY_SCOPE_SYSTEM);
}
__device__ __forceinline__ void st_sys(char* p, unsigned v) {
    __hip_atomic_store((GAS unsigned*)p, v, __ATOMIC_RELAXED, __HIP_MEMORY_SCOPE_SYSTEM);
}
__device__ __forceinline__ unsigned ld_agent(const char* p) {
    return __hip_atomic_load((const GAS unsigned*)p, __ATOMIC_RELAXED, __HIP_MEMORY_SCOPE_AGENT);
}
__device__ __forceinline__ void st_agent(char* p, unsigned v) {
    __hip_atomic_store((GAS unsigned*)p, v, __ATOMIC_RELAXED, __HIP_MEMORY_SCOPE_AGENT);
}

__device__ __forceinline__ unsigned pk_bf16(float lo, float hi) {
    f32x2 v = {lo, hi};
    return __builtin_bit_cast(unsigned, __builtin_convertvector(v, bf16x2));
}
__device__ __forceinline__ float silu(float y) {
    const float c = __builtin_amdgcn_fmed3f(y, -60.0f, 60.0f);
    const float e = __builtin_amdgcn_exp2f(c * -1.44269504088896341f);
    return y * __builtin_amdgcn_rcpf(1.0f + e);
}

__device__ __forceinline__ unsigned lane_x(unsigned lane) { return (lane >> 3) * XROW + (((lane & 7u) ^ (lane >> 4)) * 16u); }
__device__ __forceinline__ unsigned lane_w(unsigned lane) { return (lane >> 2) * WROW + (((lane & 3u) ^ ((lane >> 5) << 1)) * 16u); }

#define STAMP(i) do { } while (0)

__device__ __forceinline__ void stage_head(const Params& P, unsigned V) {
    const unsigned lane = threadIdx.x & 63u, vx = lane_x(lane);
    const unsigned done = ld_agent(P.me + OFF_CTR + blockIdx.x * 128u);
    const __amdgpu_buffer_rsrc_t rs = mk_brsrc(P.x, X_BYTES), rd = mk_brsrc(P.me, OFF_YL);
    u32x4 r[4];
#pragma unroll
    for (int i = 0; i < 4; ++i) r[i] = __builtin_amdgcn_raw_buffer_load_b128(rs, (i & 1) ? (vx ^ 64u) : vx, (V * 4u + i) * (8u * XROW), 0);
    asm volatile("" ::: "memory");
    const unsigned dofs = OFF_SX + ((rfl(done) + 1u) & 1u) * X_BYTES + V * 4096u;
#pragma unroll
    for (int i = 0; i < 4; ++i) st128<17>(r[i], rd, lane * 16u, dofs + i * 1024u);
}
__device__ __forceinline__ void push_head(const Params& P, unsigned g, unsigned q, unsigned W) {
    const unsigned lane = threadIdx.x & 63u;
    const unsigned done = ld_agent(P.me + OFF_CTR + blockIdx.x * 128u);
    const __amdgpu_buffer_rsrc_t rs = mk_brsrc(P.w, W_BYTES);
    const unsigned so = (W & 1u) * 32u * WROW + q * 4096u + (W >> 1) * 128u;
    u32x4 r[4];
#pragma unroll
    for (int i = 0; i < 4; ++i) r[i] = __builtin_amdgcn_raw_buffer_load_b128(rs, lane_w(lane), so + (i & 1) * 64u + (i >> 1) * (16u * WROW), 0);
    asm volatile("" ::: "memory");
    const __amdgpu_buffer_rsrc_t rd = mk_brsrc(P.wsg[g], OFF_SX);
    const unsigned dofs = OFF_WS + (unsigned)P.rank * WS_SRC + ((rfl(done) + 1u) & 1u) * (2u << 20) + (W >> 1) * 8192u + (W & 1u) * 2048u;
#pragma unroll
    for (int i = 0; i < 4; ++i) { if (i >= HS) WAIT_V(HS - 1); st128<17>(r[i], rd, lane * 16u, dofs + (i & 1) * 4096u + (i >> 1) * 1024u); }
}

__device__ __forceinline__ void stager(const Params& P, unsigned idx, unsigned call, unsigned dbg_role) {
    const unsigned lane = threadIdx.x & 63u, w = rfl(threadIdx.x >> 6), V = idx * 8u + w, par = call & 1u;
    const unsigned vx = lane_x(lane), vw = lane_w(lane);
    const unsigned long long pw = (unsigned long long)P.w, px = (unsigned long long)P.x;
    char* const flag = P.wsg[lane & 7u] + OFF_STG + (unsigned)P.rank * 256u + idx * 4u;
    for (unsigned s = 0u; s < 8u; ++s) {
        u32x4 r[4][4];
#pragma unroll
        for (int k = 0; k < 4; ++k) {
            const unsigned n = V + k * (NSTAGE * 8u), bi = s ? n : (n / WREST) * 64u + NPUSH + n % WREST;
            const bool x = bi >= 512u, valid = s || n < 8u * WREST;
            const unsigned so = x ? (bi - 512u) * (32u * XROW) + s * 128u : (s * 64u + (bi & 1u) * 32u) * WROW + (bi >> 1) * 128u;
            const __amdgpu_buffer_rsrc_t rs = mk_brsrc((const void*)(x ? px : pw), valid ? (x ? X_BYTES : W_BYTES) : 0u);
            const unsigned v0 = x ? vx : vw, v1 = x ? (vx ^ 64u) : vw, o1 = x ? 8u * XROW : 64u, o2 = x ? 16u * XROW : 16u * WROW;
#pragma unroll
            for (int i = 0; i < 4; ++i) r[k][i] = __builtin_amdgcn_raw_buffer_load_b128(rs, (i & 1) ? v1 : v0, valid ? so + (i & 1) * o1 + (i >> 1) * o2 : 0u, 0);
        }
#pragma unroll
        for (int k = 0; k < 4; ++k) {
            const unsigned n = V + k * (NSTAGE * 8u), bi = s ? n : (n / WREST) * 64u + NPUSH + n % WREST;
            const bool x = bi >= 512u, valid = s || n < 8u * WREST;
            const __amdgpu_buffer_rsrc_t rd = mk_brsrc(P.me, valid ? OFF_YL : 0u);
            const unsigned dofs = x ? OFF_SX + par * X_BYTES + s * (512u << 10) + (bi - 512u) * 4096u
                                    : OFF_SW + par * W_BYTES + s * (2u << 20) + (bi >> 1) * 8192u + (bi & 1u) * 2048u;
            const unsigned o1 = x ? 1024u : 4096u, o2 = x ? 2048u : 1024u;
#pragma unroll
            for (int i = 0; i < 4; ++i) st128<17>(r[k][i], rd, lane * 16u, valid ? dofs + (i & 1) * o1 + (i >> 1) * o2 : 0u);
        }
        WAIT_V(0);
        BAR();
        if (threadIdx.x < 8u) st_sys(flag, call * 16u + s + 2u);
        STAMP(s);
    }
}

struct Batch { unsigned piece, so, dofs; };
__device__ __forceinline__ void mover(const Params& P, unsigned g, unsigned q, unsigned j, unsigned call, LAS char* lds, unsigned dbg_role) {
    const unsigned lane = threadIdx.x & 63u, w = rfl(threadIdx.x >> 6);
    const unsigned rank = (unsigned)P.rank;
    char* const me = P.me;
    char* const peer = P.wsg[g];
    volatile LAS unsigned* const lw = (volatile LAS unsigned*)lds;
    const unsigned ps = (lane >> 2) & 7u, pt = lane & 3u;
    const unsigned size = lane >= 32u ? 0u : (lane == 1u ? WREST : (pt == 1u ? 64u : (pt == 3u ? 16u : 48u)));
    const unsigned x0 = 112u * ps + 48u * min(ps, 8u - LAG) + 16u * min(ps, 8u - FILL) + (ps ? WREST : 0u);
    const unsigned start = lane == 1u ? 112u : pt < 2u ? x0 - 64u * pt
                         : (pt == 2u ? (ps < 8u - LAG ? x0 + 48u : NPULL - 16u * FILL - 48u * (8u - ps))
                                     : (ps < 8u - FILL ? x0 + (ps < 8u - LAG ? 96u : 48u) : NPULL - 16u * (8u - ps)));

    if (w == 0u) {
        GAS unsigned* const cnt = (GAS unsigned*)(me + OFF_FLG + q * 256u) + lane;
        const u32x4 rf = mk_rsrc(me + OFF_STG + q * 256u, 128u);
        const unsigned long long t0 = __builtin_amdgcn_s_memrealtime();
        unsigned seen = 0u, rounds = 1u;
        bool pushed = j != 0u;
        for (unsigned n = 0u;; ++n) {
            const unsigned gone = rfl(lw[63]), v = lw[lane], d = v - seen;
            if (lane < 32u && d != 0u && (rdl(v, 60u) >= start + size || gone != 0u)) {
                __hip_atomic_fetch_add(cnt, d, __ATOMIC_RELAXED, __HIP_MEMORY_SCOPE_SYSTEM);
                seen = v;
            }
            if (gone == MW && pushed) break;
            if ((n & 4095u) == 4095u && (rfl(ld_agent(me + OFF_TMO)) == call || __builtin_amdgcn_s_memrealtime() - t0 > DEADLINE_TICKS)) {
                if (lane == 0u) st_agent(me + OFF_TMO, call);
                break;
            }
            if (rounds < 9u || !pushed) {
                const unsigned sv = lw[64u + lane];
                if (!pushed && __ballot(lane - 24u < NPUSH / 7u && (int)(sv - call) < 0) == 0ull) {
                    if (lane == 1u) __hip_atomic_fetch_add(cnt, (unsigned)NPUSH, __ATOMIC_RELAXED, __HIP_MEMORY_SCOPE_SYSTEM);
                    pushed = true;
                }
                while (rounds < 9u && __ballot(lane < NSTAGE && (int)(sv - (call * 16u + rounds + 1u)) < 0) == 0ull) ++rounds;
                if (lane == 0u) lw[62] = rounds;
                if ((n & (rounds < 3u ? 3u : 7u)) == 0u)
                    asm volatile("s_mov_b32 m0, %0\n\ts_nop 0\n\tbuffer_load_dword %1, %2, 0 offen sc0 sc1 lds\n\ts_waitcnt vmcnt(24)"
                                 :: "s"((unsigned)(uintptr_t)lds + 256u), "v"(lane * 4u), "s"(rf) : "memory");
            }
            __builtin_amdgcn_s_sleep(1);
        }
        return;
    }

    if (w > MW) return;
    const unsigned par = call & 1u, vd = lane * 16u;
    const unsigned lo = min(rank, q), hi = max(rank, q);
    auto batch_of = [&](unsigned U) {
        Batch b;
        b.piece = (unsigned)__builtin_ctz((unsigned)__ballot(U - start < size) | 0x80000000u);
        const unsigned r = U - rdl(start, b.piece), s = b.piece >> 2, t = b.piece & 3u;
        if (t == 1u) {
            const unsigned cb = s ? r : r + NPUSH;
            b.so = OFF_SW + par * W_BYTES + s * (2u << 20) + rank * (256u << 10) + cb * 4096u;
            b.dofs = OFF_WS + q * WS_SRC + (s ? s : 8u * par) * (256u << 10) + cb * 4096u;
        } else {
            unsigned mb = r >> 3; mb += mb >= lo ? 1u : 0u; mb += mb >= hi ? 1u : 0u;
            const unsigned o = s * (512u << 10) + (t == 3u ? rank * 64u + r * 4u : mb * 64u + t * 16u + (r & 7u) * 4u) * 1024u;
            b.so = OFF_SX + par * X_BYTES + o;
            b.dofs = OFF_XS + q * XS_SRC + o;
        }
        b.piece = rfl(b.piece); b.so = rfl(b.so); b.dofs = rfl(b.dofs);
        asm("" : "+s"(b.piece), "+s"(b.so), "+s"(b.dofs));
        return b;
    };
    const __amdgpu_buffer_rsrc_t rd = mk_brsrc(me, OFF_SX);
    const __amdgpu_buffer_rsrc_t rc = mk_brsrc(me + OFF_CNT + q * 256u, 256u);
    const unsigned vc = lane == 0u ? 0u : 4096u;
    const unsigned tbase = (call - 1u) * NPULL - 3u * NMOVER;
    static_assert(GO_AT <= 0 && LAG <= 7 && FILL <= 7 && NPUSH % 14 == 0 && NPUSH < 64, "");
    auto bump = [&](unsigned word) { if (lane == 0u) __hip_atomic_fetch_add((LAS unsigned*)lds + word, 1u, __ATOMIC_RELAXED, __HIP_MEMORY_SCOPE_WORKGROUP); };
    unsigned known = 1u;
    auto staged = [&](unsigned piece) {
        const unsigned round = piece < 4u ? (piece == 1u ? 1u : 0u) : (piece >> 2) + 1u;
        if (round < known) return true;
        const unsigned long long t0 = __builtin_amdgcn_s_memrealtime();
        for (unsigned n = 1u; (known = rfl(lw[62])) <= round; ++n) {
            if ((n & 1023u) == 0u && (rfl(ld_agent(me + OFF_TMO)) == call || __builtin_amdgcn_s_memrealtime() - t0 > DEADLINE_TICKS)) {
                if (lane == 0u) st_agent(me + OFF_TMO, call);
                return false;
            }
            __builtin_amdgcn_s_sleep(1);
        }
        return true;
    };
    auto pull = [&](unsigned so, bool valid) { return __builtin_amdgcn_raw_buffer_load_b128(mk_brsrc(peer, valid ? OFF_YL : 0u), vd, so, 17); };
    STAMP(0);
    const unsigned W = j * MW + w - 1u;
    Batch b0 = batch_of(W);
    bool go = true;
    {
        const char* const fp = lane < 63u ? me + OFF_STG + q * 256u + (32u + min(lane, NHEAD - 1u)) * 4u : me + OFF_TMO;
        const unsigned long long t0 = __builtin_amdgcn_s_memrealtime();
        for (;;) {
            const unsigned v = ld_sys(fp);
            if (__ballot(lane < 63u && (int)(v - call) < 0) == 0ull) break;
            if (rdl(v, 63u) == call || __builtin_amdgcn_s_memrealtime() - t0 > DEADLINE_TICKS) {
                if (lane == 0u) st_agent(me + OFF_TMO, call);
                go = false; break;
            }
            __builtin_amdgcn_s_sleep(2);
        }
    }
    if (go) {
        STAMP(6);
        u32x4 cur[4], nxt[4];
#pragma unroll
        for (int i = 0; i < 4; ++i) cur[i] = pull(b0.so + i * 1024u, true);
        unsigned T0 = W, T1 = NMOVER + W, T2 = 2u * NMOVER + W;
        Batch b1 = batch_of(T1);
        __builtin_amdgcn_s_waitcnt(0x0F70);
        for (;;) {
            const bool v1 = T1 < NPULL && staged(b1.piece);
#pragma unroll
            for (int i = 0; i < 4; ++i) nxt[i] = pull(b1.so + i * 1024u, v1);
            const unsigned tk = (unsigned)__builtin_amdgcn_raw_ptr_buffer_atomic_add_i32(1, rc, vc, 0, 0);
            asm volatile("" ::: "memory");
#pragma unroll
            for (int i = 0; i < 4; ++i) st128<16>(cur[i], rd, vd, b0.dofs + i * 1024u);
            if (T1 == NPULL + GO_AT && lane == 0u) st_agent(me + OFF_GO + q * 128u, call);
            const Batch b2 = batch_of(T2 < NPULL ? T2 : 0u);
            WAIT_V(0);
            bump(b0.piece);
            if (lane == 0u) __hip_atomic_fetch_max((LAS unsigned*)lds + 60, T0, __ATOMIC_RELAXED, __HIP_MEMORY_SCOPE_WORKGROUP);
            if (!v1) break;
#pragma unroll
            for (int i = 0; i < 4; ++i) cur[i] = nxt[i];
            b0 = b1; b1 = b2; T0 = T1; T1 = T2; T2 = rfl(tk) - tbase;
        }
    }
    bump(63u);
    STAMP(1);
}

__device__ __forceinline__ void gemm_tile(const Params& P, unsigned grp, unsigned q, unsigned tile, unsigned call, LAS char* lds, unsigned dbg_role) {
    const unsigned lane = threadIdx.x & 63u, g16 = lane >> 4;
    const unsigned w = rfl(threadIdx.x >> 6), wm = w >> 1, wn = w & 1u;
    const unsigned rank = (unsigned)P.rank, g = (q - rank - 1u) & 7u;
    const unsigned h = tile & 1u, c = tile >> 1;
    const bool own = g == 7u;
    char* const me = P.me;
    char* const peer = P.wsg[grp];
    const unsigned par = call & 1u;
    const unsigned lds0 = (unsigned)(uintptr_t)lds;
    LAS unsigned* ctl = (LAS unsigned*)(lds + LDS_CTL);
    STAMP(0);
    if (tile < (own ? NHEAD : NPUSH / 7u)) {
        WAIT_V(0);
        BAR();
        if (threadIdx.x < (own ? 8u : 1u)) st_sys((own ? P.wsg[lane] : peer) + OFF_STG + rank * 256u + ((own ? 32u : 24u) + tile) * 4u, call);
        STAMP(19);
    }

    const unsigned nraw = own ? 8u : 16u, nsrc = own ? 7u : 6u;
    unsigned tabA, tabB, vF;
    unsigned long long gend;
    {
        const bool extra = !own && lane < 8u, raw = lane < nraw;
        const unsigned e = lane - nraw, s = raw ? lane & 7u : (own ? e / 7u : e / 6u), i = e - s * nsrc;
        const unsigned p = (rank + 1u + i + (i >= g ? 1u : 0u)) & 7u;
        tabA = raw ? ((extra ? rank : q) * 512u + h * 256u) * XROW + s * 128u : p * XS_SRC + s * (512u << 10) + (q * 2u + h) * 32768u;
        tabB = raw ? s * 64u * WROW + (extra ? q : rank) * 4096u + c * 320u : p * WS_SRC + (s ? s : 8u * par) * (256u << 10) + c * 20480u;
        vF = raw ? rank * 256u : p * 256u + s * 16u;
        gend = __ballot(raw ? lane == nraw - 1u : i == nsrc - 1u);
    }
    const unsigned b0 = w < 4u ? 3u * w : 12u + 2u * (w - 4u);
    const unsigned cbmax = c == 12u ? 3u : 4u;
    u32x4 ra, rb, rx;
    unsigned za, zb, zx1, vP[7];
    const unsigned zx0 = w == 7u ? 2048u : 0u, mx = w < 4u ? ~0u : 0u;
    auto staging = [&](bool raw) {
        ra = raw ? mk_rsrc(P.x, X_BYTES) : mk_rsrc(me + OFF_XS, 8u * XS_SRC);
        rb = raw ? mk_rsrc(P.w, W_BYTES) : mk_rsrc(me + OFF_WS, 8u * WS_SRC);
        rx = w < 4u ? rb : mk_rsrc(me + OFF_FLG, zx0);
        za = ra.z; zb = rb.z; zx1 = rx.z;
#pragma unroll
        for (int i = 0; i < 4; ++i) vP[i] = (raw ? (lane_x(lane) ^ ((i & 1) * 64u)) + (4u * w + i) * (8u * XROW) : lane * 16u + (4u * w + i) * 1024u) - i * 1024u;
#pragma unroll
        for (int i = 0; i < 3; ++i) { const unsigned b = b0 + i, cb = min(b >> 2, cbmax);
            vP[4 + i] = raw ? lane_w(lane) + cb * 64u + (b & 3u) * (16u * WROW) : lane * 16u + cb * 4096u + (b & 3u) * 1024u; }
        if (w >= 4u) vP[6] = vF;
    };
    staging(true);
    const unsigned la0 = lds0 + w * 4096u, lb0 = lds0 + 32768u + b0 * 1024u;
    const unsigned lx0 = w < 4u ? lb0 + 2048u : (w == 7u ? lds0 + 53248u : lds0 + 0x30000u), kx = (w < 4u || w == 7u) ? ~0u : 0u;
    unsigned oa = 0u, ob = 0u;
    auto piece = [&](int i, unsigned so) {
        if (i == 0) dma16<0, false>(ra, vP[0], oa, la0 + so);
        if (i == 1) dma16<1024, false>(ra, vP[1], oa, la0 + so);
        if (i == 2) dma16<2048, false>(ra, vP[2], oa, la0 + so);
        if (i == 3) dma16<3072, false>(ra, vP[3], oa, la0 + so);
        if (i == 4) dma16<0, false>(rb, vP[4], ob, lb0 + so);
        if (i == 5) dma16<0, false>(rb, vP[5], ob, lb0 + so + 1024u);
        if (i == 6) dma16<0, true>(rx, vP[6], ob & mx, lx0 + (so & kx));
    };
    unsigned it = 0u, ready = nraw;
    unsigned gm = 0u;
    auto decide = [&]() { asm volatile("s_cmp_lt_u32 %1, %2\n\ts_cselect_b32 %0, -1, 0" : "=s"(gm) : "s"(it), "s"(ready) : "scc"); };
    auto planA = [&]() { oa = rdl(tabA, it & 63u); ra.z = za & gm; asm volatile("" : "+s"(oa), "+s"(ra)); };
    auto planB = [&]() { ob = rdl(tabB, it & 63u); rb.z = zb & gm; asm volatile("" : "+s"(ob), "+s"(rb)); };
    auto planX = [&]() { rx.z = zx1 & gm & (it + 3u == ready ? ~0u : mx); it -= gm; asm volatile("" : "+s"(it), "+s"(rx)); };
    auto plan = [&]() { decide(); planA(); planB(); planX(); asm volatile("s_nop 4"); };
    const unsigned xw = own ? 12u : 8u * h, needx = (own ? 16u : 48u) * call, needw = 64u * call;
    auto arrived = [&](unsigned vw, unsigned vx) {
        const unsigned long long miss = __ballot((int)(vw - needw) < 0 || (int)(vx - needx) < 0) >> nraw << nraw;
        return 64u - (unsigned)__builtin_clzll(gend & ((miss & (0ull - miss)) - 1ull));
    };

    const unsigned aoff = wm * 8192u + (lane & 15u) * 128u + ((g16 ^ ((lane & 15u) >> 1)) * 16u);
    const unsigned lp = (g16 >> 1) * 1024u + (8u * (g16 & 1u) + ((lane & 15u) >> 2)) * 64u + (lane & 3u) * 8u;
    const unsigned be = 32768u + ((5u * wn) >> 1) * 4096u + lp + (((wn ^ g16) & 1u) * 32u);
    const unsigned bo = 32768u + ((5u * wn + 1u) >> 1) * 4096u + lp + (((wn ^ g16 ^ 1u) & 1u) * 32u);

    f32x4 acc[4][5];
#pragma unroll
    for (int m = 0; m < 4; ++m)
#pragma unroll
        for (int n = 0; n < 5; ++n) acc[m][n] = (f32x4){0.f, 0.f, 0.f, 0.f};
    const __amdgpu_buffer_rsrc_t rv = mk_brsrc(me + OFF_SV + ((min(g, 6u) * NTILE + tile) * 8u + w) * SV_WAVE, SV_WAVE);
    s16x8 fa[2][4]; s16x4 fb[2][5][2];
    auto zero1 = [&]() {
#pragma unroll
        for (int m = 0; m < 4; ++m) fa[1][m] = (s16x8){0, 0, 0, 0, 0, 0, 0, 0};
#pragma unroll
        for (int n = 0; n < 5; ++n) { fb[1][n][0] = (s16x4){0, 0, 0, 0}; fb[1][n][1] = (s16x4){0, 0, 0, 0}; }
    };
    auto rd = [&](int kh, int r, LAS char* st) {
        if (r >= 2 && r < 6) fa[kh][r - 2] = *(const LAS s16x8*)(st + (kh ? (aoff ^ 64u) : aoff) + (r - 2) * 2048);
        else { const int n = r < 2 ? 0 : (r - 4) >> 1, hi = r & 1;
               fb[kh][n][hi] = __builtin_amdgcn_ds_read_tr16_b64_v4i16((LAS s16x4*)(st + ((n & 1) ? bo : be) + (n >> 1) * 4096 + kh * 2048 + hi * 256)); }
    };
    auto mm = [&](int kh, int i) {
        const int n = i >> 2, m = i & 3;
        acc[m][n] = __builtin_amdgcn_mfma_f32_16x16x32_bf16(
            __builtin_bit_cast(bf16x8, __builtin_shufflevector(fb[kh][n][0], fb[kh][n][1], 0, 1, 2, 3, 4, 5, 6, 7)),
            __builtin_bit_cast(bf16x8, fa[kh][m]), acc[m][n], 0, 0, 0);
    };
    zero1();

    unsigned t = 0u;
    unsigned s0 = 0u, s1 = STAGE, s2 = 2u * STAGE;
    bool bad = false;
    for (;;) {
        if (it == t) {
            plan();
#pragma unroll
            for (int i = 0; i < 7; ++i) piece(i, s0);
            plan();
#pragma unroll
            for (int i = 0; i < 7; ++i) piece(i, s1);
            plan();
        }
        const unsigned park = (!own && t < 8u) ? 8u : 64u;
        do {
            LAS char* st = lds + s0;
#pragma unroll
            for (int i = 0; i < 20; ++i) {
                if (i == 2) {
                    __builtin_amdgcn_s_waitcnt(0xC07F);
                    WAIT_V(7);
                    BAR();
                    __builtin_amdgcn_sched_barrier(0);
                    __builtin_amdgcn_s_setprio(3);
                }
                if (i == 12) __builtin_amdgcn_s_setprio(2);
                mm(1, i);
                if (i >= 2 && i < 16) rd(0, i - 2, st);
                if (i % 5 == 4) piece(i / 5, s2);
                if (i == 17 && __builtin_expect(it == ready && it == t + 3u && t >= nraw, 0)) {
                    const LAS char* fr = st + 53248u + lane * 16u;
                    ready = max(ready, arrived(*(const LAS unsigned*)(fr + 4), *(const LAS unsigned*)(fr + xw)));
                }
                if (i == 18) decide();
                __builtin_amdgcn_sched_barrier(0);
            }
#pragma unroll
            for (int i = 0; i < 20; ++i) {
                if (i == 2) __builtin_amdgcn_s_setprio(1);
                if (i == 12) __builtin_amdgcn_s_setprio(0);
                mm(0, i);
                if (i < 14) rd(1, i, st);
                if (i % 5 == 4 && i < 15) piece(4 + i / 5, s2);
                if (i == 17) planA();
                if (i == 18) planB();
                if (i == 19) planX();
                __builtin_amdgcn_sched_barrier(0);
            }
            { const unsigned x = s0; s0 = s1; s1 = s2; s2 = x; }
            ++t;
        } while (t < min(it, park));
        if (t == 64u) break;
        if (t < it) {
#pragma unroll
            for (int i = 0; i < 20; ++i) mm(1, i);
#pragma unroll
            for (int m = 0; m < 4; ++m)
#pragma unroll
                for (int n = 0; n < 5; ++n) { st128<0>(__builtin_bit_cast(u32x4, acc[m][n]), rv, lane * 16u, (n * 4 + m) * 1024); acc[m][n] = (f32x4){0.f, 0.f, 0.f, 0.f}; }
            zero1();
            continue;
        }
        STAMP(1 + 2 * (seg < 8u ? seg : 8u));
        if (t == nraw) staging(false);
        if (w == 0u) {
            const char* fp = me + OFF_FLG + vF;
            const unsigned long long t0 = __builtin_amdgcn_s_memrealtime();
            unsigned res;
            for (unsigned n = 1u;; ++n) {
                const unsigned vw = ld_sys(fp + 4), vx = ld_sys(fp + xw);
                res = arrived(vw, vx);
                if (res > t) break;
                if ((n & 15u) == 0u && (rfl(ld_agent(me + OFF_TMO)) == call || __builtin_amdgcn_s_memrealtime() - t0 > DEADLINE_TICKS)) {
                    st_agent(me + OFF_TMO, call);
                    res = 0x100u + 64u; break;
                }
                __builtin_amdgcn_s_sleep(4);
            }
            ctl[0] = res;
            asm volatile("s_waitcnt vmcnt(0) lgkmcnt(0)" ::: "memory");
        }
        BAR();
        const unsigned res = rfl(*(volatile LAS unsigned*)ctl);
        bad |= res > 0xffu;
        ready = res & 0xffu;
        STAMP(2 + 2 * (seg < 8u ? seg : 8u));
    }
#pragma unroll
    for (int i = 0; i < 20; ++i) mm(1, i);
    STAMP(20);

    const unsigned nu = (c == 12u && wn == 1u) ? 6u : 10u;
    const __amdgpu_buffer_rsrc_t ro = mk_brsrc(P.out, OUT_BYTES);
    const unsigned vo = ((g16 & 1u) * 16u + (lane & 15u)) * WROW + (g16 >> 1) * 16u;
    const unsigned vo2 = nu == 10u ? vo : 0x80000000u;
    unsigned so0 = (h * 256u + wm * 64u) * WROW + q * 4096u + c * 320u + wn * 160u, so1 = so0 + 32u * WROW;
    asm("" : "+s"(so0), "+s"(so1));
    auto emit = [&](int u, const f32x4& v0, const f32x4& v1) {
        const unsigned a0 = pk_bf16(silu(v0[0]), silu(v0[1])), a1 = pk_bf16(silu(v0[2]), silu(v0[3]));
        const unsigned c0 = pk_bf16(silu(v1[0]), silu(v1[1])), c1 = pk_bf16(silu(v1[2]), silu(v1[3]));
        const auto x0 = __builtin_amdgcn_permlane16_swap(a0, c0, false, false);
        const auto x1 = __builtin_amdgcn_permlane16_swap(a1, c1, false, false);
        u32x4 d = {x0[0], x1[0], x0[1], x1[1]};
        if (bad) d = (u32x4){0x7fc07fc0u, 0x7fc07fc0u, 0x7fc07fc0u, 0x7fc07fc0u};
        st128<16>(d, ro, (u < 6 ? vo : vo2) + (u >> 1) * 32u, (u & 1) ? so1 : so0);
    };
    if (own) {
        bad |= rfl(ld_agent(me + OFF_TMO)) == call;
#pragma unroll
        for (int u = 0; u < 10; ++u) emit(u, acc[2 * (u & 1)][u >> 1], acc[2 * (u & 1) + 1][u >> 1]);
        STAMP(23);
        return;
    }
    const unsigned yo = OFF_YL + par * Y_PAR + (tile * 8u + w) * Y_WAVE;
    {
        const __amdgpu_buffer_rsrc_t ry = mk_brsrc(me + yo + q * Y_PEER, Y_WAVE);
#pragma unroll
        for (int u = 0; u < 10; ++u) {
            const f32x4 v0 = acc[2 * (u & 1)][u >> 1], v1 = acc[2 * (u & 1) + 1][u >> 1];
            u32x4 d = {pk_bf16(v0[0], v0[1]), pk_bf16(v0[2], v0[3]), pk_bf16(v1[0], v1[1]), pk_bf16(v1[2], v1[3])};
            if (bad) d = (u32x4){0x7fc07fc0u, 0x7fc07fc0u, 0x7fc07fc0u, 0x7fc07fc0u};
            st128<17>(d, ry, lane * 16u, u * 1024);
        }
        WAIT_V(0);
        BAR();
        if (threadIdx.x == 0u) st_sys(me + OFF_YF + q * 4096u + tile * 128u, call);
    }
    STAMP(21);
    if (w == 0u) {
        const char* fp = lane == 0u ? peer + OFF_YF + rank * 4096u + tile * 128u : (lane == 1u ? me + OFF_GO + q * 128u : me + OFF_TMO);
        const unsigned long long t0 = __builtin_amdgcn_s_memrealtime();
        bool have = false;
        for (;;) {
            const unsigned v = ld_sys(fp);
            if (rdl(v, 2u) == call) { bad = true; break; }
            const unsigned m = (unsigned)__ballot((int)(v - call) >= 0);
            if (!have && (m & 1u)) { have = true; if (lane == 0u) fp = me + OFF_TMO; }
            if (have && (m & 2u)) break;
            if (__builtin_amdgcn_s_memrealtime() - t0 > DEADLINE_TICKS) {
                if (lane == 0u) st_agent(me + OFF_TMO, call);
                bad = true; break;
            }
            __builtin_amdgcn_s_sleep(2);
        }
        if (lane == 0u) ctl[0] = bad ? 1u : 0u;
        asm volatile("s_waitcnt lgkmcnt(0)" ::: "memory");
    }
    BAR();
    bad |= rfl(*(volatile LAS unsigned*)ctl) != 0u;
    STAMP(22);
    const __amdgpu_buffer_rsrc_t rp = mk_brsrc(peer + yo + rank * Y_PEER, bad ? 0u : nu * 1024u);
    u32x4 pv[10];
    auto fetch = [&](int u) {
        unsigned n = 0u;
        if (lane == 0u) n = __hip_atomic_fetch_add(ctl + 4, 1u, __ATOMIC_RELAXED, __HIP_MEMORY_SCOPE_WORKGROUP);
        n = rfl(n);
        for (unsigned k = 0u; n >= rfl(*(volatile LAS unsigned*)(ctl + 5)) + TOK && k < (1u << 20); ++k) __builtin_amdgcn_s_sleep(1);
        pv[u] = __builtin_amdgcn_raw_buffer_load_b128(rp, lane * 16u + u * 1024u, 0, 17);
        asm volatile("" ::: "memory");
#pragma unroll
        for (int k = 0; k < 2; ++k) acc[2 * (u & 1) + k][u >> 1] = __builtin_bit_cast(f32x4, __builtin_amdgcn_raw_buffer_load_b128(rv, lane * 16u, ((u >> 1) * 4 + 2 * (u & 1) + k) * 1024, 0));
        asm volatile("" ::: "memory");
    };
    fetch(0);
#pragma unroll
    for (int u = 0; u < 10; ++u) {
        asm volatile("" : "+v"(pv[u]));
        if (lane == 0u) __hip_atomic_fetch_add(ctl + 5, 1u, __ATOMIC_RELAXED, __HIP_MEMORY_SCOPE_WORKGROUP);
        if (u < 9) fetch(u + 1);
        __builtin_amdgcn_sched_barrier(0);
        const u32x4 d = pv[u];
        f32x4 v0 = acc[2 * (u & 1)][u >> 1], v1 = acc[2 * (u & 1) + 1][u >> 1];
        v0[0] += __builtin_bit_cast(float, d.x << 16); v0[1] += __builtin_bit_cast(float, d.x & 0xffff0000u);
        v0[2] += __builtin_bit_cast(float, d.y << 16); v0[3] += __builtin_bit_cast(float, d.y & 0xffff0000u);
        v1[0] += __builtin_bit_cast(float, d.z << 16); v1[1] += __builtin_bit_cast(float, d.z & 0xffff0000u);
        v1[2] += __builtin_bit_cast(float, d.w << 16); v1[3] += __builtin_bit_cast(float, d.w & 0xffff0000u);
        emit(u, v0, v1);
        __builtin_amdgcn_sched_barrier(0);
    }
    STAMP(23);
}

__global__ void __launch_bounds__(512, 2) fused_gemm_rs(Params P) {
    __shared__ __attribute__((aligned(16))) char smem[LDS_BYTES];
    LAS char* lds = (LAS char*)smem;
    volatile LAS unsigned* const lw = (volatile LAS unsigned*)lds;
    const unsigned w = rfl(threadIdx.x >> 6), rank = (unsigned)P.rank;
    const unsigned guess = (blockIdx.x & ~7u) | ((unsigned)__builtin_amdgcn_s_getreg((3 << 11) | 20) & GUESS_MASK);
    auto head = [&](unsigned role) {
        const unsigned slot = role >> 3, q = (P.qmap >> ((role & 7u) * 4u)) & 7u;
        if (w == 0u || slot >= 32u) return;
        if (slot < NSW + NPW) return;
        const unsigned W = (slot - NSW - NPW) * 7u + w - 1u;
        if (q != rank) { if (W < NPUSH) push_head(P, role & 7u, q, W); }
        else if (W < 128u) stage_head(P, W);
    };
    if (w == 0u) {
        char* const me = P.me;
        const __amdgpu_buffer_rsrc_t rc = mk_brsrc(me + OFF_CLM, 32768u);
        const unsigned v0 = threadIdx.x == 0u ? 0u : 65536u;
        unsigned role = guess, old = (unsigned)__builtin_amdgcn_raw_ptr_buffer_atomic_add_i32(1, rc, v0, role * 128u, 0);
        const unsigned done = rfl(ld_agent(me + OFF_CTR + blockIdx.x * 128u));
        lw[threadIdx.x] = 0u; lw[64u + threadIdx.x] = 0u;
        while (rfl(old) != done) {
            __builtin_amdgcn_raw_ptr_buffer_atomic_add_i32(-1, rc, v0, role * 128u, 0);
            role = (role + 1u) & 255u;
            if (role == guess) { role = 256u; st_agent(me + OFF_TMO, done + 1u); break; }
            old = (unsigned)__builtin_amdgcn_raw_ptr_buffer_atomic_add_i32(1, rc, v0, role * 128u, 0);
        }
        if (threadIdx.x == 0u) { lw[LDS_CTL / 4u + 1u] = role; lw[LDS_CTL / 4u + 2u] = done + 1u; lw[LDS_CTL / 4u + 4u] = 0u; lw[LDS_CTL / 4u + 5u] = 0u; }
        __builtin_amdgcn_s_waitcnt(0x0070);
    } else head(guess);
    BAR();
    const unsigned role = rfl(lw[LDS_CTL / 4u + 1u]), call = rfl(lw[LDS_CTL / 4u + 2u]);
    if (role != guess) { head(role); BAR(); }
    if (threadIdx.x == 0u) st_agent(P.me + OFF_CTR + blockIdx.x * 128u, call);
    const unsigned q = (P.qmap >> ((role & 7u) * 4u)) & 7u, slot = role >> 3;
    if (slot >= 32u) return;
    if (slot >= NSW + NPW) gemm_tile(P, role & 7u, q, slot - NSW - NPW, call, lds, role);
    else if (slot >= NSW && q != rank) mover(P, role & 7u, q, slot - NSW, call, lds, role);
    else stager(P, slot < NSW ? role : 8u * NSW + slot - NSW, call, role);
}

unsigned make_qmap(int rank) {
    static const unsigned char kNear[NR][NR] = {
        {6, 0, 2, 3, 1, 5, 4, 7}, {7, 1, 4, 5, 0, 3, 6, 2}, {3, 2, 0, 6, 4, 5, 7, 1}, {2, 3, 6, 0, 5, 1, 7, 4},
        {5, 4, 1, 7, 2, 0, 6, 3}, {4, 5, 7, 1, 3, 2, 0, 6}, {0, 6, 3, 2, 7, 4, 1, 5}, {1, 7, 5, 4, 6, 3, 2, 0}};
    unsigned m = 0u;
    for (int g = 0; g < NR; ++g) m |= (unsigned)kNear[rank][g] << (4 * g);
    return m;
}

}

extern "C" void kernel_launch(int rank, int nranks, ncclComm_t comm,
                              hipStream_t stream,
                              void* const* d_in, const int* in_sizes, int n_in,
                              void* d_out, int out_size,
                              const int* in_dtypes, int out_dtype,
                              void* const* d_out_all,
                              void* const* d_ws_all, size_t ws_bytes) {
    (void)comm; (void)in_dtypes; (void)out_dtype; (void)d_out_all;
    if (nranks != NR || (unsigned)rank >= (unsigned)NR || n_in != 2 || ws_bytes < (size_t)WS_NEED ||
        in_sizes[0] != 4096 * 512 || in_sizes[1] != 512 * 16384 || out_size != 512 * 16384) {
        fprintf(stderr, "unexpected problem geometry\n");
        return;
    }
    Params P;
    P.x = (const char*)d_in[0];
    P.w = (const char*)d_in[1];
    P.out = (char*)d_out;
    P.me = (char*)d_ws_all[rank];
    P.rank = rank;
    P.qmap = make_qmap(rank);
    for (int g = 0; g < NR; ++g) P.wsg[g] = (char*)d_ws_all[(P.qmap >> (4 * g)) & 7u];
    fused_gemm_rs<<<GRID, 512, 0, stream>>>(P);
    HIP_CHECK(hipGetLastError());
}
```

```cpp
#include <hip/hip_runtime.h>
#include <rccl/rccl.h>
#include <cstdio>
#include <cstdint>

#define HIP_CHECK(expr) do { hipError_t _e = (expr); if (_e != hipSuccess) { \
    fprintf(stderr, "HIP error %s at %s:%d\n", hipGetErrorString(_e), __FILE__, __LINE__); \
    return; } } while (0)

#define LAS __attribute__((address_space(3)))
#define GAS __attribute__((address_space(1)))

namespace {

typedef __bf16 bf16x8 __attribute__((ext_vector_type(8)));
typedef __bf16 bf16x2 __attribute__((ext_vector_type(2)));
typedef short s16x4 __attribute__((ext_vector_type(4)));
typedef short s16x8 __attribute__((ext_vector_type(8)));
typedef float f32x4 __attribute__((ext_vector_type(4)));
typedef float f32x2 __attribute__((ext_vector_type(2)));
typedef unsigned u32x4 __attribute__((ext_vector_type(4)));

constexpr int NR = 8;
constexpr unsigned LAG = 4u;
constexpr unsigned FILL = 3u;
constexpr unsigned NPUSH = 28u;
constexpr unsigned MW = 6u;
constexpr unsigned GO_LEAD = 24u;
constexpr unsigned TOK = 4u;
constexpr unsigned XROW = 1024u;
constexpr unsigned WROW = 32768u;
constexpr unsigned X_BYTES = 4096u * XROW, W_BYTES = 512u * WROW, OUT_BYTES = 512u * WROW;
constexpr unsigned NSW = 2u, NPW = 4u, NTILE = 26u;
constexpr unsigned NMOVER = NPW * MW;
constexpr unsigned NSTAGE = 8u * NSW + NPW;
constexpr unsigned NPULL = 1408u - NPUSH, WREST = 64u - NPUSH, NHEAD = 19u;
constexpr int GRID = 256;
static_assert(NSW + NPW + NTILE == 32u && NSTAGE * 8u * 4u == 640u && NSTAGE * 8u * 2u >= 8u * WREST && NHEAD * 7u >= 128u, "");

constexpr unsigned OFF_CTR = 0u;
constexpr unsigned OFF_TMO = 32768u;
constexpr unsigned OFF_CNT = 40960u;
constexpr unsigned OFF_STG = 49152u;
constexpr unsigned OFF_GO = 53248u;
constexpr unsigned OFF_FLG = 65536u;
constexpr unsigned OFF_CLM = 98304u;
constexpr unsigned OFF_YF = 131072u;
constexpr unsigned OFF_XS = 1u << 20;
constexpr unsigned XS_SRC = 4u << 20;
constexpr unsigned OFF_WS = OFF_XS + 8u * XS_SRC;
constexpr unsigned WS_SRC = 9u * (256u << 10);
constexpr unsigned OFF_SX = OFF_WS + 8u * WS_SRC;
constexpr unsigned OFF_SW = OFF_SX + 2u * X_BYTES;
constexpr unsigned Y_WAVE = 10240u, Y_PEER = NTILE * 8u * Y_WAVE, Y_PAR = NR * Y_PEER;
constexpr unsigned OFF_YL = OFF_SW + 2u * W_BYTES;
constexpr unsigned OFF_SV = OFF_YL + 2u * Y_PAR;
constexpr unsigned SV_WAVE = 20480u;
constexpr unsigned WS_NEED = OFF_SV + 7u * NTILE * 8u * SV_WAVE;

constexpr unsigned STAGE = 54272u;
constexpr unsigned LDS_CTL = 3u * STAGE;
constexpr unsigned LDS_BYTES = LDS_CTL + 64u;

constexpr unsigned long long DEADLINE_TICKS = 100000000ull;

struct Params {
    const char* x;
    const char* w;
    char* out;
    char* me;
    char* wsg[NR];
    int rank;
    unsigned qmap;
};

__device__ __forceinline__ unsigned rfl(unsigned v) { return (unsigned)__builtin_amdgcn_readfirstlane((int)v); }
__device__ __forceinline__ unsigned rdl(unsigned v, unsigned l) { return (unsigned)__builtin_amdgcn_readlane((int)v, (int)l); }

__device__ __forceinline__ u32x4 mk_rsrc(const void* p, unsigned bytes) {
    const unsigned long long a = (unsigned long long)p;
    u32x4 r; r.x = (unsigned)a; r.y = (unsigned)(a >> 32) & 0xffffu; r.z = bytes; r.w = 0x00020000u; return r;
}
__device__ __forceinline__ __amdgpu_buffer_rsrc_t mk_brsrc(const void* p, unsigned bytes) {
    return __builtin_amdgcn_make_buffer_rsrc((void*)p, 0, (int)bytes, 0x00020000);
}

template <int OFF, bool SYS> __device__ __forceinline__ void dma16(u32x4 rs, unsigned voff, unsigned soff, unsigned ldsaddr) {
    if constexpr (SYS)
        asm volatile("s_mov_b32 m0, %0\n\ts_nop 0\n\tbuffer_load_dwordx4 %1, %2, %3 offen offset:%4 sc0 sc1 lds"
                     :: "s"(ldsaddr), "v"(voff), "s"(rs), "s"(soff), "i"(OFF) : "memory");
    else
        asm volatile("s_mov_b32 m0, %0\n\ts_nop 0\n\tbuffer_load_dwordx4 %1, %2, %3 offen offset:%4 sc1 lds"
                     :: "s"(ldsaddr), "v"(voff), "s"(rs), "s"(soff), "i"(OFF) : "memory");
}
template <int AUX> __device__ __forceinline__ void st128(u32x4 d, __amdgpu_buffer_rsrc_t rs, unsigned voff, unsigned soff) {
    __builtin_amdgcn_raw_buffer_store_b128(d, rs, voff, soff, AUX);
    asm volatile("s_nop 3" :: "v"(d) : "memory");
}
#define WAIT_V(n) asm volatile("s_waitcnt vmcnt(%0)" :: "i"(n) : "memory")
#define BAR() do { asm volatile("" ::: "memory"); __builtin_amdgcn_s_barrier(); asm volatile("" ::: "memory"); } while (0)

__device__ __forceinline__ unsigned ld_sys(const char* p) {
    return __hip_atomic_load((const GAS unsigned*)p, __ATOMIC_RELAXED, __HIP_MEMORY_SCOPE_SYSTEM);
}
__device__ __forceinline__ void st_sys(char* p, unsigned v) {
    __hip_atomic_store((GAS unsigned*)p, v, __ATOMIC_RELAXED, __HIP_MEMORY_SCOPE_SYSTEM);
}
__device__ __forceinline__ unsigned ld_agent(const char* p) {
    return __hip_atomic_load((const GAS unsigned*)p, __ATOMIC_RELAXED, __HIP_MEMORY_SCOPE_AGENT);
}
__device__ __forceinline__ void st_agent(char* p, unsigned v) {
    __hip_atomic_store((GAS unsigned*)p, v, __ATOMIC_RELAXED, __HIP_MEMORY_SCOPE_AGENT);
}

__device__ __forceinline__ unsigned pk_bf16(float lo, float hi) {
    f32x2 v = {lo, hi};
    return __builtin_bit_cast(unsigned, __builtin_convertvector(v, bf16x2));
}
__device__ __forceinline__ float silu(float y) {
    const float c = __builtin_amdgcn_fmed3f(y, -60.0f, 60.0f);
    const float e = __builtin_amdgcn_exp2f(c * -1.44269504088896341f);
    return y * __builtin_amdgcn_rcpf(1.0f + e);
}

__device__ __forceinline__ unsigned lane_x(unsigned lane) { return (lane >> 3) * XROW + (((lane & 7u) ^ (lane >> 4)) * 16u); }
__device__ __forceinline__ unsigned lane_w(unsigned lane) { return (lane >> 2) * WROW + (((lane & 3u) ^ ((lane >> 5) << 1)) * 16u); }

__device__ __forceinline__ void stage_head(const Params& P, unsigned V) {
    const unsigned lane = threadIdx.x & 63u, vx = lane_x(lane);
    const unsigned done = ld_agent(P.me + OFF_CTR + blockIdx.x * 128u);
    const __amdgpu_buffer_rsrc_t rs = mk_brsrc(P.x, X_BYTES), rd = mk_brsrc(P.me, OFF_YL);
    u32x4 r[4];
#pragma unroll
    for (int i = 0; i < 4; ++i) r[i] = __builtin_amdgcn_raw_buffer_load_b128(rs, (i & 1) ? (vx ^ 64u) : vx, (V * 4u + i) * (8u * XROW), 0);
    asm volatile("" ::: "memory");
    const unsigned dofs = OFF_SX + ((rfl(done) + 1u) & 1u) * X_BYTES + V * 4096u;
#pragma unroll
    for (int i = 0; i < 4; ++i) st128<17>(r[i], rd, lane * 16u, dofs + i * 1024u);
}
__device__ __forceinline__ void push_head(const Params& P, unsigned g, unsigned q, unsigned W) {
    const unsigned lane = threadIdx.x & 63u;
    const unsigned done = ld_agent(P.me + OFF_CTR + blockIdx.x * 128u);
    const __amdgpu_buffer_rsrc_t rs = mk_brsrc(P.w, W_BYTES);
    const unsigned so = (W & 1u) * 32u * WROW + q * 4096u + (W >> 1) * 128u;
    u32x4 r[4];
#pragma unroll
    for (int i = 0; i < 4; ++i) r[i] = __builtin_amdgcn_raw_buffer_load_b128(rs, lane_w(lane), so + (i & 1) * 64u + (i >> 1) * (16u * WROW), 0);
    asm volatile("" ::: "memory");
    const __amdgpu_buffer_rsrc_t rd = mk_brsrc(P.wsg[g], OFF_SX);
    const unsigned dofs = OFF_WS + (unsigned)P.rank * WS_SRC + ((rfl(done) + 1u) & 1u) * (2u << 20) + (W >> 1) * 8192u + (W & 1u) * 2048u;
#pragma unroll
    for (int i = 0; i < 4; ++i) { if (i >= 2) WAIT_V(1); st128<17>(r[i], rd, lane * 16u, dofs + (i & 1) * 4096u + (i >> 1) * 1024u); }
}

__device__ __forceinline__ void stager(const Params& P, unsigned idx, unsigned call) {
    const unsigned lane = threadIdx.x & 63u, w = rfl(threadIdx.x >> 6), V = idx * 8u + w, par = call & 1u;
    const unsigned vx = lane_x(lane), vw = lane_w(lane);
    const unsigned long long pw = (unsigned long long)P.w, px = (unsigned long long)P.x;
    char* const flag = P.wsg[lane & 7u] + OFF_STG + (unsigned)P.rank * 256u + idx * 4u;
    for (unsigned s = 0u; s < 8u; ++s) {
        u32x4 r[4][4];
#pragma unroll
        for (int k = 0; k < 4; ++k) {
            const unsigned n = V + k * (NSTAGE * 8u), bi = s ? n : (n / WREST) * 64u + NPUSH + n % WREST;
            const bool x = bi >= 512u, valid = s || n < 8u * WREST;
            const unsigned so = x ? (bi - 512u) * (32u * XROW) + s * 128u : (s * 64u + (bi & 1u) * 32u) * WROW + (bi >> 1) * 128u;
            const __amdgpu_buffer_rsrc_t rs = mk_brsrc((const void*)(x ? px : pw), valid ? (x ? X_BYTES : W_BYTES) : 0u);
            const unsigned v0 = x ? vx : vw, v1 = x ? (vx ^ 64u) : vw, o1 = x ? 8u * XROW : 64u, o2 = x ? 16u * XROW : 16u * WROW;
#pragma unroll
            for (int i = 0; i < 4; ++i) r[k][i] = __builtin_amdgcn_raw_buffer_load_b128(rs, (i & 1) ? v1 : v0, valid ? so + (i & 1) * o1 + (i >> 1) * o2 : 0u, 0);
        }
#pragma unroll
        for (int k = 0; k < 4; ++k) {
            const unsigned n = V + k * (NSTAGE * 8u), bi = s ? n : (n / WREST) * 64u + NPUSH + n % WREST;
            const bool x = bi >= 512u, valid = s || n < 8u * WREST;
            const __amdgpu_buffer_rsrc_t rd = mk_brsrc(P.me, valid ? OFF_YL : 0u);
            const unsigned dofs = x ? OFF_SX + par * X_BYTES + s * (512u << 10) + (bi - 512u) * 4096u
                                    : OFF_SW + par * W_BYTES + s * (2u << 20) + (bi >> 1) * 8192u + (bi & 1u) * 2048u;
            const unsigned o1 = x ? 1024u : 4096u, o2 = x ? 2048u : 1024u;
#pragma unroll
            for (int i = 0; i < 4; ++i) st128<17>(r[k][i], rd, lane * 16u, valid ? dofs + (i & 1) * o1 + (i >> 1) * o2 : 0u);
        }
        WAIT_V(0);
        BAR();
        if (threadIdx.x < 8u) st_sys(flag, call * 16u + s + 2u);
    }
}

struct Batch { unsigned piece, so, dofs; };
__device__ __forceinline__ void mover(const Params& P, unsigned g, unsigned q, unsigned j, unsigned call, LAS char* lds) {
    const unsigned lane = threadIdx.x & 63u, w = rfl(threadIdx.x >> 6);
    const unsigned rank = (unsigned)P.rank;
    char* const me = P.me;
    char* const peer = P.wsg[g];
    volatile LAS unsigned* const lw = (volatile LAS unsigned*)lds;
    const unsigned ps = (lane >> 2) & 7u, pt = lane & 3u;
    const unsigned size = lane >= 32u ? 0u : (lane == 1u ? WREST : (pt == 1u ? 64u : (pt == 3u ? 16u : 48u)));
    const unsigned x0 = 112u * ps + 48u * min(ps, 8u - LAG) + 16u * min(ps, 8u - FILL) + (ps ? WREST : 0u);
    const unsigned start = lane == 1u ? 112u : pt < 2u ? x0 - 64u * pt
                         : (pt == 2u ? (ps < 8u - LAG ? x0 + 48u : NPULL - 16u * FILL - 48u * (8u - ps))
                                     : (ps < 8u - FILL ? x0 + (ps < 8u - LAG ? 96u : 48u) : NPULL - 16u * (8u - ps)));

    if (w == 0u) {
        GAS unsigned* const cnt = (GAS unsigned*)(me + OFF_FLG + q * 256u) + lane;
        const u32x4 rf = mk_rsrc(me + OFF_STG + q * 256u, 128u);
        const unsigned long long t0 = __builtin_amdgcn_s_memrealtime();
        unsigned seen = 0u, rounds = 1u;
        bool pushed = j != 0u;
        for (unsigned n = 0u;; ++n) {
            const unsigned gone = rfl(lw[63]), v = lw[lane], d = v - seen;
            if (lane < 32u && d != 0u && (rdl(v, 60u) >= start + size || gone != 0u)) {
                __hip_atomic_fetch_add(cnt, d, __ATOMIC_RELAXED, __HIP_MEMORY_SCOPE_SYSTEM);
                seen = v;
            }
            if (gone == MW && pushed) break;
            if ((n & 4095u) == 4095u && (rfl(ld_agent(me + OFF_TMO)) == call || __builtin_amdgcn_s_memrealtime() - t0 > DEADLINE_TICKS)) {
                if (lane == 0u) st_agent(me + OFF_TMO, call);
                break;
            }
            if (rounds < 9u || !pushed) {
                const unsigned sv = lw[64u + lane];
                if (!pushed && __ballot(lane - 24u < NPUSH / 7u && (int)(sv - call) < 0) == 0ull) {
                    if (lane == 1u) __hip_atomic_fetch_add(cnt, (unsigned)NPUSH, __ATOMIC_RELAXED, __HIP_MEMORY_SCOPE_SYSTEM);
                    pushed = true;
                }
                while (rounds < 9u && __ballot(lane < NSTAGE && (int)(sv - (call * 16u + rounds + 1u)) < 0) == 0ull) ++rounds;
                if (lane == 0u) lw[62] = rounds;
                if ((n & (rounds < 3u ? 3u : 7u)) == 0u)
                    asm volatile("s_mov_b32 m0, %0\n\ts_nop 0\n\tbuffer_load_dword %1, %2, 0 offen sc0 sc1 lds\n\ts_waitcnt vmcnt(24)"
                                 :: "s"((unsigned)(uintptr_t)lds + 256u), "v"(lane * 4u), "s"(rf) : "memory");
            }
            __builtin_amdgcn_s_sleep(1);
        }
        return;
    }

    if (w > MW) return;
    const unsigned par = call & 1u, vd = lane * 16u;
    const unsigned lo = min(rank, q), hi = max(rank, q);
    auto batch_of = [&](unsigned U) {
        Batch b;
        b.piece = (unsigned)__builtin_ctz((unsigned)__ballot(U - start < size) | 0x80000000u);
        const unsigned r = U - rdl(start, b.piece), s = b.piece >> 2, t = b.piece & 3u;
        if (t == 1u) {
            const unsigned cb = s ? r : r + NPUSH;
            b.so = OFF_SW + par * W_BYTES + s * (2u << 20) + rank * (256u << 10) + cb * 4096u;
            b.dofs = OFF_WS + q * WS_SRC + (s ? s : 8u * par) * (256u << 10) + cb * 4096u;
        } else {
            unsigned mb = r >> 3; mb += mb >= lo ? 1u : 0u; mb += mb >= hi ? 1u : 0u;
            const unsigned o = s * (512u << 10) + (t == 3u ? rank * 64u + r * 4u : mb * 64u + t * 16u + (r & 7u) * 4u) * 1024u;
            b.so = OFF_SX + par * X_BYTES + o;
            b.dofs = OFF_XS + q * XS_SRC + o;
        }
        b.piece = rfl(b.piece); b.so = rfl(b.so); b.dofs = rfl(b.dofs);
        asm("" : "+s"(b.piece), "+s"(b.so), "+s"(b.dofs));
        return b;
    };
    const __amdgpu_buffer_rsrc_t rd = mk_brsrc(me, OFF_SX);
    const __amdgpu_buffer_rsrc_t rc = mk_brsrc(me + OFF_CNT + q * 256u, 256u);
    const unsigned vc = lane == 0u ? 0u : 4096u;
    const unsigned tbase = (call - 1u) * NPULL - 3u * NMOVER;
    static_assert(LAG <= 7 && FILL <= 7 && NPUSH % 14 == 0 && NPUSH < 64, "");
    auto bump = [&](unsigned word) { if (lane == 0u) __hip_atomic_fetch_add((LAS unsigned*)lds + word, 1u, __ATOMIC_RELAXED, __HIP_MEMORY_SCOPE_WORKGROUP); };
    unsigned known = 1u;
    auto staged = [&](unsigned piece) {
        const unsigned round = piece < 4u ? (piece == 1u ? 1u : 0u) : (piece >> 2) + 1u;
        if (round < known) return true;
        const unsigned long long t0 = __builtin_amdgcn_s_memrealtime();
        for (unsigned n = 1u; (known = rfl(lw[62])) <= round; ++n) {
            if ((n & 1023u) == 0u && (rfl(ld_agent(me + OFF_TMO)) == call || __builtin_amdgcn_s_memrealtime() - t0 > DEADLINE_TICKS)) {
                if (lane == 0u) st_agent(me + OFF_TMO, call);
                return false;
            }
            __builtin_amdgcn_s_sleep(1);
        }
        asm volatile("" ::: "memory");
        return true;
    };
    auto pull = [&](unsigned so, bool valid) { return __builtin_amdgcn_raw_buffer_load_b128(mk_brsrc(peer, valid ? OFF_YL : 0u), vd, so, 17); };
    const unsigned W = j * MW + w - 1u;
    Batch b0 = batch_of(W);
    bool go = true;
    {
        const char* const fp = lane < 63u ? me + OFF_STG + q * 256u + (32u + min(lane, NHEAD - 1u)) * 4u : me + OFF_TMO;
        const unsigned long long t0 = __builtin_amdgcn_s_memrealtime();
        for (;;) {
            const unsigned v = ld_sys(fp);
            if (__ballot(lane < 63u && (int)(v - call) < 0) == 0ull) break;
            if (rdl(v, 63u) == call || __builtin_amdgcn_s_memrealtime() - t0 > DEADLINE_TICKS) {
                if (lane == 0u) st_agent(me + OFF_TMO, call);
                go = false; break;
            }
            __builtin_amdgcn_s_sleep(2);
        }
        asm volatile("" ::: "memory");
    }
    if (go) {
        u32x4 cur[4], nxt[4];
#pragma unroll
        for (int i = 0; i < 4; ++i) cur[i] = pull(b0.so + i * 1024u, true);
        unsigned T0 = W, T1 = NMOVER + W, T2 = 2u * NMOVER + W;
        Batch b1 = batch_of(T1);
        __builtin_amdgcn_s_waitcnt(0x0F70);
        for (;;) {
            const bool v1 = T1 < NPULL && staged(b1.piece);
#pragma unroll
            for (int i = 0; i < 4; ++i) nxt[i] = pull(b1.so + i * 1024u, v1);
            const unsigned tk = (unsigned)__builtin_amdgcn_raw_ptr_buffer_atomic_add_i32(1, rc, vc, 0, 0);
            asm volatile("" ::: "memory");
#pragma unroll
            for (int i = 0; i < 4; ++i) st128<16>(cur[i], rd, vd, b0.dofs + i * 1024u);
            if (T1 == NPULL - GO_LEAD && lane == 0u) st_agent(me + OFF_GO + q * 128u, call);
            const Batch b2 = batch_of(T2 < NPULL ? T2 : 0u);
            WAIT_V(0);
            bump(b0.piece);
            if (lane == 0u) __hip_atomic_fetch_max((LAS unsigned*)lds + 60, T0, __ATOMIC_RELAXED, __HIP_MEMORY_SCOPE_WORKGROUP);
            if (!v1) break;
#pragma unroll
            for (int i = 0; i < 4; ++i) cur[i] = nxt[i];
            b0 = b1; b1 = b2; T0 = T1; T1 = T2; T2 = rfl(tk) - tbase;
        }
    }
    bump(63u);
}

__device__ __forceinline__ void gemm_tile(const Params& P, unsigned grp, unsigned q, unsigned tile, unsigned call, LAS char* lds) {
    const unsigned lane = threadIdx.x & 63u, g16 = lane >> 4;
    const unsigned w = rfl(threadIdx.x >> 6), wm = w >> 1, wn = w & 1u;
    const unsigned rank = (unsigned)P.rank, g = (q - rank - 1u) & 7u;
    const unsigned h = tile & 1u, c = tile >> 1;
    const bool own = g == 7u;
    char* const me = P.me;
    char* const peer = P.wsg[grp];
    const unsigned par = call & 1u;
    const unsigned lds0 = (unsigned)(uintptr_t)lds;
    LAS unsigned* ctl = (LAS unsigned*)(lds + LDS_CTL);
    if (tile < (own ? NHEAD : NPUSH / 7u)) {
        WAIT_V(0);
        BAR();
        if (threadIdx.x < (own ? 8u : 1u)) st_sys((own ? P.wsg[lane] : peer) + OFF_STG + rank * 256u + ((own ? 32u : 24u) + tile) * 4u, call);
    }

    const unsigned nraw = own ? 8u : 16u, nsrc = own ? 7u : 6u;
    unsigned tabA, tabB, vF;
    unsigned long long gend;
    {
        const bool extra = !own && lane < 8u, raw = lane < nraw;
        const unsigned e = lane - nraw, s = raw ? lane & 7u : (own ? e / 7u : e / 6u), i = e - s * nsrc;
        const unsigned p = (rank + 1u + i + (i >= g ? 1u : 0u)) & 7u;
        tabA = raw ? ((extra ? rank : q) * 512u + h * 256u) * XROW + s * 128u : p * XS_SRC + s * (512u << 10) + (q * 2u + h) * 32768u;
        tabB = raw ? s * 64u * WROW + (extra ? q : rank) * 4096u + c * 320u : p * WS_SRC + (s ? s : 8u * par) * (256u << 10) + c * 20480u;
        vF = raw ? rank * 256u : p * 256u + s * 16u;
        gend = __ballot(raw ? lane == nraw - 1u : i == nsrc - 1u);
    }
    const unsigned b0 = w < 4u ? 3u * w : 12u + 2u * (w - 4u);
    const unsigned cbmax = c == 12u ? 3u : 4u;
    u32x4 ra, rb, rx;
    unsigned za, zb, zx1, vP[7];
    const unsigned zx0 = w == 7u ? 2048u : 0u, mx = w < 4u ? ~0u : 0u;
    auto staging = [&](bool raw) {
        ra = raw ? mk_rsrc(P.x, X_BYTES) : mk_rsrc(me + OFF_XS, 8u * XS_SRC);
        rb = raw ? mk_rsrc(P.w, W_BYTES) : mk_rsrc(me + OFF_WS, 8u * WS_SRC);
        rx = w < 4u ? rb : mk_rsrc(me + OFF_FLG, zx0);
        za = ra.z; zb = rb.z; zx1 = rx.z;
#pragma unroll
        for (int i = 0; i < 4; ++i) vP[i] = (raw ? (lane_x(lane) ^ ((i & 1) * 64u)) + (4u * w + i) * (8u * XROW) : lane * 16u + (4u * w + i) * 1024u) - i * 1024u;
#pragma unroll
        for (int i = 0; i < 3; ++i) { const unsigned b = b0 + i, cb = min(b >> 2, cbmax);
            vP[4 + i] = raw ? lane_w(lane) + cb * 64u + (b & 3u) * (16u * WROW) : lane * 16u + cb * 4096u + (b & 3u) * 1024u; }
        if (w >= 4u) vP[6] = vF;
    };
    staging(true);
    const unsigned la0 = lds0 + w * 4096u, lb0 = lds0 + 32768u + b0 * 1024u;
    const unsigned lx0 = w < 4u ? lb0 + 2048u : (w == 7u ? lds0 + 53248u : lds0 + 0x30000u), kx = (w < 4u || w == 7u) ? ~0u : 0u;
    unsigned oa = 0u, ob = 0u;
    auto piece = [&](int i, unsigned so) {
        if (i == 0) dma16<0, false>(ra, vP[0], oa, la0 + so);
        if (i == 1) dma16<1024, false>(ra, vP[1], oa, la0 + so);
        if (i == 2) dma16<2048, false>(ra, vP[2], oa, la0 + so);
        if (i == 3) dma16<3072, false>(ra, vP[3], oa, la0 + so);
        if (i == 4) dma16<0, false>(rb, vP[4], ob, lb0 + so);
        if (i == 5) dma16<0, false>(rb, vP[5], ob, lb0 + so + 1024u);
        if (i == 6) dma16<0, true>(rx, vP[6], ob & mx, lx0 + (so & kx));
    };
    unsigned it = 0u, ready = nraw;
    unsigned gm = 0u;
    auto decide = [&]() { asm volatile("s_cmp_lt_u32 %1, %2\n\ts_cselect_b32 %0, -1, 0" : "=s"(gm) : "s"(it), "s"(ready) : "scc"); };
    auto planA = [&]() { oa = rdl(tabA, it & 63u); ra.z = za & gm; asm volatile("" : "+s"(oa), "+s"(ra)); };
    auto planB = [&]() { ob = rdl(tabB, it & 63u); rb.z = zb & gm; asm volatile("" : "+s"(ob), "+s"(rb)); };
    auto planX = [&]() { rx.z = zx1 & gm & (it + 3u == ready ? ~0u : mx); it -= gm; asm volatile("" : "+s"(it), "+s"(rx)); };
    auto plan = [&]() { decide(); planA(); planB(); planX(); asm volatile("s_nop 4"); };
    const unsigned xw = own ? 12u : 8u * h, needx = (own ? 16u : 48u) * call, needw = 64u * call;
    auto arrived = [&](unsigned vw, unsigned vx) {
        const unsigned long long miss = __ballot((int)(vw - needw) < 0 || (int)(vx - needx) < 0) >> nraw << nraw;
        return 64u - (unsigned)__builtin_clzll(gend & ((miss & (0ull - miss)) - 1ull));
    };

    const unsigned aoff = wm * 8192u + (lane & 15u) * 128u + ((g16 ^ ((lane & 15u) >> 1)) * 16u);
    const unsigned lp = (g16 >> 1) * 1024u + (8u * (g16 & 1u) + ((lane & 15u) >> 2)) * 64u + (lane & 3u) * 8u;
    const unsigned be = 32768u + ((5u * wn) >> 1) * 4096u + lp + (((wn ^ g16) & 1u) * 32u);
    const unsigned bo = 32768u + ((5u * wn + 1u) >> 1) * 4096u + lp + (((wn ^ g16 ^ 1u) & 1u) * 32u);

    f32x4 acc[4][5];
#pragma unroll
    for (int m = 0; m < 4; ++m)
#pragma unroll
        for (int n = 0; n < 5; ++n) acc[m][n] = (f32x4){0.f, 0.f, 0.f, 0.f};
    const __amdgpu_buffer_rsrc_t rv = mk_brsrc(me + OFF_SV + ((min(g, 6u) * NTILE + tile) * 8u + w) * SV_WAVE, SV_WAVE);
    s16x8 fa[2][4]; s16x4 fb[2][5][2];
    auto zero1 = [&]() {
#pragma unroll
        for (int m = 0; m < 4; ++m) fa[1][m] = (s16x8){0, 0, 0, 0, 0, 0, 0, 0};
#pragma unroll
        for (int n = 0; n < 5; ++n) { fb[1][n][0] = (s16x4){0, 0, 0, 0}; fb[1][n][1] = (s16x4){0, 0, 0, 0}; }
    };
    auto rd = [&](int kh, int r, LAS char* st) {
        if (r >= 2 && r < 6) fa[kh][r - 2] = *(const LAS s16x8*)(st + (kh ? (aoff ^ 64u) : aoff) + (r - 2) * 2048);
        else { const int n = r < 2 ? 0 : (r - 4) >> 1, hi = r & 1;
               fb[kh][n][hi] = __builtin_amdgcn_ds_read_tr16_b64_v4i16((LAS s16x4*)(st + ((n & 1) ? bo : be) + (n >> 1) * 4096 + kh * 2048 + hi * 256)); }
    };
    auto mm = [&](int kh, int i) {
        const int n = i >> 2, m = i & 3;
        acc[m][n] = __builtin_amdgcn_mfma_f32_16x16x32_bf16(
            __builtin_bit_cast(bf16x8, __builtin_shufflevector(fb[kh][n][0], fb[kh][n][1], 0, 1, 2, 3, 4, 5, 6, 7)),
            __builtin_bit_cast(bf16x8, fa[kh][m]), acc[m][n], 0, 0, 0);
    };
    zero1();

    unsigned t = 0u;
    unsigned s0 = 0u, s1 = STAGE, s2 = 2u * STAGE;
    bool bad = false;
    for (;;) {
        if (it == t) {
            plan();
#pragma unroll
            for (int i = 0; i < 7; ++i) piece(i, s0);
            plan();
#pragma unroll
            for (int i = 0; i < 7; ++i) piece(i, s1);
            plan();
        }
        const unsigned park = (!own && t < 8u) ? 8u : 64u;
        do {
            LAS char* st = lds + s0;
#pragma unroll
            for (int i = 0; i < 20; ++i) {
                if (i == 2) {
                    __builtin_amdgcn_s_waitcnt(0xC07F);
                    WAIT_V(7);
                    BAR();
                    __builtin_amdgcn_sched_barrier(0);
                    __builtin_amdgcn_s_setprio(3);
                }
                if (i == 12) __builtin_amdgcn_s_setprio(2);
                mm(1, i);
                if (i >= 2 && i < 16) rd(0, i - 2, st);
                if (i % 5 == 4) piece(i / 5, s2);
                if (i == 17 && __builtin_expect(it == ready && it == t + 3u && t >= nraw, 0)) {
                    const LAS char* fr = st + 53248u + lane * 16u;
                    ready = max(ready, arrived(*(const LAS unsigned*)(fr + 4), *(const LAS unsigned*)(fr + xw)));
                }
                if (i == 18) decide();
                __builtin_amdgcn_sched_barrier(0);
            }
#pragma unroll
            for (int i = 0; i < 20; ++i) {
                if (i == 2) __builtin_amdgcn_s_setprio(1);
                if (i == 12) __builtin_amdgcn_s_setprio(0);
                mm(0, i);
                if (i < 14) rd(1, i, st);
                if (i % 5 == 4 && i < 15) piece(4 + i / 5, s2);
                if (i == 17) planA();
                if (i == 18) planB();
                if (i == 19) planX();
                __builtin_amdgcn_sched_barrier(0);
            }
            { const unsigned x = s0; s0 = s1; s1 = s2; s2 = x; }
            ++t;
        } while (t < min(it, park));
        if (t == 64u) break;
        if (t < it) {
#pragma unroll
            for (int i = 0; i < 20; ++i) mm(1, i);
#pragma unroll
            for (int m = 0; m < 4; ++m)
#pragma unroll
                for (int n = 0; n < 5; ++n) { st128<0>(__builtin_bit_cast(u32x4, acc[m][n]), rv, lane * 16u, (n * 4 + m) * 1024); acc[m][n] = (f32x4){0.f, 0.f, 0.f, 0.f}; }
            zero1();
            continue;
        }
        if (t == nraw) staging(false);
        if (w == 0u) {
            const char* fp = me + OFF_FLG + vF;
            const unsigned long long t0 = __builtin_amdgcn_s_memrealtime();
            unsigned res;
            for (unsigned n = 1u;; ++n) {
                const unsigned vw = ld_sys(fp + 4), vx = ld_sys(fp + xw);
                res = arrived(vw, vx);
                if (res > t) break;
                if ((n & 15u) == 0u && (rfl(ld_agent(me + OFF_TMO)) == call || __builtin_amdgcn_s_memrealtime() - t0 > DEADLINE_TICKS)) {
                    st_agent(me + OFF_TMO, call);
                    res = 0x100u + 64u; break;
                }
                __builtin_amdgcn_s_sleep(4);
            }
            ctl[0] = res;
            asm volatile("s_waitcnt vmcnt(0) lgkmcnt(0)" ::: "memory");
        }
        BAR();
        const unsigned res = rfl(*(volatile LAS unsigned*)ctl);
        bad |= res > 0xffu;
        ready = res & 0xffu;
    }
#pragma unroll
    for (int i = 0; i < 20; ++i) mm(1, i);

    const unsigned nu = (c == 12u && wn == 1u) ? 6u : 10u;
    const __amdgpu_buffer_rsrc_t ro = mk_brsrc(P.out, OUT_BYTES);
    const unsigned vo = ((g16 & 1u) * 16u + (lane & 15u)) * WROW + (g16 >> 1) * 16u;
    const unsigned vo2 = nu == 10u ? vo : 0x80000000u;
    unsigned so0 = (h * 256u + wm * 64u) * WROW + q * 4096u + c * 320u + wn * 160u, so1 = so0 + 32u * WROW;
    asm("" : "+s"(so0), "+s"(so1));
    auto emit = [&](int u, const f32x4& v0, const f32x4& v1) {
        const unsigned a0 = pk_bf16(silu(v0[0]), silu(v0[1])), a1 = pk_bf16(silu(v0[2]), silu(v0[3]));
        const unsigned c0 = pk_bf16(silu(v1[0]), silu(v1[1])), c1 = pk_bf16(silu(v1[2]), silu(v1[3]));
        const auto x0 = __builtin_amdgcn_permlane16_swap(a0, c0, false, false);
        const auto x1 = __builtin_amdgcn_permlane16_swap(a1, c1, false, false);
        u32x4 d = {x0[0], x1[0], x0[1], x1[1]};
        if (bad) d = (u32x4){0x7fc07fc0u, 0x7fc07fc0u, 0x7fc07fc0u, 0x7fc07fc0u};
        st128<16>(d, ro, (u < 6 ? vo : vo2) + (u >> 1) * 32u, (u & 1) ? so1 : so0);
    };
    if (own) {
        bad |= rfl(ld_agent(me + OFF_TMO)) == call;
#pragma unroll
        for (int u = 0; u < 10; ++u) emit(u, acc[2 * (u & 1)][u >> 1], acc[2 * (u & 1) + 1][u >> 1]);
        return;
    }
    const unsigned yo = OFF_YL + par * Y_PAR + (tile * 8u + w) * Y_WAVE;
    {
        const __amdgpu_buffer_rsrc_t ry = mk_brsrc(me + yo + q * Y_PEER, Y_WAVE);
#pragma unroll
        for (int u = 0; u < 10; ++u) {
            const f32x4 v0 = acc[2 * (u & 1)][u >> 1], v1 = acc[2 * (u & 1) + 1][u >> 1];
            u32x4 d = {pk_bf16(v0[0], v0[1]), pk_bf16(v0[2], v0[3]), pk_bf16(v1[0], v1[1]), pk_bf16(v1[2], v1[3])};
            if (bad) d = (u32x4){0x7fc07fc0u, 0x7fc07fc0u, 0x7fc07fc0u, 0x7fc07fc0u};
            st128<17>(d, ry, lane * 16u, u * 1024);
        }
        WAIT_V(0);
        BAR();
        if (threadIdx.x == 0u) st_sys(me + OFF_YF + q * 4096u + tile * 128u, call);
    }
    if (w == 0u) {
        const char* fp = lane == 0u ? peer + OFF_YF + rank * 4096u + tile * 128u : (lane == 1u ? me + OFF_GO + q * 128u : me + OFF_TMO);
        const unsigned long long t0 = __builtin_amdgcn_s_memrealtime();
        bool have = false;
        for (;;) {
            const unsigned v = ld_sys(fp);
            if (rdl(v, 2u) == call) { bad = true; break; }
            const unsigned m = (unsigned)__ballot((int)(v - call) >= 0);
            if (!have && (m & 1u)) { have = true; if (lane == 0u) fp = me + OFF_TMO; }
            if (have && (m & 2u)) break;
            if (__builtin_amdgcn_s_memrealtime() - t0 > DEADLINE_TICKS) {
                if (lane == 0u) st_agent(me + OFF_TMO, call);
                bad = true; break;
            }
            __builtin_amdgcn_s_sleep(2);
        }
        if (lane == 0u) ctl[0] = bad ? 1u : 0u;
        asm volatile("s_waitcnt lgkmcnt(0)" ::: "memory");
    }
    BAR();
    bad |= rfl(*(volatile LAS unsigned*)ctl) != 0u;
    const __amdgpu_buffer_rsrc_t rp = mk_brsrc(peer + yo + rank * Y_PEER, bad ? 0u : nu * 1024u);
    u32x4 pv[10];
    auto fetch = [&](int u) {
        unsigned n = 0u;
        if (lane == 0u) n = __hip_atomic_fetch_add(ctl + 4, 1u, __ATOMIC_RELAXED, __HIP_MEMORY_SCOPE_WORKGROUP);
        n = rfl(n);
        for (unsigned k = 0u; n >= rfl(*(volatile LAS unsigned*)(ctl + 5)) + TOK && k < (1u << 20); ++k) __builtin_amdgcn_s_sleep(1);
        pv[u] = __builtin_amdgcn_raw_buffer_load_b128(rp, lane * 16u + u * 1024u, 0, 17);
        asm volatile("" ::: "memory");
#pragma unroll
        for (int k = 0; k < 2; ++k) acc[2 * (u & 1) + k][u >> 1] = __builtin_bit_cast(f32x4, __builtin_amdgcn_raw_buffer_load_b128(rv, lane * 16u, ((u >> 1) * 4 + 2 * (u & 1) + k) * 1024, 0));
        asm volatile("" ::: "memory");
    };
    fetch(0);
#pragma unroll
    for (int u = 0; u < 10; ++u) {
        asm volatile("" : "+v"(pv[u]));
        if (lane == 0u) __hip_atomic_fetch_add(ctl + 5, 1u, __ATOMIC_RELAXED, __HIP_MEMORY_SCOPE_WORKGROUP);
        if (u < 9) fetch(u + 1);
        __builtin_amdgcn_sched_barrier(0);
        const u32x4 d = pv[u];
        f32x4 v0 = acc[2 * (u & 1)][u >> 1], v1 = acc[2 * (u & 1) + 1][u >> 1];
        v0[0] += __builtin_bit_cast(float, d.x << 16); v0[1] += __builtin_bit_cast(float, d.x & 0xffff0000u);
        v0[2] += __builtin_bit_cast(float, d.y << 16); v0[3] += __builtin_bit_cast(float, d.y & 0xffff0000u);
        v1[0] += __builtin_bit_cast(float, d.z << 16); v1[1] += __builtin_bit_cast(float, d.z & 0xffff0000u);
        v1[2] += __builtin_bit_cast(float, d.w << 16); v1[3] += __builtin_bit_cast(float, d.w & 0xffff0000u);
        emit(u, v0, v1);
        __builtin_amdgcn_sched_barrier(0);
    }
}

__global__ void __launch_bounds__(512, 2) fused_gemm_rs(Params P) {
    __shared__ __attribute__((aligned(16))) char smem[LDS_BYTES];
    LAS char* lds = (LAS char*)smem;
    volatile LAS unsigned* const lw = (volatile LAS unsigned*)lds;
    const unsigned w = rfl(threadIdx.x >> 6), rank = (unsigned)P.rank;
    const unsigned guess = (blockIdx.x & ~7u) | ((unsigned)__builtin_amdgcn_s_getreg((3 << 11) | 20) & 7u);
    auto head = [&](unsigned role) {
        const unsigned slot = role >> 3, q = (P.qmap >> ((role & 7u) * 4u)) & 7u;
        if (w == 0u || slot >= 32u) return;
        if (slot < NSW + NPW) return;
        const unsigned W = (slot - NSW - NPW) * 7u + w - 1u;
        if (q != rank) { if (W < NPUSH) push_head(P, role & 7u, q, W); }
        else if (W < 128u) stage_head(P, W);
    };
    if (w == 0u) {
        char* const me = P.me;
        const __amdgpu_buffer_rsrc_t rc = mk_brsrc(me + OFF_CLM, 32768u);
        const unsigned v0 = threadIdx.x == 0u ? 0u : 65536u;
        unsigned role = guess, old = (unsigned)__builtin_amdgcn_raw_ptr_buffer_atomic_add_i32(1, rc, v0, role * 128u, 0);
        const unsigned done = rfl(ld_agent(me + OFF_CTR + blockIdx.x * 128u));
        lw[threadIdx.x] = 0u; lw[64u + threadIdx.x] = 0u;
        while (rfl(old) != done) {
            __builtin_amdgcn_raw_ptr_buffer_atomic_add_i32(-1, rc, v0, role * 128u, 0);
            role = (role + 1u) & 255u;
            if (role == guess) { role = 256u; st_agent(me + OFF_TMO, done + 1u); break; }
            old = (unsigned)__builtin_amdgcn_raw_ptr_buffer_atomic_add_i32(1, rc, v0, role * 128u, 0);
        }
        if (threadIdx.x == 0u) { lw[LDS_CTL / 4u + 1u] = role; lw[LDS_CTL / 4u + 2u] = done + 1u; lw[LDS_CTL / 4u + 4u] = 0u; lw[LDS_CTL / 4u + 5u] = 0u; }
        __builtin_amdgcn_s_waitcnt(0x0070);
    } else head(guess);
    BAR();
    const unsigned role = rfl(lw[LDS_CTL / 4u + 1u]), call = rfl(lw[LDS_CTL / 4u + 2u]);
    if (role != guess) { head(role); BAR(); }
    if (threadIdx.x == 0u) st_agent(P.me + OFF_CTR + blockIdx.x * 128u, call);
    const unsigned q = (P.qmap >> ((role & 7u) * 4u)) & 7u, slot = role >> 3;
    if (slot >= 32u) return;
    if (slot >= NSW + NPW) gemm_tile(P, role & 7u, q, slot - NSW - NPW, call, lds);
    else if (slot >= NSW && q != rank) mover(P, role & 7u, q, slot - NSW, call, lds);
    else stager(P, slot < NSW ? role : 8u * NSW + slot - NSW, call);
}

unsigned make_qmap(int rank) {
    static const unsigned char kNear[NR][NR] = {
        {6, 0, 2, 3, 1, 5, 4, 7}, {7, 1, 4, 5, 0, 3, 6, 2}, {3, 2, 0, 6, 4, 5, 7, 1}, {2, 3, 6, 0, 5, 1, 7, 4},
        {5, 4, 1, 7, 2, 0, 6, 3}, {4, 5, 7, 1, 3, 2, 0, 6}, {0, 6, 3, 2, 7, 4, 1, 5}, {1, 7, 5, 4, 6, 3, 2, 0}};
    unsigned m = 0u;
    for (int g = 0; g < NR; ++g) m |= (unsigned)kNear[rank][g] << (4 * g);
    return m;
}

}

extern "C" void kernel_launch(int rank, int nranks, ncclComm_t comm,
                              hipStream_t stream,
                              void* const* d_in, const int* in_sizes, int n_in,
                              void* d_out, int out_size,
                              const int* in_dtypes, int out_dtype,
                              void* const* d_out_all,
                              void* const* d_ws_all, size_t ws_bytes) {
    (void)comm; (void)in_dtypes; (void)out_dtype; (void)d_out_all;
    if (nranks != NR || (unsigned)rank >= (unsigned)NR || n_in != 2 || ws_bytes < (size_t)WS_NEED ||
        in_sizes[0] != 4096 * 512 || in_sizes[1] != 512 * 16384 || out_size != 512 * 16384) {
        fprintf(stderr, "unexpected problem geometry\n");
        return;
    }
    Params P;
    P.x = (const char*)d_in[0];
    P.w = (const char*)d_in[1];
    P.out = (char*)d_out;
    P.me = (char*)d_ws_all[rank];
    P.rank = rank;
    P.qmap = make_qmap(rank);
    for (int g = 0; g < NR; ++g) P.wsg[g] = (char*)d_ws_all[(P.qmap >> (4 * g)) & 7u];
    fused_gemm_rs<<<GRID, 512, 0, stream>>>(P);
    HIP_CHECK(hipGetLastError());
}
```
